```python
import numpy as np

kernel_name = "gpu_a2a_gemm_m4096_k4096_n4096_e5m2e5m2_dqrelu"


ABSMAX_RTOL = 0.05

COMPUTE_DTYPE = "float32"
EPILOGUE = "dequant_relu"
IN_QUANT = None
OUT_QUANT = None
IN_FP8 = "e5m2e5m2"
IN_DTYPES = {"x": "e5m2", "w_mat": "e5m2",
             "scale_x": "float32", "scale_w": "float32"}
WORLD_SIZE = 8

def _snap_fp8(x, mant_bits, max_abs, min_normal):
    a = np.abs(x.astype(np.float64))
    _m, _e = np.frexp(a)
    q = np.where(a >= min_normal, np.ldexp(1.0, _e - 1 - mant_bits),
                 min_normal * 2.0 ** -mant_bits)
    snapped = np.minimum(np.round(a / q) * q, max_abs)
    return (np.sign(x) * snapped).astype(np.float32)

def setup_inputs():

    x = _snap_fp8(np.random.randn(4096, 4096).astype(np.float32), 2, 57344.0, 2.0 ** -14)
    w_mat = _snap_fp8(np.random.randn(4096, 4096).astype(np.float32), 2, 57344.0, 2.0 ** -14)
    scale_x = (0.02 * (1.0 + np.random.rand(1))).astype(np.float32)
    scale_w = (0.02 * (1.0 + np.random.rand(1))).astype(np.float32)
    return {"x": x, "w_mat": w_mat, "scale_x": scale_x, "scale_w": scale_w}

def reference(x, w_mat, scale_x, scale_w):
    acc = (x @ w_mat).astype(np.float32)
    y = (acc * (scale_x * scale_w)).astype(np.float32)
    return np.maximum(y, 0.0).astype(np.float32)
```

```cpp
#include <hip/hip_runtime.h>
#include <rccl/rccl.h>
#include <cstdio>

#define HIP_CHECK(expr) do { hipError_t _e = (expr); if (_e != hipSuccess) { \
    fprintf(stderr, "HIP error %s at %s:%d\n", hipGetErrorString(_e), __FILE__, __LINE__); \
    return; } } while (0)

#define GAS __attribute__((address_space(1)))
#define LAS __attribute__((address_space(3)))
typedef int v2i __attribute__((ext_vector_type(2)));
typedef int v4i __attribute__((ext_vector_type(4)));
typedef int v8i __attribute__((ext_vector_type(8)));
typedef unsigned v4u __attribute__((ext_vector_type(4)));
typedef float v16f __attribute__((ext_vector_type(16)));

constexpr int NR = 8;
constexpr int GN = 4096;
constexpr int GK = 4096;
constexpr int SH = 512;
constexpr int NWG = 256;
constexpr int NTHREADS = 7 * 64;
constexpr int NSTEP = 64;
constexpr int NLOC = 8;
constexpr int NREM = NSTEP - NLOC;
constexpr int FRAG = 2048;

constexpr size_t WS_CTR = 0;
constexpr size_t WS_TMO = 64 * 1024;
constexpr size_t WS_FLAG = 128 * 1024;
constexpr size_t WS_DBG = 256 * 1024;
constexpr size_t WS_A = 1024 * 1024;
constexpr size_t A_PAR = (size_t)16 * NREM * FRAG;
constexpr size_t WS_END = WS_A + 2 * A_PAR;

constexpr int RING = 32;
constexpr int SLOT = 4096;
constexpr int L_SYNC = RING * SLOT;
constexpr int LDS_BYTES = L_SYNC + 256;
constexpr int LAG = 8;

constexpr unsigned long long DEADLINE_TICKS = 40000000ull;

struct Params {
    const unsigned char* x;
    const unsigned char* w;
    const float* sx;
    const float* sw;
    float* out;
    unsigned char* ws[NR];
    int rank;
};

#define STAMP(slot) do {} while (0)

__device__ __forceinline__ unsigned lds_ld(LAS unsigned* p) {
    return __builtin_amdgcn_readfirstlane(*(volatile LAS unsigned*)p);
}

__device__ __forceinline__ void glds16(const unsigned char* gsrc, unsigned lds_dst) {
    unsigned keep;
    asm volatile("s_mov_b32 %0, m0\n\ts_mov_b32 m0, %2\n\ts_nop 0\n\tglobal_load_lds_dwordx4 %1, off\n\ts_mov_b32 m0, %0"
                 : "=&s"(keep) : "v"(gsrc), "s"(lds_dst) : "memory");
}

__global__ void __launch_bounds__(NTHREADS, 2) a2a_gemm_kernel(Params p) {
    extern __shared__ __attribute__((aligned(16))) unsigned char smem[];
    LAS unsigned char* L8 = (LAS unsigned char*)smem;
    LAS unsigned* sync = (LAS unsigned*)(L8 + L_SYNC);

    const int tid = threadIdx.x;
    const int lane = tid & 63;
    const int wave = __builtin_amdgcn_readfirstlane(tid >> 6);
    const int b = blockIdx.x;
    const int rank = p.rank;
    unsigned char* myws = p.ws[rank];

    const int xg = b & 7, idx = b >> 3;
    const int rb = idx & 3;
    const int n0 = (xg * 8 + (idx >> 2)) * 64;

    STAMP(wave);
    GAS unsigned* ctr = (GAS unsigned*)(myws + WS_CTR) + b * 32 + wave;
    const unsigned callv = __hip_atomic_load(ctr, __ATOMIC_RELAXED, __HIP_MEMORY_SCOPE_AGENT);

    if (wave == 6) sync[lane] = 0u;

    if (wave < 4) {
        const int rg = rb * 4 + wave;
        const auto xr = __builtin_amdgcn_make_buffer_rsrc((void*)p.x, 0, 4096 * SH, 0x00020000);
        const auto ar = __builtin_amdgcn_make_buffer_rsrc((void*)(myws + WS_A), 0, (int)(2 * A_PAR), 0x00020000);
        v4i af[8][2];
        {
            const int voff = (lane & 31) * SH + (lane >> 5) * 32;
            const int soff = (rank * SH + rg * 32) * SH;
#pragma unroll
            for (int t = 0; t < 8; ++t) {
                af[t][0] = __builtin_bit_cast(v4i, __builtin_amdgcn_raw_buffer_load_b128(xr, voff + t * 64, soff, 0));
                af[t][1] = __builtin_bit_cast(v4i, __builtin_amdgcn_raw_buffer_load_b128(xr, voff + t * 64 + 16, soff, 0));
            }
        }
        asm volatile("s_barrier" ::: "memory");
        const unsigned call = __builtin_amdgcn_readfirstlane(callv) + 1u;
        if (lane == 0) __hip_atomic_store(ctr, call, __ATOMIC_RELAXED, __HIP_MEMORY_SCOPE_AGENT);
        __builtin_amdgcn_s_setprio(2);

        int asoff = (int)((call & 1u) * A_PAR) + rg * NREM * FRAG;
        const int avoff = lane * 16;
        const int g = lane >> 4, q = (lane & 15) >> 1;
        const int rd0 = ((g >> 1) * 32 + q) * 64 + (((g & 1) * 16 + (lane & 1) * 8) | ((q >> 2) << 5));
        const int rd1 = rd0 ^ 32;
        v16f acc0 = {}, acc1 = {};
        unsigned wrdy = 0, ardy = 0;
        v2i bf[2][8];

#define LOADB(dst, step_slot_off) do { \
        _Pragma("unroll") for (int u = 0; u < 4; ++u) { \
            dst[u] = __builtin_amdgcn_ds_read_tr8_b64_v2i32((LAS v2i*)(L8 + base0 + (step_slot_off) + u * 512)); \
            dst[4 + u] = __builtin_amdgcn_ds_read_tr8_b64_v2i32((LAS v2i*)(L8 + base1 + (step_slot_off) + u * 512)); \
        } } while (0)
#define WAITW(need) do { if (wrdy < (unsigned)(need)) { \
            do { wrdy = lds_ld(sync + 0); } while (wrdy < (unsigned)(need)); } \
        asm volatile("" ::: "memory"); } while (0)

        int base0 = rd0, base1 = rd1;
        WAITW(1);
        STAMP(8 + wave);
        LOADB(bf[0], 0);
        for (int it = 0; it < 8; ++it) {
            const int t0 = it * 8;
#pragma unroll
            for (int s = 0; s < 8; ++s) {
                const int t = t0 + s;
                if (s == 7) {
                    base0 = (base0 + 8 * SLOT) & (RING * SLOT - 1);
                    base1 = (base1 + 8 * SLOT) & (RING * SLOT - 1);
                    WAITW(min(t + 2, NSTEP));
                    LOADB(bf[0], 0);
                } else {
                    WAITW(t + 2);
                    LOADB(bf[(s + 1) & 1], (s + 1) * SLOT);
                }
                if ((s & 3) == 3 && lane == 0) *(volatile LAS unsigned*)(sync + 32 + wave) = (unsigned)(t + 1);
                const v8i a = __builtin_shufflevector(af[s][0], af[s][1], 0, 1, 2, 3, 4, 5, 6, 7);
                const v2i* bb = bf[s & 1];
                const v8i b0 = {bb[0].x, bb[0].y, bb[1].x, bb[1].y, bb[2].x, bb[2].y, bb[3].x, bb[3].y};
                const v8i b1 = {bb[4].x, bb[4].y, bb[5].x, bb[5].y, bb[6].x, bb[6].y, bb[7].x, bb[7].y};
                acc0 = __builtin_amdgcn_mfma_scale_f32_32x32x64_f8f6f4(a, b0, acc0, 1, 1, 0, 0, 0, 0);
                acc1 = __builtin_amdgcn_mfma_scale_f32_32x32x64_f8f6f4(a, b1, acc1, 1, 1, 0, 0, 0, 0);
                {
                    if (ardy <= (unsigned)t) {
                        do { ardy = lds_ld(sync + 16 + wave); } while (ardy <= (unsigned)t);
                    }
                    asm volatile("" ::: "memory");
                    af[s][0] = __builtin_bit_cast(v4i, __builtin_amdgcn_raw_buffer_load_b128(ar, avoff, asoff + s * FRAG, 16));
                    af[s][1] = __builtin_bit_cast(v4i, __builtin_amdgcn_raw_buffer_load_b128(ar, avoff + 1024, asoff + s * FRAG, 16));
                }
            }
            asoff = it < 6 ? asoff + 8 * FRAG : (int)(2 * A_PAR);
            STAMP(16 + wave * 8 + it);
        }
        const float scale = p.sx[0] * p.sw[0];
        const bool bad = lds_ld(sync + 20) != 0u;
        GAS float* o = (GAS float*)p.out + (size_t)(rg * 32 + 4 * (lane >> 5)) * GN + n0 + (lane & 31);
#pragma unroll
        for (int r = 0; r < 16; ++r) {
            float y0 = acc0[r] * scale, y1 = acc1[r] * scale;
            y0 = y0 < 0.f ? 0.f : y0;
            y1 = y1 < 0.f ? 0.f : y1;
            if (bad) { y0 = __builtin_nanf(""); y1 = y0; }
            const int row = (r & 3) + 8 * (r >> 2);
            o[(size_t)row * GN] = y0;
            o[(size_t)row * GN + 32] = y1;
        }
        STAMP(12 + wave);
    } else if (wave == 4) {
        const bool active = b < 224;
        const int jh = b >= 112 ? 1 : 0;
        const int e = active ? b - jh * 112 : 0;
        const int pp = e % 7 + 1, rg = e / 7;
        const int dst = (rank + pp) & 7;
        const auto xr = __builtin_amdgcn_make_buffer_rsrc((void*)p.x, 0, 4096 * SH, 0x00020000);
        v4u d[4][2];
        const int voff = (lane & 31) * SH + (lane >> 5) * 32;
        const int soff = (dst * SH + rg * 32) * SH + jh * 64;
#pragma unroll
        for (int u = 0; u < 4; ++u) {
            d[u][0] = __builtin_amdgcn_raw_buffer_load_b128(xr, voff + u * 128, soff, 0);
            d[u][1] = __builtin_amdgcn_raw_buffer_load_b128(xr, voff + u * 128 + 16, soff, 0);
        }
        asm volatile("s_barrier" ::: "memory");
        const unsigned call = __builtin_amdgcn_readfirstlane(callv) + 1u;
        if (active) {
            unsigned char* pw = p.ws[dst];
            const auto dr = __builtin_amdgcn_make_buffer_rsrc((void*)(pw + WS_A), 0, (int)(2 * A_PAR), 0x00020000);
            const int i0 = jh * 7 + (7 - pp);
            const int dso = (int)((call & 1u) * A_PAR) + (rg * NREM + i0) * FRAG;
#pragma unroll
            for (int u = 0; u < 4; ++u) {
                __builtin_amdgcn_raw_buffer_store_b128(d[u][0], dr, lane * 16, dso + u * 14 * FRAG, 17);
                __builtin_amdgcn_raw_buffer_store_b128(d[u][1], dr, lane * 16 + 1024, dso + u * 14 * FRAG, 17);
            }
            STAMP(48);
            GAS unsigned* fl = (GAS unsigned*)(pw + WS_FLAG) + ((rg >> 2) * 64 + i0) * 4 + (rg & 3);
            asm volatile("s_waitcnt vmcnt(6)" ::: "memory");
            if (lane == 0) __hip_atomic_store(fl, call, __ATOMIC_RELAXED, __HIP_MEMORY_SCOPE_SYSTEM);
            asm volatile("s_waitcnt vmcnt(5)" ::: "memory");
            if (lane == 0) __hip_atomic_store(fl + 14 * 4, call, __ATOMIC_RELAXED, __HIP_MEMORY_SCOPE_SYSTEM);
            asm volatile("s_waitcnt vmcnt(4)" ::: "memory");
            if (lane == 0) __hip_atomic_store(fl + 28 * 4, call, __ATOMIC_RELAXED, __HIP_MEMORY_SCOPE_SYSTEM);
            asm volatile("s_waitcnt vmcnt(3)" ::: "memory");
            if (lane == 0) __hip_atomic_store(fl + 42 * 4, call, __ATOMIC_RELAXED, __HIP_MEMORY_SCOPE_SYSTEM);
        }
        if (lane == 0) __hip_atomic_store(ctr, call, __ATOMIC_RELAXED, __HIP_MEMORY_SCOPE_AGENT);
    } else if (wave == 5) {
        const unsigned char* src = p.w + (size_t)(lane >> 2) * GN + n0 + (((lane & 3) ^ (((lane >> 4) & 1) << 1)) * 16);
        const unsigned lbase = (unsigned)(size_t)L8;
        int jj = 0, qq = 0;
        unsigned fre = 0;
        for (int t = 0; t < NSTEP; ++t) {
            int kb;
            if (t < NLOC) {
                kb = rank * SH + t * 64;
            } else {
                kb = ((rank + 1 + qq) & 7) * SH + jj * 64;
                if (++qq == 7) { qq = 0; ++jj; }
            }
            if (t >= RING && fre < (unsigned)(t - RING + 1)) {
                asm volatile("s_waitcnt vmcnt(0)" ::: "memory");
                if (lane == 0) *(volatile LAS unsigned*)(sync + 0) = (unsigned)t;
                do {
                    const v4u c = *(volatile LAS v4u*)(sync + 32);
                    fre = __builtin_amdgcn_readfirstlane(min(min(c.x, c.y), min(c.z, c.w)));
                    if (fre < (unsigned)(t - RING + 1)) __builtin_amdgcn_s_sleep(1);
                } while (fre < (unsigned)(t - RING + 1));
            }
            const unsigned char* s0 = src + (size_t)kb * GN;
            const unsigned ld = lbase + (t & (RING - 1)) * SLOT;
#pragma unroll
            for (int u = 0; u < 4; ++u) glds16(s0 + (size_t)u * 16 * GN, ld + u * 1024);
            if (t == 0) asm volatile("s_barrier" ::: "memory");
            if (t >= LAG) {
                asm volatile("s_waitcnt vmcnt(%0)" :: "i"(LAG * 4) : "memory");
                if (lane == 0) __hip_atomic_fetch_max(sync + 0, (unsigned)(t - LAG + 1), __ATOMIC_RELAXED, __HIP_MEMORY_SCOPE_WORKGROUP);
            }
        }
#pragma unroll
        for (int r = LAG - 1; r >= 0; --r) {
            asm volatile("s_waitcnt vmcnt(%0)" :: "i"(r * 4) : "memory");
            if (lane == 0) __hip_atomic_fetch_max(sync + 0, (unsigned)(NSTEP - r), __ATOMIC_RELAXED, __HIP_MEMORY_SCOPE_WORKGROUP);
        }
        STAMP(50);
        if (lane == 0) __hip_atomic_store(ctr, callv + 1u, __ATOMIC_RELAXED, __HIP_MEMORY_SCOPE_AGENT);
    } else {
        asm volatile("s_waitcnt lgkmcnt(0)\n\ts_barrier" ::: "memory");
        const unsigned call = __builtin_amdgcn_readfirstlane(callv) + 1u;
        if (lane == 0) __hip_atomic_store(ctr, call, __ATOMIC_RELAXED, __HIP_MEMORY_SCOPE_AGENT);
        const auto fr = __builtin_amdgcn_make_buffer_rsrc((void*)(myws + WS_FLAG), 0, 4096, 0x00020000);
        GAS unsigned* tmo = (GAS unsigned*)(myws + WS_TMO);
        const unsigned long long t0 = __builtin_amdgcn_s_memrealtime();
        const unsigned long long hi = ~0ull << NREM;
        unsigned spins = 0;
        for (;;) {
            const v4u f = __builtin_amdgcn_raw_buffer_load_b128(fr, lane * 16, rb * 1024, 17);
            const unsigned long long m0 = __ballot(f.x >= call) | hi, m1 = __ballot(f.y >= call) | hi;
            const unsigned long long m2 = __ballot(f.z >= call) | hi, m3 = __ballot(f.w >= call) | hi;
            v4u c;
            c.x = ~m0 ? __builtin_ctzll(~m0) : 64u;
            c.y = ~m1 ? __builtin_ctzll(~m1) : 64u;
            c.z = ~m2 ? __builtin_ctzll(~m2) : 64u;
            c.w = ~m3 ? __builtin_ctzll(~m3) : 64u;
            if (lane == 0) *(volatile LAS v4u*)(sync + 16) = c;
            if ((m0 & m1 & m2 & m3) == ~0ull) break;
            if ((++spins & 63u) == 0u) {
                if (__hip_atomic_load(tmo, __ATOMIC_RELAXED, __HIP_MEMORY_SCOPE_AGENT) == call
                    || __builtin_amdgcn_s_memrealtime() - t0 > DEADLINE_TICKS) {
                    if (lane == 0) {
                        __hip_atomic_store(tmo, call, __ATOMIC_RELAXED, __HIP_MEMORY_SCOPE_AGENT);
                        *(volatile LAS unsigned*)(sync + 20) = 1u;
                        *(volatile LAS v4u*)(sync + 16) = v4u{64u, 64u, 64u, 64u};
                    }
                    break;
                }
            }
            __builtin_amdgcn_s_sleep(1);
        }
    }
}

extern "C" void kernel_launch(int rank, int nranks, ncclComm_t comm,
                              hipStream_t stream,
                              void* const* d_in, const int* in_sizes, int n_in,
                              void* d_out, int out_size,
                              const int* in_dtypes, int out_dtype,
                              void* const* d_out_all,
                              void* const* d_ws_all, size_t ws_bytes) {
    if (nranks != NR || (unsigned)rank >= (unsigned)NR || ws_bytes < WS_END || n_in < 4 ||
        in_sizes[0] != 4096 * SH || in_sizes[1] != GK * GN || out_size != SH * GN) {
        fprintf(stderr, "unexpected problem geometry\n");
        return;
    }
    HIP_CHECK(hipFuncSetAttribute((const void*)a2a_gemm_kernel, hipFuncAttributeMaxDynamicSharedMemorySize, LDS_BYTES));
    Params p;
    p.x = (const unsigned char*)d_in[0];
    p.w = (const unsigned char*)d_in[1];
    p.sx = (const float*)d_in[2];
    p.sw = (const float*)d_in[3];
    p.out = (float*)d_out;
    for (int i = 0; i < NR; ++i) p.ws[i] = (unsigned char*)d_ws_all[i];
    p.rank = rank;
    a2a_gemm_kernel<<<NWG, NTHREADS, LDS_BYTES, stream>>>(p);
    HIP_CHECK(hipGetLastError());
}
```

```cpp
#include <hip/hip_runtime.h>
#include <rccl/rccl.h>
#include <cstdio>

#define HIP_CHECK(expr) do { hipError_t _e = (expr); if (_e != hipSuccess) { \
    fprintf(stderr, "HIP error %s at %s:%d\n", hipGetErrorString(_e), __FILE__, __LINE__); \
    return; } } while (0)

#define GAS __attribute__((address_space(1)))
#define LAS __attribute__((address_space(3)))
typedef int v2i __attribute__((ext_vector_type(2)));
typedef int v4i __attribute__((ext_vector_type(4)));
typedef int v8i __attribute__((ext_vector_type(8)));
typedef unsigned v2u __attribute__((ext_vector_type(2)));
typedef unsigned v4u __attribute__((ext_vector_type(4)));
typedef float v4f __attribute__((ext_vector_type(4)));
typedef float v16f __attribute__((ext_vector_type(16)));
typedef int v16i __attribute__((ext_vector_type(16)));
typedef unsigned v8u __attribute__((ext_vector_type(8)));

constexpr int NR = 8;
constexpr int GN = 4096;
constexpr int GK = 4096;
constexpr int SH = 512;
constexpr int NWG = 256;
constexpr int NTHREADS = 8 * 64;
constexpr int NREM = 56;
constexpr int FRAG = 2048;

constexpr size_t WS_CTR = 0;
constexpr size_t WS_TMO = 32 * 1024;
constexpr size_t WS_DBG = 64 * 1024;
constexpr size_t WS_FLAG = 256 * 1024;
constexpr size_t WS_A = 1024 * 1024;
constexpr int A_PAR = 1 << 21;
static_assert(16 * NREM * FRAG <= A_PAR, "");
constexpr size_t WS_END = WS_A + 3 * (size_t)A_PAR;
constexpr int A_OOB = 4 * A_PAR;

#define RING 9
#define SWH 4
#define IDLE_SLEEP 2
#define NFLY 4
#define PRIO_COMP 2
#define PRIO_LOAD 1
#define NREP 4
#define POLL_SLEEP 2
constexpr int FSTRIDE = 4352;
static_assert(WS_FLAG + 16 * NREP * FSTRIDE <= WS_A, "");
#define SEND_AUX 17
#define OUT_AUX 17
constexpr int SLOT = 8192;
constexpr int L_SYNC = 2 * RING * SLOT;
constexpr int L_STAGE = L_SYNC + 256;
constexpr int LDS_BYTES = L_STAGE + 12288;
static_assert(RING >= 8 && RING <= 9 && LDS_BYTES <= 160 * 1024, "");

constexpr unsigned long long DEADLINE_TICKS = 40000000ull;

struct Params {
    const unsigned char* x;
    const unsigned char* w;
    const float* sx;
    const float* sw;
    float* out;
    unsigned char* myws;
    int rank;
    int pad;
    unsigned char* peer[NR - 1];
};

#define STAMP(k) do {} while (0)
#define STAMP_OUT() do {} while (0)

#define WAIT_VM(n) asm volatile("s_waitcnt vmcnt(%0)" :: "i"(n) : "memory")

__device__ __forceinline__ unsigned rfl(unsigned v) { return __builtin_amdgcn_readfirstlane(v); }

__device__ constexpr int send_younger(int k) {
    int n = SWH - 1;
    for (int i = k - SWH; i < k; ++i) n += i & 1;
    return n;
}

__global__ void __launch_bounds__(NTHREADS, 2) a2a_gemm_kernel(Params p) {
    extern __shared__ __attribute__((aligned(16))) unsigned char smem[];
    LAS unsigned char* L8 = (LAS unsigned char*)smem;
    LAS unsigned* sync = (LAS unsigned*)(L8 + L_SYNC);
    STAMP(0);
    const int tid = threadIdx.x;
    const int lane = tid & 63;
    const int wave = rfl(tid >> 6);
    const int b = blockIdx.x;
    const int rank = p.rank;

    const int xg = b & 7, idx = b >> 3;
    const int mb = idx & 7;
    const int n0 = (xg * 4 + (idx >> 3)) * 128;
    GAS unsigned* ctr = (GAS unsigned*)(p.myws + WS_CTR) + b * 32 + wave;

    if (wave == 6) {
        __builtin_amdgcn_s_setprio(3);
        if (b < 112) {
            const int pi = b % 7, rg = b / 7;
            unsigned char* pw = p.peer[pi];
            unsigned callv;
            asm volatile("global_load_dword %0, %1, %2 sc1" : "=&v"(callv) : "v"(0), "s"(ctr) : "memory");
            v4i xr;
            xr.x = (int)(size_t)p.x; xr.y = (int)((size_t)p.x >> 32) & 0xffff; xr.z = 4096 * SH; xr.w = 0x00020000;
            const int xv0 = (lane >> 3) * SH + (((lane & 7) ^ ((lane >> 4) & 3)) * 16);
            const int xv1 = (xv0 ^ 64) + 8 * SH - 1024;
            const int soff = ((((rank + 1 + pi) & 7) * SH) + rg * 32) * SH;
            const int lst = (int)(size_t)L8 + L_STAGE;
#define SFILL(jp, area) \
            asm volatile("s_mov_b32 m0, %6\n\ts_nop 0\n\t" \
                         "buffer_load_dwordx4 %0, %4, %5 offen lds\n\t" \
                         "buffer_load_dwordx4 %1, %4, %5 offen offset:1024 lds\n\t" \
                         "buffer_load_dwordx4 %2, %4, %5 offen offset:2048 lds\n\t" \
                         "buffer_load_dwordx4 %3, %4, %5 offen offset:3072 lds" \
                         :: "v"(xv0), "v"(xv1), "v"(xv0 + 0x1800), "v"(xv1 + 0x1800), "s"(xr), \
                            "s"(soff + (((jp) ^ (rg >> 2)) * 128)), "s"(lst + (area) * 4096) : "memory")
            SFILL(0, 0); SFILL(1, 1); SFILL(2, 2);
            asm volatile("s_barrier" ::: "memory");
            STAMP(1);
            const int rv = (L_STAGE + (lane & 31) * 128 + ((((lane >> 5) * 2) ^ ((lane >> 1) & 7)) * 16)) ^ ((rg & 2) * 32);
            const auto dr = __builtin_amdgcn_make_buffer_rsrc((void*)(pw + WS_A), 0, 3 * A_PAR, 0x00020000);
            const int qq = 6 - pi;
            const unsigned char* fl = pw + WS_FLAG + rg * NREP * FSTRIDE + qq * 4;
            v4u d[8][2];
#define TURN(jp, area) do { \
            _Pragma("unroll") for (int c = 0; c < 4; ++c) d[2 * (jp) + (c >> 1)][c & 1] = *(LAS v4u*)(L8 + (area) * 4096 + (rv ^ ((c >> 1) * 64 + (c & 1) * 16))); \
            asm volatile("" ::: "memory"); } while (0)
#define SEND1(k) do { \
            v4u u_ = d[(k) >> 1][(k) & 1]; \
            u_.x = max(~u_.x, 1u); \
            if ((k) >= SWH) WAIT_VM(send_younger(k)); \
            __builtin_amdgcn_raw_buffer_store_b128(u_, dr, lane * 16 + ((k) & 1) * 1024, dso + ((k) >> 1) * 7 * FRAG, SEND_AUX); } while (0)
#define SEND(j) do { \
            SEND1(2 * (j)); SEND1(2 * (j) + 1); \
            asm volatile("s_mov_b64 exec, %3\n\tglobal_store_dword %0, %1, %2 sc0 sc1\n\ts_mov_b64 exec, -1" \
                         :: "v"((lane & (NREP - 1)) * FSTRIDE), "v"(call * 64u + (unsigned)(((j) + 1) * 7 + qq)), "s"(fl), "i"((1 << NREP) - 1) : "memory"); \
            } while (0)
            unsigned callr;
            asm volatile("s_waitcnt vmcnt(8)\n\tv_readfirstlane_b32 %0, %1" : "=s"(callr) : "v"(callv) : "memory");
            const unsigned call = callr + 1u;
            const int dso = (int)(call % 3u) * A_PAR + (rg * NREM + qq) * FRAG;
            TURN(0, 0);
            asm volatile("s_waitcnt lgkmcnt(0)" ::: "memory");
            SFILL(3, 0);
            STAMP(2);
            SEND(0);
            STAMP(3);
            WAIT_VM(3);
            TURN(1, 1);
            TURN(2, 2);
            TURN(3, 0);
            SEND(1);
            STAMP(4);
            SEND(2); SEND(3);
            STAMP(5);
            SEND(4); SEND(5); SEND(6); SEND(7);
            STAMP(6);
            if (lane == 0) __hip_atomic_store(ctr, call, __ATOMIC_RELAXED, __HIP_MEMORY_SCOPE_AGENT);
        }
        STAMP_OUT();
    } else if (wave < 4) {
        __builtin_amdgcn_s_setprio(PRIO_COMP);
        const int mh = wave & 1, kp = wave >> 1;
        const int rg = mb * 2 + mh;
        const unsigned char* ab = p.myws + WS_A + rg * NREM * FRAG;
        v4i xr, ar;
        xr.x = (int)(size_t)p.x; xr.y = (int)((size_t)p.x >> 32) & 0xffff; xr.z = 4096 * SH; xr.w = 0x00020000;
        ar.x = (int)(size_t)ab; ar.y = (int)((size_t)ab >> 32) & 0xffff; ar.z = 2 * A_PAR + NREM * FRAG; ar.w = 0x00020000;
        const int avoff = lane * 16;
        const int xv0 = (lane >> 3) * SH + (((lane & 7) ^ ((lane >> 4) & 3)) * 16);
        const int xv1 = (xv0 ^ 64) + 8 * SH - 1024;
        const int xsoff = (rank * SH + rg * 32) * SH + kp * 128;
        const int lb = (int)(size_t)L8;
        const int stg = lb + (kp * RING + RING - 2 + mh) * SLOT;
        const int rv = stg + (lane & 31) * 128 + ((((lane >> 5) * 2) ^ ((lane >> 1) & 7)) * 16);
        const int g = lane >> 4, q = (lane & 15) >> 1;
        const int rdl = lb + kp * RING * SLOT + ((g >> 1) * 32 + q) * 128 + (g & 1) * 16 + (lane & 1) * 8;
        const int sw = (q >> 1) ^ (2 * kp);
        const int rd0 = rdl + ((0 ^ sw) * 32), rd1 = rdl + ((1 ^ sw) * 32);
        const int rd2 = rdl + ((2 ^ sw) * 32), rd3 = rdl + ((3 ^ sw) * 32);
        const int a_ardy = lb + L_SYNC + 4 * mh, a_wrdy = lb + L_SYNC + 8 + 4 * kp, a_cons = lb + L_SYNC + 16 + 4 * wave;
        int aoff = kp * FRAG;
        v16f acc0, acc1, acc2, acc3;
        v16i clob0, clob1, clob2, clob3, clob4, clob5;
        int va0, va1, va2, va3, vt0, vt1, vt2, vt3;
        int s_valid, s_need, s_wslot, s_wrdy, s_t0, s_t1, s_so;
        unsigned long long s_ret, s_now;
        const unsigned tlo = (unsigned)__builtin_amdgcn_s_memrealtime();
#define DBG_ASM(s_)
#define DBG_SIDE(s_)
#define DBG_ASM0
#define DBG_OPS
#define AREG(s_) "v[160+8*" #s_ ":167+8*" #s_ "]"
#define ALO(s_) "v[160+8*" #s_ ":163+8*" #s_ "]"
#define AHI(s_) "v[164+8*" #s_ ":167+8*" #s_ "]"
#define LDLO(p_) "buffer_load_dwordx4 " ALO(p_) ", %[avoff], %[ar], %[so] offen sc1\n\t"
#define LDHI(p_) "buffer_load_dwordx4 " AHI(p_) ", %[avoff], %[ar], %[so] offen offset:1024 sc1\n\t"
#define RDB(base_, va_) \
        "ds_read_b64_tr_b8 v[" #base_ ":" #base_ "+1], " va_ "\n\t" \
        "ds_read_b64_tr_b8 v[" #base_ "+2:" #base_ "+3], " va_ " offset:1024\n\t" \
        "ds_read_b64_tr_b8 v[" #base_ "+4:" #base_ "+5], " va_ " offset:2048\n\t" \
        "ds_read_b64_tr_b8 v[" #base_ "+6:" #base_ "+7], " va_ " offset:3072\n\t"
#define PICK \
        "v_readfirstlane_b32 %[t0], %[vt0]\n\t" \
        "v_readfirstlane_b32 %[wrdy], %[vt1]\n\t" \
        "s_and_b32 %[t1], %[t0], 0x600000\n\t" \
        "s_or_b32 %[t1], %[t1], %[aoff]\n\t" \
        "s_cmp_lt_u32 %[t1], %[t0]\n\t"
#define AHITX
#define TAKE \
        "s_cselect_b32 %[so], %[t1], %[oob]\n\t" \
        "s_cselect_b32 %[t0], 0x1000, 0\n\t" \
        "s_addc_u32 %[valid], %[valid], %[valid]\n\t" \
        "s_add_u32 %[aoff], %[aoff], %[t0]\n\t" AHITX
#define COND \
        "s_bitcmp1_b32 %[valid], 7\n\t" \
        "s_cselect_b32 %[t0], %[wrdy], 0\n\t" \
        "s_cmp_lt_u32 %[t0], %[need]\n\t"
#define CHKBR(s_) "s_cbranch_vccnz .Lretry" #s_ "_%=\n"
#define POS(s_, p_) \
        ".Lpos" #s_ "_%=:\n\t" \
        "s_cbranch_scc1 .Lslow" #s_ "_%=\n" \
        ".Lcomp" #s_ "_%=:\n\t" \
        "s_waitcnt vmcnt(12)\n" \
        ".Lchk" #s_ "_%=:\n\t" \
        "v_min_u32 %[vt3], v[160+8*" #s_ "], v[164+8*" #s_ "]\n\t" \
        "v_cmp_eq_u32 vcc, 0, %[vt3]\n\t" \
        "v_not_b32 v[160+8*" #s_ "], v[160+8*" #s_ "]\n\t" \
        "v_not_b32 v[164+8*" #s_ "], v[164+8*" #s_ "]\n\t" \
        "s_waitcnt lgkmcnt(11)\n\t" \
        CHKBR(s_) \
        ".Lok" #s_ "_%=:\n\t" \
        "v_mfma_f32_32x32x64_f8f6f4 v[64:79], " AREG(s_) ", v[128:135], v[64:79] cbsz:1 blgp:1\n\t" \
        LDLO(p_) \
        RDB(128, "%[va0]") \
        "s_add_u32 %[wslot], %[wslot], %[slotb]\n\t" \
        "s_cmp_eq_u32 %[wslot], %[ringb]\n\t" \
        "s_cselect_b32 %[wslot], 0, %[wslot]\n\t" \
        "v_add_u32 %[va0], %[wslot], %[rd0]\n\t" \
        "s_waitcnt lgkmcnt(11)\n\t" \
        "v_mfma_f32_32x32x64_f8f6f4 v[80:95], " AREG(s_) ", v[136:143], v[80:95] cbsz:1 blgp:1\n\t" \
        LDHI(p_) \
        RDB(136, "%[va1]") \
        "v_add_u32 %[va1], %[wslot], %[rd1]\n\t" \
        "v_add_u32 %[vt2], 1, %[vt2]\n\t" \
        "s_add_u32 %[need], %[need], 1\n\t" \
        DBG_ASM(s_) \
        "s_waitcnt lgkmcnt(9)\n\t" \
        "v_mfma_f32_32x32x64_f8f6f4 v[96:111], " AREG(s_) ", v[144:151], v[96:111] cbsz:1 blgp:1\n\t" \
        RDB(144, "%[va2]") \
        "v_add_u32 %[va2], %[wslot], %[rd2]\n\t" \
        PICK TAKE \
        "v_mfma_f32_32x32x64_f8f6f4 v[112:127], " AREG(s_) ", v[152:159], v[112:127] cbsz:1 blgp:1\n\t" \
        RDB(152, "%[va3]") \
        "ds_read_b32 %[vt0], %[aardy]\n\t" \
        "ds_read_b32 %[vt1], %[awrdy]\n\t" \
        "ds_write_b32 %[acons], %[vt2]\n\t" \
        "v_add_u32 %[va3], %[wslot], %[rd3]\n\t" \
        COND
#define SIDE(s_, p_, n_) \
        ".Lslow" #s_ "_%=:\n\t" \
        "s_bitcmp1_b32 %[valid], 7\n\t" \
        "s_cbranch_scc1 .Lwspin" #s_ "_%=\n\t" \
        "s_cmp_eq_u32 %[need], 34\n\t" \
        "s_cbranch_scc1 .Ldone_%=\n\t" \
        LDLO(p_) LDHI(p_) \
        "s_cmp_lt_u32 %[aoff], %[aend]\n\t" \
        "s_cbranch_scc0 .Ltake" #s_ "_%=\n" \
        ".Lidle" #s_ "_%=:\n\t" \
        "s_sleep %[slp]\n\t" \
        "ds_read_b32 %[vt0], %[aardy]\n\t" \
        "ds_read_b32 %[vt1], %[awrdy]\n\t" \
        "s_waitcnt lgkmcnt(0)\n\t" \
        PICK \
        "s_cbranch_scc1 .Ltake" #s_ "_%=\n\t" \
        "s_and_b32 %[t0], %[valid], 0x7f\n\t" \
        "s_cbranch_scc0 .Lidle" #s_ "_%=\n\t" \
        "s_cmp_eq_u32 %[t0], 0\n" \
        ".Ltake" #s_ "_%=:\n\t" \
        TAKE COND \
        "s_branch .Lpos" #n_ "_%=\n" \
        ".Lwspin" #s_ "_%=:\n\t" \
        "ds_read_b32 %[vt1], %[awrdy]\n\t" \
        "s_waitcnt lgkmcnt(0)\n\t" \
        "v_readfirstlane_b32 %[wrdy], %[vt1]\n\t" \
        "s_cmp_lt_u32 %[wrdy], %[need]\n\t" \
        "s_cbranch_scc1 .Lwspin" #s_ "_%=\n\t" \
        "s_branch .Lcomp" #s_ "_%=\n" \
        ".Lretry" #s_ "_%=:\n\t" \
        "s_getpc_b64 %[ret]\n\t" \
        "s_branch .Lagain_%=\n\t" \
        "s_cbranch_scc1 .Lok" #s_ "_%=\n\t" \
        "buffer_load_dwordx4 " ALO(s_) ", %[avoff], %[ar], %[t1] offen sc1\n\t" \
        "buffer_load_dwordx4 " AHI(s_) ", %[avoff], %[ar], %[t1] offen offset:1024 sc1\n\t" \
        "s_waitcnt vmcnt(0)\n\t" \
        "s_branch .Lchk" #s_ "_%=\n" \
        DBG_SIDE(s_)
#define XLD(so_) \
        "buffer_load_dwordx4 %[xv0], %[xr], " so_ " offen lds\n\t" \
        "buffer_load_dwordx4 %[xv1], %[xr], " so_ " offen offset:1024 lds\n\t" \
        "buffer_load_dwordx4 %[vt0], %[xr], " so_ " offen offset:2048 lds\n\t" \
        "buffer_load_dwordx4 %[vt1], %[xr], " so_ " offen offset:3072 lds\n\t"
#define XRD(d_, off_) \
        "ds_read_b128 v[" #d_ ":" #d_ "+3], %[rv] offset:" #off_ "\n\t" \
        "ds_read_b128 v[" #d_ "+4:" #d_ "+7], %[va1] offset:" #off_ "\n\t" \
        "ds_read_b128 v[" #d_ "+8:" #d_ "+11], %[va2] offset:" #off_ "\n\t" \
        "ds_read_b128 v[" #d_ "+12:" #d_ "+15], %[va3] offset:" #off_ "\n\t"
#define XINV(r_) "v_not_b32 v" #r_ ", v" #r_ "\n\tv_max_u32 v" #r_ ", 1, v" #r_ "\n\t"
#define Z8(b_) \
        "v_mov_b64 v[" #b_ ":" #b_ "+1], 0\n\tv_mov_b64 v[" #b_ "+2:" #b_ "+3], 0\n\t" \
        "v_mov_b64 v[" #b_ "+4:" #b_ "+5], 0\n\tv_mov_b64 v[" #b_ "+6:" #b_ "+7], 0\n\t"
        asm volatile(
            "v_add_u32 %[vt0], 0x1800, %[xv0]\n\t"
            "v_add_u32 %[vt1], 0x1800, %[xv1]\n\t"
            "s_mov_b32 m0, %[stg]\n\t"
            "s_add_u32 %[t0], %[xsoff], 256\n\t"
            "s_barrier\n\t"
            XLD("%[xsoff]")
            "s_add_u32 m0, m0, 4096\n\t"
            "s_nop 0\n\t"
            XLD("%[t0]")
            "v_xor_b32 %[va1], 16, %[rv]\n\t"
            "v_xor_b32 %[va2], 64, %[rv]\n\t"
            "v_xor_b32 %[va3], 0x50, %[rv]\n\t"
            Z8(64) Z8(72) Z8(80) Z8(88) Z8(96) Z8(104) Z8(112) Z8(120)
            "s_mov_b32 %[valid], 0xf0\n\t"
            "s_mov_b32 %[need], 2\n\t"
            "s_mov_b32 %[wslot], %[slotb]\n\t"
            "s_mov_b32 %[so], %[oob]\n\t"
            "v_mov_b32 %[vt2], 1\n\t"
            "s_waitcnt vmcnt(4)\n\t"
            XRD(160, 0)
            "s_waitcnt vmcnt(0)\n\t"
            XRD(176, 4096)
            "ds_read_b32 %[vt0], %[aardy]\n"
            ".Lfirst_%=:\n\t"
            "ds_read_b32 %[vt1], %[awrdy]\n\t"
            "s_waitcnt lgkmcnt(0)\n\t"
            "v_readfirstlane_b32 %[wrdy], %[vt1]\n\t"
            "s_cmp_eq_u32 %[wrdy], 0\n\t"
            "s_cbranch_scc1 .Lfirst_%=\n\t"
            DBG_ASM0
            XINV(160) XINV(164) XINV(168) XINV(172) XINV(176) XINV(180) XINV(184) XINV(188)
            RDB(128, "%[rd0]") RDB(136, "%[rd1]") RDB(144, "%[rd2]") RDB(152, "%[rd3]")
            "ds_write_b32 %[acons], %[vt2]\n\t"
            "v_add_u32 %[va0], %[wslot], %[rd0]\n\t"
            "v_add_u32 %[va1], %[wslot], %[rd1]\n\t"
            "v_add_u32 %[va2], %[wslot], %[rd2]\n\t"
            "v_add_u32 %[va3], %[wslot], %[rd3]\n\t"
            "s_waitcnt lgkmcnt(1)\n\t"
            COND
            POS(0, 7) POS(1, 0) POS(2, 1) POS(3, 2) POS(4, 3) POS(5, 4) POS(6, 5) POS(7, 6)
            "s_branch .Lpos0_%=\n"
            SIDE(0, 7, 1) SIDE(1, 0, 2) SIDE(2, 1, 3) SIDE(3, 2, 4) SIDE(4, 3, 5) SIDE(5, 4, 6) SIDE(6, 5, 7) SIDE(7, 6, 0)
            ".Lagain_%=:\n\t"
            "s_sleep 2\n\t"
            "ds_read_b32 %[vt0], %[aardy]\n\t"
            "s_memrealtime %[now]\n\t"
            "s_add_u32 s40, s40, 4\n\t"
            "s_addc_u32 s41, s41, 0\n\t"
            "s_waitcnt lgkmcnt(0)\n\t"
            "v_readfirstlane_b32 %[t0], %[vt0]\n\t"
            "s_lshl_b32 %[t1], %[need], 12\n\t"
            "s_add_u32 %[t1], %[t1], %[abias]\n\t"
            "s_and_b32 %[t0], %[t0], 0x600000\n\t"
            "s_or_b32 %[t1], %[t1], %[t0]\n\t"
            "s_sub_u32 %[t0], s42, %[tlo]\n\t"
            "s_cmp_gt_u32 %[t0], %[dl]\n\t"
            "s_cbranch_scc0 .Lback_%=\n\t"
            "v_mov_b32 %[vt3], 0x7fc00000\n\t"
            "ds_write_b32 %[ascale], %[vt3]\n"
            ".Lback_%=:\n\t"
            "s_setpc_b64 %[ret]\n"
            ".Ldone_%=:\n\t"
            "s_waitcnt vmcnt(0) lgkmcnt(0)\n\t"
            "s_nop 15"
            : "=&{v[64:79]}"(acc0), "=&{v[80:95]}"(acc1), "=&{v[96:111]}"(acc2), "=&{v[112:127]}"(acc3),
              "=&{v[128:143]}"(clob0), "=&{v[144:159]}"(clob1), "=&{v[160:175]}"(clob2), "=&{v[176:191]}"(clob3),
              "=&{v[192:207]}"(clob4), "=&{v[208:223]}"(clob5),
              [va0] "=&v"(va0), [va1] "=&v"(va1), [va2] "=&v"(va2), [va3] "=&v"(va3),
              [vt0] "=&v"(vt0), [vt1] "=&v"(vt1), [vt2] "=&v"(vt2), [vt3] "=&v"(vt3),
              [ret] "=&{s[40:41]}"(s_ret), [now] "=&{s[42:43]}"(s_now),
              [valid] "=&s"(s_valid), [need] "=&s"(s_need), [wslot] "=&s"(s_wslot),
              [wrdy] "=&s"(s_wrdy), [t0] "=&s"(s_t0), [t1] "=&s"(s_t1), [so] "=&s"(s_so),
              [aoff] "+s"(aoff) DBG_OPS
            : [xr] "s"(xr), [ar] "s"(ar), [xsoff] "s"(xsoff), [xv0] "v"(xv0), [xv1] "v"(xv1), [avoff] "v"(avoff),
              [rd0] "v"(rd0), [rd1] "v"(rd1), [rd2] "v"(rd2), [rd3] "v"(rd3),
              [stg] "s"(stg), [rv] "v"(rv), [oob] "s"(A_OOB), [aardy] "v"(a_ardy), [awrdy] "v"(a_wrdy), [acons] "v"(a_cons),
              [slotb] "i"(SLOT), [ringb] "i"(RING * SLOT), [slp] "i"(IDLE_SLEEP), [aend] "i"(NREM * FRAG),
              [abias] "s"(kp * FRAG - 6 * 4096), [tlo] "s"(tlo), [dl] "i"((unsigned)DEADLINE_TICKS), [ascale] "v"(lb + L_SYNC + 48)
            : "memory", "scc", "vcc");
        STAMP(5);
        {
            while (rfl(*(volatile LAS unsigned*)(sync + 4 + (wave ^ 1))) < (unsigned)(34 - RING)) {}
            asm volatile("" ::: "memory");
            const int xs = (mh ? (33 - RING) % RING : (32 - RING) % RING) * SLOT;
            LAS v4f* xo = (LAS v4f*)(L8 + kp * RING * SLOT + xs) + lane;
            LAS v4f* xi = (LAS v4f*)(L8 + (kp ^ 1) * RING * SLOT + xs) + lane;
#pragma unroll
            for (int r = 0; r < 4; ++r) {
                xo[r * 64] = v4f{acc2[4 * r], acc2[4 * r + 1], acc2[4 * r + 2], acc2[4 * r + 3]};
                xo[(4 + r) * 64] = v4f{acc3[4 * r], acc3[4 * r + 1], acc3[4 * r + 2], acc3[4 * r + 3]};
            }
            asm volatile("s_waitcnt lgkmcnt(0)" ::: "memory");
            *(volatile LAS unsigned*)(sync + 8 + wave) = 1u;
            const auto orr = __builtin_amdgcn_make_buffer_rsrc((void*)p.out, 0, SH * GN * 4, 0x00020000);
            const int ovoff = ((lane >> 5) * 4 * GN + (lane & 31)) * 4;
            const int osoff = (rg * 32 * GN + n0 + kp * 64) * 4;
            while (rfl(*(volatile LAS unsigned*)(sync + 8 + (wave ^ 2))) == 0u) {}
            asm volatile("" ::: "memory");
            const float sc = *(volatile LAS float*)(sync + 12);
#pragma unroll
            for (int r = 0; r < 4; ++r) {
                const v4f i0 = xi[r * 64], i1 = xi[(4 + r) * 64];
#pragma unroll
                for (int e = 0; e < 4; ++e) {
                    const float y0 = __builtin_elementwise_maximum((acc0[4 * r + e] + i0[e]) * sc, 0.f);
                    const float y1 = __builtin_elementwise_maximum((acc1[4 * r + e] + i1[e]) * sc, 0.f);
                    const int so = osoff + (8 * r + e) * GN * 4;
                    __builtin_amdgcn_raw_buffer_store_b32(__builtin_bit_cast(unsigned, y0), orr, ovoff, so, OUT_AUX);
                    __builtin_amdgcn_raw_buffer_store_b32(__builtin_bit_cast(unsigned, y1), orr, ovoff + 128, so, OUT_AUX);
                }
            }
        }
        STAMP(6);
        STAMP_OUT();
    } else if (wave < 6) {
        __builtin_amdgcn_s_setprio(PRIO_LOAD);
        const int kp = wave - 4;
        v4i rs;
        rs.x = (int)(size_t)p.w; rs.y = (int)((size_t)p.w >> 32) & 0xffff; rs.z = GK * GN; rs.w = 0x00020000;
        const int v0 = (lane >> 3) * GN + n0 + (((lane & 7) ^ (((lane >> 4) & 3) << 1)) * 16);
        const int v1 = v0 + 8 * GN - 1024, v2 = v0 + 16 * GN - 2048, v3 = v0 + 24 * GN - 3072;
        int tab;
        {
            const int n = lane & 31;
            const int ri = 2 * (n - 4) + kp, jj = (ri * 37) >> 8;
            const int kb = n < 4 ? rank * SH + ((n >> 1) * 4 + kp * 2 + (n & 1)) * 64
                                 : ((rank + 1 + ri - jj * 7) & 7) * SH + (jj ^ mb) * 64;
            const int need = n >= RING ? n - RING + 1 : (n >= RING - 2 ? 1 : 0);
            tab = lane < 32 ? kb * GN + (n % RING) * SLOT + need : 63;
        }
        const int lbase = (int)(size_t)L8 + kp * RING * SLOT;
        const int a_wrdy = (int)(size_t)L8 + L_SYNC + 8 + 4 * kp, a_cons = (int)(size_t)L8 + L_SYNC + 16 + 8 * kp;
        int s_n, s_e, s_so, s_so2, s_need, s_t, vpub;
        v2i vc;
#define WLD(v_, so_, off_) "buffer_load_dwordx4 %[" #v_ "], %[rs], %[" #so_ "] offen offset:" #off_ " lds\n\t"
#define FILL \
        WLD(v0, so, 0) \
        "ds_read_b64 %[vc], %[acons]\n\t" \
        "s_add_u32 %[n], %[n], 1\n\t" \
        WLD(v1, so, 1024) \
        "v_readlane_b32 %[e], %[tab], %[n]\n\t" \
        WLD(v2, so, 2048) \
        "v_add_u32 %[vpub], 1, %[vpub]\n\t" \
        WLD(v3, so, 3072) \
        "s_add_u32 m0, m0, 4096\n\t" \
        "s_and_b32 %[so], %[e], 0xfffc0000\n\t" \
        WLD(v0, so2, 0) \
        "s_and_b32 %[need], %[e], 63\n\t" \
        "s_and_b32 %[e], %[e], 0x1e000\n\t" \
        WLD(v1, so2, 1024) \
        "s_waitcnt lgkmcnt(0)\n\t" \
        "v_min_u32 v20, v20, v21\n\t" \
        WLD(v2, so2, 2048) \
        "v_readfirstlane_b32 %[t], v20\n\t" \
        WLD(v3, so2, 3072) \
        "s_add_u32 m0, %[e], %[lbase]\n\t" \
        "s_add_u32 %[so2], %[so], 0x20000\n\t"
#define PUB(cnt_) \
        "s_waitcnt vmcnt(" #cnt_ ")\n\t" \
        "ds_write_b32 %[awrdy], %[vpub]\n\t"
#define PUBN(cnt_) "v_add_u32 %[vpub], 1, %[vpub]\n\t" PUB(cnt_)
#define LDBG_IN
#define LDBG_OUT
#define LDBG_OPS
        asm volatile(
            "s_mov_b32 %[n], 0\n\t"
            "v_mov_b32 %[vpub], %[pub0]\n\t"
            "v_readlane_b32 %[so], %[tab], 0\n\t"
            "s_mov_b32 m0, %[lbase]\n\t"
            "s_add_u32 %[so2], %[so], 0x20000\n\t"
            "s_nop 2\n\t"
            "s_barrier\n"
            ".Lpro_%=:\n\t"
            FILL
            "s_cmp_lt_u32 %[n], %[nf]\n\t"
            "s_cbranch_scc1 .Lpro_%=\n"
            ".Lmain_%=:\n\t"
            PUB(%[cnt])
            ".Lgo_%=:\n\t"
            LDBG_IN
            FILL
            "s_cmp_lt_u32 %[t], %[need]\n\t"
            "s_cbranch_scc0 .Lmain_%=\n\t"
            PUB(%[cnt])
            "s_cmp_eq_u32 %[n], 32\n\t"
            "s_cbranch_scc1 .Lend_%=\n"
            ".Lspin_%=:\n\t"
            "ds_read_b64 %[vc], %[acons]\n\t"
            "s_waitcnt lgkmcnt(0)\n\t"
            "v_min_u32 v20, v20, v21\n\t"
            "s_nop 0\n\t"
            "v_readfirstlane_b32 %[t], v20\n\t"
            "s_cmp_lt_u32 %[t], %[need]\n\t"
            "s_cbranch_scc0 .Lgo_%=\n\t"
            "s_sleep 1\n\t"
            "s_branch .Lspin_%=\n"
            LDBG_OUT
            ".Lend_%=:\n\t"
            PUBN(16)
            PUBN(8)
            "v_add_u32 %[vpub], 32, %[vpub]\n\t"
            PUB(0)
            "s_waitcnt lgkmcnt(0)"
            : [n] "=&s"(s_n), [e] "=&s"(s_e), [so] "=&s"(s_so), [so2] "=&s"(s_so2), [need] "=&s"(s_need), [t] "=&s"(s_t),
              [vpub] "=&v"(vpub), [vc] "=&{v[20:21]}"(vc) LDBG_OPS
            : [tab] "v"(tab), [rs] "s"(rs), [lbase] "s"(lbase), [v0] "v"(v0), [v1] "v"(v1), [v2] "v"(v2), [v3] "v"(v3),
              [awrdy] "v"(a_wrdy), [acons] "v"(a_cons), [nf] "i"(NFLY), [cnt] "i"(8 * (NFLY - 1)), [pub0] "i"(1 - NFLY)
            : "memory", "scc");
        STAMP(7);
        STAMP_OUT();
    } else {
        __builtin_amdgcn_s_setprio(3);
        sync[lane] = 0u;
        const unsigned callv = __hip_atomic_load(ctr, __ATOMIC_RELAXED, __HIP_MEMORY_SCOPE_AGENT);
        asm volatile("s_waitcnt lgkmcnt(0)\n\ts_barrier" ::: "memory");
        *(volatile LAS float*)(sync + 12) = p.sx[0] * p.sw[0];
        const unsigned call = rfl(callv) + 1u;
        __hip_atomic_store(ctr, call, __ATOMIC_RELAXED, __HIP_MEMORY_SCOPE_AGENT);
        const unsigned char* fl = p.myws + WS_FLAG + (mb * 2 * NREP + (xg & (NREP - 1))) * FSTRIDE;
        GAS unsigned* tmo = (GAS unsigned*)(p.myws + WS_TMO);
        const unsigned long long t0 = __builtin_amdgcn_s_memrealtime();
        const unsigned tag = (call % 3u) << 21;
        {
            const auto zr = __builtin_amdgcn_make_buffer_rsrc((void*)(p.myws + WS_A), 0, 3 * A_PAR, 0x00020000);
            const int zo = (int)((call + 2u) % 3u) * A_PAR + b * 7168;
#pragma unroll
            for (int u = 0; u < 7; ++u) __builtin_amdgcn_raw_buffer_store_b128(v4u{0u, 0u, 0u, 0u}, zr, lane * 16, zo + u * 1024, 17);
        }
        unsigned spins = 0;
        int last0 = 0, last1 = 0;
        for (;;) {
            v8u f, h;
            asm volatile("s_load_dwordx8 %0, %2, 0x0 glc\n\ts_load_dwordx8 %1, %2, %3 glc\n\ts_waitcnt lgkmcnt(0)"
                         : "=&s"(f), "=&s"(h) : "s"(fl), "i"(NREP * FSTRIDE) : "memory");
            const unsigned m0 = min(min(min(f[0], f[1]), min(f[2], f[3])), min(min(f[4], f[5]), f[6]));
            const unsigned m1 = min(min(min(h[0], h[1]), min(h[2], h[3])), min(min(h[4], h[5]), h[6]));
            const int r0 = min(max((int)(m0 - call * 64u), 0), NREM), r1 = min(max((int)(m1 - call * 64u), 0), NREM);
            if (r0 != last0 || r1 != last1) {
                *(volatile LAS v2u*)(sync) = v2u{(unsigned)r0 * FRAG | tag, (unsigned)r1 * FRAG | tag};
                last0 = r0; last1 = r1;
                if (min(r0, r1) >= NREM) break;
            }
            if ((++spins & 255u) == 0u) {
                if (rfl(__hip_atomic_load(tmo, __ATOMIC_RELAXED, __HIP_MEMORY_SCOPE_AGENT)) == call
                    || __builtin_amdgcn_s_memrealtime() - t0 > DEADLINE_TICKS) {
                    __hip_atomic_store(tmo, call, __ATOMIC_RELAXED, __HIP_MEMORY_SCOPE_AGENT);
                    *(volatile LAS float*)(sync + 12) = __builtin_nanf("");
                    *(volatile LAS v2u*)(sync) = v2u{(unsigned)NREM * FRAG | tag, (unsigned)NREM * FRAG | tag};
                    break;
                }
            }
            if (POLL_SLEEP) __builtin_amdgcn_s_sleep(POLL_SLEEP);
        }
        STAMP(7);
        STAMP_OUT();
    }
}

extern "C" void kernel_launch(int rank, int nranks, ncclComm_t comm,
                              hipStream_t stream,
                              void* const* d_in, const int* in_sizes, int n_in,
                              void* d_out, int out_size,
                              const int* in_dtypes, int out_dtype,
                              void* const* d_out_all,
                              void* const* d_ws_all, size_t ws_bytes) {
    if (nranks != NR || (unsigned)rank >= (unsigned)NR || ws_bytes < WS_END || n_in < 4 ||
        in_sizes[0] != 4096 * SH || in_sizes[1] != GK * GN || out_size != SH * GN) {
        fprintf(stderr, "unexpected problem geometry\n");
        return;
    }
    HIP_CHECK(hipFuncSetAttribute((const void*)a2a_gemm_kernel, hipFuncAttributeMaxDynamicSharedMemorySize, LDS_BYTES));
    Params p;
    p.x = (const unsigned char*)d_in[0];
    p.w = (const unsigned char*)d_in[1];
    p.sx = (const float*)d_in[2];
    p.sw = (const float*)d_in[3];
    p.out = (float*)d_out;
    p.myws = (unsigned char*)d_ws_all[rank];
    p.rank = rank;
    p.pad = 0;
    for (int i = 0; i < NR - 1; ++i) p.peer[i] = (unsigned char*)d_ws_all[(rank + 1 + i) & 7];
    a2a_gemm_kernel<<<NWG, NTHREADS, LDS_BYTES, stream>>>(p);
    HIP_CHECK(hipGetLastError());
}
```

```cpp
#include <hip/hip_runtime.h>
#include <rccl/rccl.h>
#include <cstdio>

#define HIP_CHECK(expr) do { hipError_t _e = (expr); if (_e != hipSuccess) { \
    fprintf(stderr, "HIP error %s at %s:%d\n", hipGetErrorString(_e), __FILE__, __LINE__); \
    return; } } while (0)

#define GAS __attribute__((address_space(1)))
#define LAS __attribute__((address_space(3)))
typedef int v2i __attribute__((ext_vector_type(2)));
typedef int v4i __attribute__((ext_vector_type(4)));
typedef int v8i __attribute__((ext_vector_type(8)));
typedef unsigned v2u __attribute__((ext_vector_type(2)));
typedef unsigned v4u __attribute__((ext_vector_type(4)));
typedef float v4f __attribute__((ext_vector_type(4)));
typedef float v16f __attribute__((ext_vector_type(16)));
typedef int v16i __attribute__((ext_vector_type(16)));
typedef unsigned long long v4l __attribute__((ext_vector_type(4)));

constexpr int NR = 8;
constexpr int GN = 4096;
constexpr int GK = 4096;
constexpr int SH = 512;
constexpr int NWG = 256;
constexpr int NTHREADS = 8 * 64;
constexpr int NREM = 56;
constexpr int FRAG = 2048;

constexpr size_t WS_CTR = 0;
constexpr size_t WS_TMO = 32 * 1024;
constexpr size_t WS_DBG = 64 * 1024;
constexpr size_t WS_A = 0;
constexpr int A_PAR = 1 << 21;
static_assert(16 * NREM * FRAG <= A_PAR, "");
constexpr size_t WS_END = WS_A + 4 * (size_t)A_PAR;
constexpr int A_OOB = 4 * A_PAR;

#define RING 9
#define SLOT_Q 70
#define SLOT_T0 60
#define NFLY 4
#define PRIO_COMP 2
#define PRIO_LOAD 1
#define NREP 4
#define POLL_SLEEP 2
constexpr int A_FLAG = 16 * NREM * FRAG;
constexpr int FSTRIDE = 3840;
static_assert(A_FLAG + 16 * NREP * FSTRIDE <= A_PAR, "");
#define SEND_AUX 17
#define ZERO_AT 8
#define ROT 3
#define WARM_SLEEP 0
#define RETRY_SLEEP 1
#define HOLD 32
#define OUT_AUX 17
constexpr int SLOT = 8192;
constexpr int L_SYNC = 2 * RING * SLOT;
constexpr int L_STAGE = L_SYNC + 256;
constexpr int LDS_BYTES = L_STAGE + 8192;
static_assert(RING >= 8 && RING <= 9 && LDS_BYTES <= 160 * 1024, "");

constexpr unsigned long long DEADLINE_TICKS = 40000000ull;

struct Params {
    const unsigned char* x;
    const unsigned char* w;
    const float* sx;
    const float* sw;
    float* out;
    unsigned char* myws;
    int rank;
    int pad;
    unsigned char* peer[NR - 1];
};

#define STAMP(k) do {} while (0)
#define STAMP_OUT() do {} while (0)

#define WAIT_VM(n) asm volatile("s_waitcnt vmcnt(%0)" :: "i"(n) : "memory")

template <int AUX>
__device__ __forceinline__ void store16(v4u v, __amdgpu_buffer_rsrc_t r, int voff, int soff) {
    __builtin_amdgcn_raw_buffer_store_b128(v, r, voff, soff, AUX);
    asm volatile("s_nop 1" :: "v"(v) : "memory");
}

__device__ __forceinline__ unsigned rfl(unsigned v) { return __builtin_amdgcn_readfirstlane(v); }

__global__ void __launch_bounds__(NTHREADS, 2) a2a_gemm_kernel(Params p) {
    extern __shared__ __attribute__((aligned(16))) unsigned char smem[];
    LAS unsigned char* L8 = (LAS unsigned char*)smem;
    LAS unsigned* sync = (LAS unsigned*)(L8 + L_SYNC);
    STAMP(0);
    unsigned long long tentry;
    asm volatile("s_memrealtime %0" : "=s"(tentry));
    const int tid = threadIdx.x;
    const int lane = tid & 63;
    const int wave = rfl(tid >> 6);
    const int b = blockIdx.x;
    const int rank = p.rank;
    asm volatile("s_waitcnt lgkmcnt(0)" : "+s"(tentry) : "s"(p.x), "s"(p.w), "s"(p.sx), "s"(p.sw), "s"(p.out), "s"(p.myws), "s"(rank), "s"(p.peer[0]), "s"(p.peer[1]),
                       "s"(p.peer[2]), "s"(p.peer[3]), "s"(p.peer[4]), "s"(p.peer[5]), "s"(p.peer[6]));

    const int xg = b & 7, idx = b >> 3;
    const int mb = idx & 7;
    const int n0 = (xg * 4 + (idx >> 3)) * 128;
    GAS unsigned* ctr = (GAS unsigned*)(p.myws + WS_CTR) + b * 32 + wave;

    if (wave == 0) {
        __builtin_amdgcn_s_setprio(3);
        {
            const unsigned tstart = (unsigned)tentry;
            if (b >= 64) asm volatile("s_barrier" ::: "memory");
            const int rg = (b >> 1) & 15, rd = b >> 5, i0 = (b & 1) * 4;
            unsigned callv;
            asm volatile("global_load_dword %0, %1, %2 sc1" : "=&v"(callv) : "v"(0), "s"(ctr) : "memory");
            v4i xr;
            xr.x = (int)(size_t)p.x; xr.y = (int)((size_t)p.x >> 32) & 0xffff; xr.z = 4096 * SH; xr.w = 0x00020000;
            const int xv0 = (lane >> 2) * SH + (((lane & 3) ^ ((lane >> 4) & 3)) * 16);
            const int xv1 = xv0 + 16 * SH - 1024;
            const int soff = rg * 32 * SH + (rd ^ ((rg >> 1) & ROT)) * 64;
            const int lst = (int)(size_t)L8 + L_STAGE;
#pragma unroll
            for (int i = 0; i < 4; ++i)
                asm volatile("s_mov_b32 m0, %4\n\ts_nop 0\n\t"
                             "buffer_load_dwordx4 %0, %2, %3 offen lds\n\t"
                             "buffer_load_dwordx4 %1, %2, %3 offen offset:1024 lds"
                             :: "v"(xv0), "v"(xv1), "s"(xr), "s"(soff + (((rank + 1 + i0 + i) & 7) << 18)), "s"(lst + i * 2048) : "memory");
            if (b < 64 && b >= HOLD) asm volatile("s_barrier" ::: "memory");
            STAMP(1);
            const int rv = L_STAGE + (lane & 31) * 64 + ((((lane >> 5) * 2) ^ ((lane >> 2) & 3)) * 16);
            unsigned char* pe[4];
#pragma unroll
            for (int i = 0; i < 4; ++i) pe[i] = i0 ? p.peer[i < 3 ? 4 + i : 6] : p.peer[i];
            unsigned callr;
            asm volatile("s_waitcnt vmcnt(0)\n\tv_readfirstlane_b32 %0, %1" : "=s"(callr) : "v"(callv) : "memory");
            const unsigned call = callr + 1u;
            STAMP(2);
            v4u d[4][2];
#pragma unroll
            for (int i = 0; i < 4; ++i) {
                d[i][0] = *(LAS v4u*)(L8 + i * 2048 + rv);
                d[i][1] = *(LAS v4u*)(L8 + i * 2048 + (rv ^ 16));
                d[i][0].x = max(~d[i][0].x, 1u);
                d[i][1].x = max(~d[i][1].x, 1u);
            }
            const int bo = (int)WS_A + (int)(call % 3u + 1u) * A_PAR;
            const int dso = bo + (rg * NREM + rd * 7 + 6 - i0) * FRAG;
            const int fso = bo + A_FLAG + rg * NREP * FSTRIDE + rd * 8 + 6 - i0;
            const int fvo = lane < NREP ? lane * FSTRIDE : (int)WS_END;
            const unsigned due = SLOT_T0 + ((unsigned)(b >> 1) * SLOT_Q >> 4);
            while ((unsigned)__builtin_amdgcn_s_memrealtime() - tstart < due) __builtin_amdgcn_s_sleep(1);
            STAMP(3);
#pragma unroll
            for (int i = 0; i < 4; ++i) {
                const auto dr = __builtin_amdgcn_make_buffer_rsrc((void*)pe[i], 0, (i == 3 && i0) ? 0 : (int)WS_END, 0x00020000);
                store16<SEND_AUX>(d[i][0], dr, lane * 16, dso - i * FRAG);
                store16<SEND_AUX>(d[i][1], dr, lane * 16 + 1024, dso - i * FRAG);
                __builtin_amdgcn_raw_buffer_store_b8((unsigned char)1, dr, fvo, fso - i, 17);
            }
            if (b < HOLD) asm volatile("s_barrier" ::: "memory");
            STAMP(4);
            if (lane == 0) __hip_atomic_store(ctr, call, __ATOMIC_RELAXED, __HIP_MEMORY_SCOPE_AGENT);
        }
        STAMP_OUT();
    } else if (wave >= 4) {
        __builtin_amdgcn_s_setprio(PRIO_COMP);
        const int cw = wave - 4;
        const int mh = cw & 1, kp = cw >> 1;
        const int rg = mb * 2 + mh;
        const unsigned char* ab = p.myws + WS_A + rg * NREM * FRAG;
        v4i xr, ar;
        xr.x = (int)(size_t)p.x; xr.y = (int)((size_t)p.x >> 32) & 0xffff; xr.z = 4096 * SH; xr.w = 0x00020000;
        ar.x = (int)(size_t)ab; ar.y = (int)((size_t)ab >> 32) & 0xffff; ar.z = 3 * A_PAR + NREM * FRAG; ar.w = 0x00020000;
        const int avoff = lane * 16;
        const int xv0 = (lane >> 3) * SH + (((lane & 7) ^ ((lane >> 4) & 3)) * 16);
        const int xv1 = (xv0 ^ 64) + 8 * SH - 1024;
        const int xsoff = (rank * SH + rg * 32) * SH + kp * 128;
        const int lb = (int)(size_t)L8;
        const int stg = lb + (kp * RING + RING - 2 + mh) * SLOT;
        const int rv = stg + (lane & 31) * 128 + ((((lane >> 5) * 2) ^ ((lane >> 1) & 7)) * 16);
        const int g = lane >> 4, q = (lane & 15) >> 1;
        const int rdl = lb + kp * RING * SLOT + ((g >> 1) * 32 + q) * 128 + (g & 1) * 16 + (lane & 1) * 8;
        const int sw = (q >> 1) ^ (2 * kp);
        const int rd0 = rdl + ((0 ^ sw) * 32), rd1 = rdl + ((1 ^ sw) * 32);
        const int rd2 = rdl + ((2 ^ sw) * 32), rd3 = rdl + ((3 ^ sw) * 32);
        const int a_ardy = lb + L_SYNC + 4 * mh, a_wrdy = lb + L_SYNC + 8 + 4 * kp, a_cons = lb + L_SYNC + 16 + 4 * cw;
        int aoff = kp * FRAG;
        v16f acc0, acc1, acc2, acc3;
        v16i clob0, clob1, clob2, clob3, clob4, clob5;
        int va0, va1, va2, va3, vt0, vt1, vt2, vt3;
        int s_valid, s_need, s_wslot, s_wrdy, s_ardy, s_t0, s_t1, s_so;
        unsigned long long s_ret, s_now;
        const unsigned tlo = (unsigned)tentry;
#define DBG_ASM(s_)
#define DBG_SIDE(s_)
#define DBG_ASM0
#define DBG_OPS
#define AREG(s_) "v[160+8*" #s_ ":167+8*" #s_ "]"
#define ALO(s_) "v[160+8*" #s_ ":163+8*" #s_ "]"
#define AHI(s_) "v[164+8*" #s_ ":167+8*" #s_ "]"
#define LDLO(p_) "buffer_load_dwordx4 " ALO(p_) ", %[avoff], %[ar], %[so] offen sc1\n\t"
#define LDHI(p_) "buffer_load_dwordx4 " AHI(p_) ", %[avoff], %[ar], %[so] offen offset:1024 sc1\n\t"
#define RDB(base_, va_) \
        "ds_read_b64_tr_b8 v[" #base_ ":" #base_ "+1], " va_ "\n\t" \
        "ds_read_b64_tr_b8 v[" #base_ "+2:" #base_ "+3], " va_ " offset:1024\n\t" \
        "ds_read_b64_tr_b8 v[" #base_ "+4:" #base_ "+5], " va_ " offset:2048\n\t" \
        "ds_read_b64_tr_b8 v[" #base_ "+6:" #base_ "+7], " va_ " offset:3072\n\t"
#define HERE \
        "s_and_b32 %[t1], %[ardy], 0x600000\n\t" \
        "s_or_b32 %[t1], %[t1], %[aoff]\n\t" \
        "s_cmp_lt_u32 %[t1], %[ardy]\n\t"
#define PICK \
        "v_readfirstlane_b32 %[t1], %[vt0]\n\t" \
        "v_readfirstlane_b32 %[wrdy], %[vt1]\n\t" \
        "s_max_u32 %[ardy], %[ardy], %[t1]\n\t" \
        HERE
#define AHITX
#define TAKE \
        "s_cselect_b32 %[so], %[t1], %[oob]\n\t" \
        "s_cselect_b32 %[t0], 0x1000, 0\n\t" \
        "s_addc_u32 %[valid], %[valid], %[valid]\n\t" \
        "s_add_u32 %[aoff], %[aoff], %[t0]\n\t" AHITX
#define COND \
        "s_bitcmp1_b32 %[valid], 7\n\t" \
        "s_cselect_b32 %[t0], %[wrdy], 0\n\t" \
        "s_cmp_lt_u32 %[t0], %[need]\n\t"
#define CHKBR(s_) "s_cbranch_vccnz .Lretry" #s_ "_%=\n"
#define LG9 "s_waitcnt lgkmcnt(9)\n\t"
#define SPEC(s_) "s_cmp_lg_u32 %[ardy], 0\n\ts_cbranch_scc0 .Lidle" #s_ "_%=\n"
#define MF(x_) x_
#define POS(s_, p_) \
        ".Lpos" #s_ "_%=:\n\t" \
        "s_cbranch_scc1 .Lslow" #s_ "_%=\n" \
        ".Lcomp" #s_ "_%=:\n\t" \
        "s_waitcnt vmcnt(12)\n" \
        ".Lchk" #s_ "_%=:\n\t" \
        "v_min_u32 %[vt3], v[160+8*" #s_ "], v[164+8*" #s_ "]\n\t" \
        "v_cmp_eq_u32 vcc, 0, %[vt3]\n\t" \
        "v_not_b32 v[160+8*" #s_ "], v[160+8*" #s_ "]\n\t" \
        "v_not_b32 v[164+8*" #s_ "], v[164+8*" #s_ "]\n\t" \
        "s_waitcnt lgkmcnt(11)\n\t" \
        CHKBR(s_) \
        ".Lok" #s_ "_%=:\n\t" \
        MF("v_mfma_f32_32x32x64_f8f6f4 v[64:79], v[128:135], " AREG(s_) ", v[64:79] cbsz:1 blgp:1\n\t") \
        LDLO(p_) \
        RDB(128, "%[va0]") \
        "s_add_u32 %[wslot], %[wslot], %[slotb]\n\t" \
        "s_cmp_eq_u32 %[wslot], %[ringb]\n\t" \
        "s_cselect_b32 %[wslot], 0, %[wslot]\n\t" \
        "v_add_u32 %[va0], %[wslot], %[rd0]\n\t" \
        "s_waitcnt lgkmcnt(11)\n\t" \
        MF("v_mfma_f32_32x32x64_f8f6f4 v[80:95], v[136:143], " AREG(s_) ", v[80:95] cbsz:1 blgp:1\n\t") \
        LDHI(p_) \
        RDB(136, "%[va1]") \
        "v_add_u32 %[va1], %[wslot], %[rd1]\n\t" \
        "v_add_u32 %[vt2], 1, %[vt2]\n\t" \
        "s_add_u32 %[need], %[need], 1\n\t" \
        DBG_ASM(s_) \
        LG9 \
        MF("v_mfma_f32_32x32x64_f8f6f4 v[96:111], v[144:151], " AREG(s_) ", v[96:111] cbsz:1 blgp:1\n\t") \
        RDB(144, "%[va2]") \
        "v_add_u32 %[va2], %[wslot], %[rd2]\n\t" \
        PICK TAKE \
        MF("v_mfma_f32_32x32x64_f8f6f4 v[112:127], v[152:159], " AREG(s_) ", v[112:127] cbsz:1 blgp:1\n\t") \
        RDB(152, "%[va3]") \
        "ds_read_b32 %[vt0], %[aardy]\n\t" \
        "ds_read_b32 %[vt1], %[awrdy]\n\t" \
        "ds_write_b32 %[acons], %[vt2]\n\t" \
        "v_add_u32 %[va3], %[wslot], %[rd3]\n\t" \
        COND
#define SIDE(s_, p_, n_) \
        ".Lslow" #s_ "_%=:\n\t" \
        "s_bitcmp1_b32 %[valid], 7\n\t" \
        "s_cbranch_scc1 .Lwspin" #s_ "_%=\n\t" \
        "s_cmp_eq_u32 %[need], 34\n\t" \
        "s_cbranch_scc1 .Ldone_%=\n\t" \
        LDLO(p_) LDHI(p_) \
        HERE \
        "s_cbranch_scc1 .Ltake" #s_ "_%=\n\t" \
        "s_cmp_lt_u32 %[aoff], %[aend]\n\t" \
        "s_cbranch_scc0 .Ltake" #s_ "_%=\n" \
        ".Lidle" #s_ "_%=:\n\t" \
        "ds_read_b32 %[vt0], %[aardy]\n\t" \
        "ds_read_b32 %[vt1], %[awrdy]\n\t" \
        "s_and_b32 %[t0], %[valid], 0x7f\n\t" \
        "s_waitcnt lgkmcnt(0)\n\t" \
        PICK \
        "s_cbranch_scc1 .Ltake" #s_ "_%=\n\t" \
        "s_cmp_eq_u32 %[t0], 0\n\t" \
        "s_cbranch_scc0 .Ltake" #s_ "_%=\n\t" \
        SPEC(s_) \
        ".Ltake" #s_ "_%=:\n\t" \
        TAKE COND \
        "s_branch .Lpos" #n_ "_%=\n" \
        ".Lwspin" #s_ "_%=:\n\t" \
        "ds_read_b32 %[vt1], %[awrdy]\n\t" \
        "s_waitcnt lgkmcnt(0)\n\t" \
        "v_readfirstlane_b32 %[wrdy], %[vt1]\n\t" \
        "s_cmp_lt_u32 %[wrdy], %[need]\n\t" \
        "s_cbranch_scc1 .Lwspin" #s_ "_%=\n\t" \
        "s_branch .Lcomp" #s_ "_%=\n" \
        ".Lretry" #s_ "_%=:\n\t" \
        "s_getpc_b64 %[ret]\n\t" \
        "s_branch .Lagain_%=\n\t" \
        "s_cbranch_scc1 .Lok" #s_ "_%=\n\t" \
        "buffer_load_dwordx4 " ALO(s_) ", %[avoff], %[ar], %[t1] offen sc1\n\t" \
        "buffer_load_dwordx4 " AHI(s_) ", %[avoff], %[ar], %[t1] offen offset:1024 sc1\n\t" \
        "s_waitcnt vmcnt(0)\n\t" \
        "s_branch .Lchk" #s_ "_%=\n" \
        DBG_SIDE(s_)
#define XLD(so_) \
        "buffer_load_dwordx4 %[xv0], %[xr], " so_ " offen lds\n\t" \
        "buffer_load_dwordx4 %[xv1], %[xr], " so_ " offen offset:1024 lds\n\t" \
        "buffer_load_dwordx4 %[vt0], %[xr], " so_ " offen offset:2048 lds\n\t" \
        "buffer_load_dwordx4 %[vt1], %[xr], " so_ " offen offset:3072 lds\n\t"
#define XRD(d_, off_) \
        "ds_read_b128 v[" #d_ ":" #d_ "+3], %[rv] offset:" #off_ "\n\t" \
        "ds_read_b128 v[" #d_ "+4:" #d_ "+7], %[va1] offset:" #off_ "\n\t" \
        "ds_read_b128 v[" #d_ "+8:" #d_ "+11], %[va2] offset:" #off_ "\n\t" \
        "ds_read_b128 v[" #d_ "+12:" #d_ "+15], %[va3] offset:" #off_ "\n\t"
#define XINV(r_) "v_not_b32 v" #r_ ", v" #r_ "\n\tv_max_u32 v" #r_ ", 1, v" #r_ "\n\t"
#define Z8(b_) \
        "v_mov_b64 v[" #b_ ":" #b_ "+1], 0\n\tv_mov_b64 v[" #b_ "+2:" #b_ "+3], 0\n\t" \
        "v_mov_b64 v[" #b_ "+4:" #b_ "+5], 0\n\tv_mov_b64 v[" #b_ "+6:" #b_ "+7], 0\n\t"
        asm volatile(
            "v_add_u32 %[vt0], 0x1800, %[xv0]\n\t"
            "v_add_u32 %[vt1], 0x1800, %[xv1]\n\t"
            "s_mov_b32 m0, %[stg]\n\t"
            "s_add_u32 %[t0], %[xsoff], 256\n\t"
            "s_barrier\n\t"
            XLD("%[xsoff]")
            "s_add_u32 m0, m0, 4096\n\t"
            "s_nop 0\n\t"
            XLD("%[t0]")
            "v_xor_b32 %[va1], 16, %[rv]\n\t"
            "v_xor_b32 %[va2], 64, %[rv]\n\t"
            "v_xor_b32 %[va3], 0x50, %[rv]\n\t"
            Z8(64) Z8(72) Z8(80) Z8(88) Z8(96) Z8(104) Z8(112) Z8(120)
            "s_mov_b32 %[valid], 0xf0\n\t"
            "s_mov_b32 %[need], 2\n\t"
            "s_mov_b32 %[wslot], %[slotb]\n\t"
            "s_mov_b32 %[so], %[oob]\n\t"
            "s_mov_b32 %[ardy], 0\n\t"
            "v_mov_b32 %[vt2], 1\n\t"
            "s_waitcnt vmcnt(4)\n\t"
            XRD(160, 0)
            "s_waitcnt vmcnt(0)\n\t"
            XRD(176, 4096)
            "ds_read_b32 %[vt0], %[aardy]\n"
            ".Lfirst_%=:\n\t"
            "ds_read_b32 %[vt1], %[awrdy]\n\t"
            "s_waitcnt lgkmcnt(0)\n\t"
            "v_readfirstlane_b32 %[wrdy], %[vt1]\n\t"
            "s_cmp_eq_u32 %[wrdy], 0\n\t"
            "s_cbranch_scc1 .Lfirst_%=\n\t"
            DBG_ASM0
            XINV(160) XINV(164) XINV(168) XINV(172) XINV(176) XINV(180) XINV(184) XINV(188)
            RDB(128, "%[rd0]") RDB(136, "%[rd1]") RDB(144, "%[rd2]") RDB(152, "%[rd3]")
            "ds_write_b32 %[acons], %[vt2]\n\t"
            "v_add_u32 %[va0], %[wslot], %[rd0]\n\t"
            "v_add_u32 %[va1], %[wslot], %[rd1]\n\t"
            "v_add_u32 %[va2], %[wslot], %[rd2]\n\t"
            "v_add_u32 %[va3], %[wslot], %[rd3]\n\t"
            "s_waitcnt lgkmcnt(1)\n\t"
            COND
            POS(0, 7) POS(1, 0) POS(2, 1) POS(3, 2) POS(4, 3) POS(5, 4) POS(6, 5) POS(7, 6)
            "s_branch .Lpos0_%=\n"
            SIDE(0, 7, 1) SIDE(1, 0, 2) SIDE(2, 1, 3) SIDE(3, 2, 4) SIDE(4, 3, 5) SIDE(5, 4, 6) SIDE(6, 5, 7) SIDE(7, 6, 0)
            ".Lagain_%=:\n\t"
            "s_sleep %[rslp]\n\t"
            "s_memrealtime %[now]\n\t"
            "s_add_u32 s40, s40, 4\n\t"
            "s_addc_u32 s41, s41, 0\n\t"
            "s_waitcnt lgkmcnt(0)\n\t"
            "s_lshl_b32 %[t1], %[need], 12\n\t"
            "s_add_u32 %[t1], %[t1], %[abias]\n\t"
            "s_mul_i32 s43, %[t1], 37\n\t"
            "s_lshr_b32 s43, s43, 19\n\t"
            "s_add_u32 s43, s43, 1\n\t"
            "s_mul_i32 s43, s43, 0x3800\n\t"
            "s_and_b32 %[t0], %[ardy], 0x600000\n\t"
            "s_or_b32 %[t1], %[t1], %[t0]\n\t"
            "s_or_b32 s43, s43, %[t0]\n\t"
            "s_max_u32 %[ardy], %[ardy], s43\n\t"
            "s_sub_u32 %[t0], s42, %[tlo]\n\t"
            "s_cmp_gt_u32 %[t0], %[dl]\n\t"
            "s_cbranch_scc0 .Lback_%=\n\t"
            "v_mov_b32 %[vt3], 0x7fc00000\n\t"
            "ds_write_b32 %[ascale], %[vt3]\n"
            ".Lback_%=:\n\t"
            "s_setpc_b64 %[ret]\n"
            ".Ldone_%=:\n\t"
            "s_waitcnt vmcnt(0) lgkmcnt(0)\n\t"
            "s_nop 15"
            : "=&{v[64:79]}"(acc0), "=&{v[80:95]}"(acc1), "=&{v[96:111]}"(acc2), "=&{v[112:127]}"(acc3),
              "=&{v[128:143]}"(clob0), "=&{v[144:159]}"(clob1), "=&{v[160:175]}"(clob2), "=&{v[176:191]}"(clob3),
              "=&{v[192:207]}"(clob4), "=&{v[208:223]}"(clob5),
              [va0] "=&v"(va0), [va1] "=&v"(va1), [va2] "=&v"(va2), [va3] "=&v"(va3),
              [vt0] "=&v"(vt0), [vt1] "=&v"(vt1), [vt2] "=&v"(vt2), [vt3] "=&v"(vt3),
              [ret] "=&{s[40:41]}"(s_ret), [now] "=&{s[42:43]}"(s_now),
              [valid] "=&s"(s_valid), [need] "=&s"(s_need), [wslot] "=&s"(s_wslot),
              [wrdy] "=&s"(s_wrdy), [ardy] "=&s"(s_ardy), [t0] "=&s"(s_t0), [t1] "=&s"(s_t1), [so] "=&s"(s_so),
              [aoff] "+s"(aoff) DBG_OPS
            : [xr] "s"(xr), [ar] "s"(ar), [xsoff] "s"(xsoff), [xv0] "v"(xv0), [xv1] "v"(xv1), [avoff] "v"(avoff),
              [rd0] "v"(rd0), [rd1] "v"(rd1), [rd2] "v"(rd2), [rd3] "v"(rd3),
              [stg] "s"(stg), [rv] "v"(rv), [oob] "s"(A_OOB), [aardy] "v"(a_ardy), [awrdy] "v"(a_wrdy), [acons] "v"(a_cons),
              [slotb] "i"(SLOT), [ringb] "i"(RING * SLOT), [aend] "i"(NREM * FRAG),
              [abias] "s"(kp * FRAG - 6 * 4096), [rslp] "i"(RETRY_SLEEP), [tlo] "s"(tlo), [dl] "i"((unsigned)DEADLINE_TICKS), [ascale] "v"(lb + L_SYNC + 48)
            : "memory", "scc", "vcc");
        STAMP(5);
        {
            while (rfl(*(volatile LAS unsigned*)(sync + 4 + (cw ^ 1))) < (unsigned)(36 - RING)) {}
            asm volatile("" ::: "memory");
            const int s0 = (mh ? (34 - RING) % RING : (32 - RING) % RING) * SLOT;
            const int s1 = (mh ? (35 - RING) % RING : (33 - RING) % RING) * SLOT;
            const int wa = kp * RING * SLOT + (lane & 31) * 256 + (((lane >> 5) ^ (lane & 7)) * 16);
#pragma unroll
            for (int gq = 0; gq < 4; ++gq) {
                *(LAS v4f*)(L8 + s1 + (wa ^ (gq * 32))) = v4f{acc2[4 * gq], acc2[4 * gq + 1], acc2[4 * gq + 2], acc2[4 * gq + 3]};
                *(LAS v4f*)(L8 + s1 + (wa ^ (gq * 32)) + 128) = v4f{acc3[4 * gq], acc3[4 * gq + 1], acc3[4 * gq + 2], acc3[4 * gq + 3]};
            }
            asm volatile("" ::: "memory");
            *(volatile LAS unsigned*)(sync + 8 + cw) = 1u;
            asm volatile("" ::: "memory");
#pragma unroll
            for (int gq = 0; gq < 4; ++gq) {
                *(LAS v4f*)(L8 + s0 + (wa ^ (gq * 32))) = v4f{acc0[4 * gq], acc0[4 * gq + 1], acc0[4 * gq + 2], acc0[4 * gq + 3]};
                *(LAS v4f*)(L8 + s0 + (wa ^ (gq * 32)) + 128) = v4f{acc1[4 * gq], acc1[4 * gq + 1], acc1[4 * gq + 2], acc1[4 * gq + 3]};
            }
            const auto orr = __builtin_amdgcn_make_buffer_rsrc((void*)p.out, 0, SH * GN * 4, 0x00020000);
            const int ovoff = (lane >> 4) * GN * 4 + (lane & 15) * 16;
            const int osoff = (rg * 32 * GN + n0 + kp * 64) * 4;
            const int ra = (lane >> 4) * 256 + (((lane & 15) ^ (lane >> 4)) * 16);
            const int own = kp * RING * SLOT + s0, oth = (kp ^ 1) * RING * SLOT + s1;
            while (rfl(*(volatile LAS unsigned*)(sync + 8 + (cw ^ 2))) == 0u) {}
            asm volatile("" ::: "memory");
            const float sc = *(volatile LAS float*)(sync + 12);
            v4f u0[8], u1[8];
#pragma unroll
            for (int i = 0; i < 8; ++i) {
                u0[i] = *(LAS v4f*)(L8 + own + i * 1024 + (ra ^ ((i & 1) * 64)));
                u1[i] = *(LAS v4f*)(L8 + oth + i * 1024 + (ra ^ ((i & 1) * 64)));
            }
#pragma unroll
            for (int i = 0; i < 8; ++i) {
                const v4f y = __builtin_elementwise_maximum((u0[i] + u1[i]) * sc, v4f{0.f, 0.f, 0.f, 0.f});
                store16<OUT_AUX>(__builtin_bit_cast(v4u, y), orr, ovoff, osoff + i * 4 * GN * 4);
            }
        }
        STAMP(6);
        STAMP_OUT();
    } else if (wave < 3) {
        __builtin_amdgcn_s_setprio(PRIO_LOAD);
        const int kp = wave - 1;
        v4i rs;
        rs.x = (int)(size_t)p.w; rs.y = (int)((size_t)p.w >> 32) & 0xffff; rs.z = GK * GN; rs.w = 0x00020000;
        const int v0 = (lane >> 3) * GN + n0 + (((lane & 7) ^ (((lane >> 4) & 3) << 1)) * 16);
        const int v1 = v0 + 8 * GN - 1024, v2 = v0 + 16 * GN - 2048, v3 = v0 + 24 * GN - 3072;
        int tab;
        {
            const int n = lane & 31;
            const int ri = 2 * (n - 4) + kp, jj = (ri * 37) >> 8;
            const int kb = n < 4 ? rank * SH + ((n >> 1) * 4 + kp * 2 + (n & 1)) * 64
                                 : ((rank + 1 + ri - jj * 7) & 7) * SH + (jj ^ (mb & ROT)) * 64;
            const int need = n >= RING ? n - RING + 1 : (n >= RING - 2 ? 1 : 0);
            tab = lane < 32 ? kb * GN + (n % RING) * SLOT + need : 63;
        }
        const int lbase = (int)(size_t)L8 + kp * RING * SLOT;
        const int a_wrdy = (int)(size_t)L8 + L_SYNC + 8 + 4 * kp, a_cons = (int)(size_t)L8 + L_SYNC + 16 + 8 * kp;
        int s_n, s_e, s_so, s_so2, s_need, s_t, vpub;
        v2i vc;
#define WLD(v_, so_, off_) "buffer_load_dwordx4 %[" #v_ "], %[rs], %[" #so_ "] offen offset:" #off_ " lds\n\t"
#define FILL \
        WLD(v0, so, 0) \
        "ds_read_b64 %[vc], %[acons]\n\t" \
        "s_add_u32 %[n], %[n], 1\n\t" \
        WLD(v1, so, 1024) \
        "v_readlane_b32 %[e], %[tab], %[n]\n\t" \
        WLD(v2, so, 2048) \
        "v_add_u32 %[vpub], 1, %[vpub]\n\t" \
        WLD(v3, so, 3072) \
        "s_add_u32 m0, m0, 4096\n\t" \
        "s_and_b32 %[so], %[e], 0xfffc0000\n\t" \
        WLD(v0, so2, 0) \
        "s_and_b32 %[need], %[e], 63\n\t" \
        "s_and_b32 %[e], %[e], 0x1e000\n\t" \
        WLD(v1, so2, 1024) \
        "s_waitcnt lgkmcnt(0)\n\t" \
        "v_min_u32 v20, v20, v21\n\t" \
        WLD(v2, so2, 2048) \
        "v_readfirstlane_b32 %[t], v20\n\t" \
        WLD(v3, so2, 3072) \
        "s_add_u32 m0, %[e], %[lbase]\n\t" \
        "s_add_u32 %[so2], %[so], 0x20000\n\t"
#define PUB(cnt_) \
        "s_waitcnt vmcnt(" #cnt_ ")\n\t" \
        "ds_write_b32 %[awrdy], %[vpub]\n\t"
#define PUBN(cnt_) "v_add_u32 %[vpub], 1, %[vpub]\n\t" PUB(cnt_)
#define LDBG_IN
#define LDBG_OUT
#define LDBG_OPS
        asm volatile(
            "s_mov_b32 %[n], 0\n\t"
            "v_mov_b32 %[vpub], %[pub0]\n\t"
            "v_readlane_b32 %[so], %[tab], 0\n\t"
            "s_mov_b32 m0, %[lbase]\n\t"
            "s_add_u32 %[so2], %[so], 0x20000\n\t"
            "s_nop 2\n\t"
            "s_barrier\n"
            ".Lpro_%=:\n\t"
            FILL
            "s_cmp_lt_u32 %[n], %[nf]\n\t"
            "s_cbranch_scc1 .Lpro_%=\n"
            ".Lmain_%=:\n\t"
            PUB(%[cnt])
            ".Lgo_%=:\n\t"
            LDBG_IN
            FILL
            "s_cmp_lt_u32 %[t], %[need]\n\t"
            "s_cbranch_scc0 .Lmain_%=\n\t"
            PUB(%[cnt])
            "s_cmp_eq_u32 %[n], 32\n\t"
            "s_cbranch_scc1 .Lend_%=\n"
            ".Lspin_%=:\n\t"
            "ds_read_b64 %[vc], %[acons]\n\t"
            "s_waitcnt lgkmcnt(0)\n\t"
            "v_min_u32 v20, v20, v21\n\t"
            "s_nop 0\n\t"
            "v_readfirstlane_b32 %[t], v20\n\t"
            "s_cmp_lt_u32 %[t], %[need]\n\t"
            "s_cbranch_scc0 .Lgo_%=\n\t"
            "s_sleep 1\n\t"
            "s_branch .Lspin_%=\n"
            LDBG_OUT
            ".Lend_%=:\n\t"
            PUBN(16)
            PUBN(8)
            "v_add_u32 %[vpub], 32, %[vpub]\n\t"
            PUB(0)
            "s_waitcnt lgkmcnt(0)"
            : [n] "=&s"(s_n), [e] "=&s"(s_e), [so] "=&s"(s_so), [so2] "=&s"(s_so2), [need] "=&s"(s_need), [t] "=&s"(s_t),
              [vpub] "=&v"(vpub), [vc] "=&{v[20:21]}"(vc) LDBG_OPS
            : [tab] "v"(tab), [rs] "s"(rs), [lbase] "s"(lbase), [v0] "v"(v0), [v1] "v"(v1), [v2] "v"(v2), [v3] "v"(v3),
              [awrdy] "v"(a_wrdy), [acons] "v"(a_cons), [nf] "i"(NFLY), [cnt] "i"(8 * (NFLY - 1)), [pub0] "i"(1 - NFLY)
            : "memory", "scc");
        STAMP(7);
        STAMP_OUT();
    } else {
        __builtin_amdgcn_s_setprio(3);
        sync[lane] = 0u;
        const unsigned callv = __hip_atomic_load(ctr, __ATOMIC_RELAXED, __HIP_MEMORY_SCOPE_AGENT);
        asm volatile("s_waitcnt lgkmcnt(0)\n\ts_barrier" ::: "memory");
        {
            const int a7 = (int)((size_t)p.w >> 7) + xg * 4;
            const int tb = ((3 - a7) & 7) < 4 ? (3 - a7) & 7 : (31 - a7) & 31;
            if (tb < 4) {
                if (WARM_SLEEP) __builtin_amdgcn_s_sleep(WARM_SLEEP);
                v4i wr;
                wr.x = (int)(size_t)p.w; wr.y = (int)((size_t)p.w >> 32) & 0xffff; wr.z = GK * GN; wr.w = 0x00020000;
                int sink0, sink1;
                asm volatile("s_nop 4\n\tbuffer_load_dword %0, %2, %3, %4 offen\n\tbuffer_load_dword %1, %2, %3, %5 offen\n\ts_waitcnt vmcnt(0)"
                             : "=&v"(sink0), "=&v"(sink1)
                             : "v"(lane * GN + (xg * 4 + tb) * 128), "s"(wr), "s"(idx * 128 * GN), "s"((idx * 128 + 64) * GN) : "memory");
            }
        }
        *(volatile LAS float*)(sync + 12) = p.sx[0] * p.sw[0];
        const unsigned call = rfl(callv) + 1u;
        __hip_atomic_store(ctr, call, __ATOMIC_RELAXED, __HIP_MEMORY_SCOPE_AGENT);
                GAS unsigned* tmo = (GAS unsigned*)(p.myws + WS_TMO);
        const unsigned long long t0 = tentry;
        const unsigned tag = (call % 3u + 1u) << 21;
        *(volatile LAS v2u*)(sync) = v2u{tag, tag};
        const unsigned char* fl = p.myws + WS_A + tag + A_FLAG + (mb * 2 * NREP + (xg & (NREP - 1))) * FSTRIDE;
        bool zeroed = false;
        const auto zr = __builtin_amdgcn_make_buffer_rsrc((void*)(p.myws + WS_A), 0, 4 * A_PAR, 0x00020000);
        const int zb = (int)((call + 2u) % 3u + 1u) * A_PAR, zo = zb + b * 7168;
#define ZERO_FILL() do { if (!zeroed) { zeroed = true; \
            _Pragma("unroll") for (int u = 0; u < 7; ++u) store16<17>(v4u{0u, 0u, 0u, 0u}, zr, lane * 16, zo + u * 1024); \
            store16<17>(v4u{0u, 0u, 0u, 0u}, zr, lane < 4 && b < 16 * NREP ? lane * 16 : A_OOB, zb + A_FLAG + b * FSTRIDE); } } while (0)
        if (ZERO_AT == 0) ZERO_FILL();
        unsigned spins = 0;
        int r0 = 0, r1 = 0;
        for (;;) {
            v4l f, h;
            asm volatile("s_load_dwordx8 %0, %2, %3 glc\n\ts_load_dwordx8 %1, %2, %4 glc\n\ts_waitcnt lgkmcnt(0)"
                         : "=&s"(f), "=&s"(h) : "s"(fl), "s"(r0 * 8), "s"(NREP * FSTRIDE + r1 * 8) : "memory");
            constexpr unsigned long long ALL7 = 0x0001010101010101ull;
            const int a0 = f[0] != ALL7 ? 0 : f[1] != ALL7 ? 1 : f[2] != ALL7 ? 2 : f[3] != ALL7 ? 3 : 4;
            const int a1 = h[0] != ALL7 ? 0 : h[1] != ALL7 ? 1 : h[2] != ALL7 ? 2 : h[3] != ALL7 ? 3 : 4;
            if (a0 + a1) {
                r0 += a0; r1 += a1;
                *(volatile LAS v2u*)(sync) = v2u{(unsigned)r0 * (7 * FRAG) | tag, (unsigned)r1 * (7 * FRAG) | tag};
                if (r0 + r1 >= ZERO_AT) ZERO_FILL();
                if (r0 + r1 == 16) break;
            }
            if ((++spins & 255u) == 0u) {
                if (rfl(__hip_atomic_load(tmo, __ATOMIC_RELAXED, __HIP_MEMORY_SCOPE_AGENT)) == call
                    || __builtin_amdgcn_s_memrealtime() - t0 > DEADLINE_TICKS) {
                    __hip_atomic_store(tmo, call, __ATOMIC_RELAXED, __HIP_MEMORY_SCOPE_AGENT);
                    *(volatile LAS float*)(sync + 12) = __builtin_nanf("");
                    *(volatile LAS v2u*)(sync) = v2u{(unsigned)NREM * FRAG | tag, (unsigned)NREM * FRAG | tag};
                    break;
                }
            }
            if (POLL_SLEEP) __builtin_amdgcn_s_sleep(POLL_SLEEP);
        }
        ZERO_FILL();
        STAMP(7);
        STAMP_OUT();
    }
}

extern "C" void kernel_launch(int rank, int nranks, ncclComm_t comm,
                              hipStream_t stream,
                              void* const* d_in, const int* in_sizes, int n_in,
                              void* d_out, int out_size,
                              const int* in_dtypes, int out_dtype,
                              void* const* d_out_all,
                              void* const* d_ws_all, size_t ws_bytes) {
    if (nranks != NR || (unsigned)rank >= (unsigned)NR || ws_bytes < WS_END || n_in < 4 ||
        in_sizes[0] != 4096 * SH || in_sizes[1] != GK * GN || out_size != SH * GN) {
        fprintf(stderr, "unexpected problem geometry\n");
        return;
    }
    HIP_CHECK(hipFuncSetAttribute((const void*)a2a_gemm_kernel, hipFuncAttributeMaxDynamicSharedMemorySize, LDS_BYTES));
    Params p;
    p.x = (const unsigned char*)d_in[0];
    p.w = (const unsigned char*)d_in[1];
    p.sx = (const float*)d_in[2];
    p.sw = (const float*)d_in[3];
    p.out = (float*)d_out;
    p.myws = (unsigned char*)d_ws_all[rank];
    p.rank = rank;
    p.pad = 0;
    for (int i = 0; i < NR - 1; ++i) p.peer[i] = (unsigned char*)d_ws_all[(rank + 1 + i) & 7];
    a2a_gemm_kernel<<<NWG, NTHREADS, LDS_BYTES, stream>>>(p);
    HIP_CHECK(hipGetLastError());
}
```

```cpp
#include <hip/hip_runtime.h>
#include <rccl/rccl.h>
#include <cstdio>

#define HIP_CHECK(expr) do { hipError_t _e = (expr); if (_e != hipSuccess) { \
    fprintf(stderr, "HIP error %s at %s:%d\n", hipGetErrorString(_e), __FILE__, __LINE__); \
    return; } } while (0)

#define GAS __attribute__((address_space(1)))
#define LAS __attribute__((address_space(3)))
typedef int v2i __attribute__((ext_vector_type(2)));
typedef int v4i __attribute__((ext_vector_type(4)));
typedef int v8i __attribute__((ext_vector_type(8)));
typedef unsigned v2u __attribute__((ext_vector_type(2)));
typedef unsigned v4u __attribute__((ext_vector_type(4)));
typedef float v4f __attribute__((ext_vector_type(4)));
typedef float v16f __attribute__((ext_vector_type(16)));
typedef int v16i __attribute__((ext_vector_type(16)));
typedef unsigned long long v4l __attribute__((ext_vector_type(4)));

constexpr int NR = 8;
constexpr int GN = 4096;
constexpr int GK = 4096;
constexpr int SH = 512;
constexpr int NWG = 256;
constexpr int NTHREADS = 8 * 64;
constexpr int NREM = 56;
constexpr int FRAG = 2048;

constexpr size_t WS_CTR = 0;
constexpr size_t WS_TMO = 32 * 1024;
constexpr size_t WS_DBG = 64 * 1024;
constexpr size_t WS_A = 0;
constexpr int A_PAR = 1 << 21;
static_assert(16 * NREM * FRAG <= A_PAR, "");
constexpr size_t WS_END = WS_A + 4 * (size_t)A_PAR;
constexpr int A_OOB = 4 * A_PAR;

#define RING 9
#define SLOT_Q 76
#define SLOT_T0 30
#define NFLY 4
#define PRIO_COMP 2
#define PRIO_LOAD 1
#define NREP 4
#define POLL_SLEEP 2
constexpr int A_FLAG = 16 * NREM * FRAG;
constexpr int FSTRIDE = 3840;
static_assert(A_FLAG + 16 * NREP * FSTRIDE <= A_PAR, "");
#define SEND_AUX 17
#define ZERO_AT 8
#define ROT 3
#define RETRY_SLEEP 1
#define HOLD 32
#define WMOD ""
#define ZERO_AUX 17
#define OUT_AUX 17
constexpr int SLOT = 8192;
constexpr int L_SYNC = 2 * RING * SLOT;
constexpr int L_STAGE = L_SYNC + 256;
constexpr int LDS_BYTES = L_STAGE + 8192;
static_assert(RING >= 8 && RING <= 9 && LDS_BYTES <= 160 * 1024, "");

#define DLT 40000000ull
constexpr unsigned long long DEADLINE_TICKS = DLT;

struct Params {
    const unsigned char* x;
    const unsigned char* w;
    const float* sx;
    const float* sw;
    float* out;
    unsigned char* myws;
    int rank;
    int pad;
    unsigned char* peer[NR];
};

#define STAMP(k) do {} while (0)
#define STAMP_OUT() do {} while (0)

#define WAIT_VM(n) asm volatile("s_waitcnt vmcnt(%0)" :: "i"(n) : "memory")

template <int AUX>
__device__ __forceinline__ void store16(v4u v, __amdgpu_buffer_rsrc_t r, int voff, int soff) {
    __builtin_amdgcn_raw_buffer_store_b128(v, r, voff, soff, AUX);
    asm volatile("s_nop 1" :: "v"(v) : "memory");
}

__device__ __forceinline__ unsigned rfl(unsigned v) { return __builtin_amdgcn_readfirstlane(v); }

__global__ void __launch_bounds__(NTHREADS, 2) a2a_gemm_kernel(Params p) {
    extern __shared__ __attribute__((aligned(16))) unsigned char smem[];
    LAS unsigned char* L8 = (LAS unsigned char*)smem;
    LAS unsigned* sync = (LAS unsigned*)(L8 + L_SYNC);
    STAMP(0);
    asm volatile("s_memrealtime s[96:97]\n\ts_load_dword s98, %0, 0x0\n\ts_load_dword s99, %0, 0x40"
                 :: "s"(__builtin_amdgcn_kernarg_segment_ptr()) : "s96", "s97", "s98", "s99");
#define ENTRY_TIME(t_) unsigned long long t_; asm volatile("s_waitcnt lgkmcnt(0)\n\ts_mov_b64 %0, s[96:97]" : "=s"(t_))
    const int tid = threadIdx.x;
    const int lane = tid & 63;
    const int wave = rfl(tid >> 6);
    const int b = blockIdx.x;

    const int xg = b & 7, idx = b >> 3;
    const int mb = idx & 7;
#define TILE_COL(v_) ((((v_) - (int)((size_t)p.w >> 7)) & 31) * 128)
    const int cls = idx >> 3;
    const bool slowband = !(xg & 1) || xg == 7;

    if (wave == 0) {
        __builtin_amdgcn_s_setprio(3);
        if (!(slowband && cls == 3)) {
            const int rg = (b >> 1) & 15, i0 = (b & 1) * 4;
            int rd = b >> 5;
            const int rank = p.rank;
            unsigned char* pe[4];
#pragma unroll
            for (int i = 0; i < 4; ++i) pe[i] = p.peer[i0 + i];
            int xv0 = (lane >> 2) * SH + (((lane & 3) ^ ((lane >> 4) & 3)) * 16);
            int xv1 = xv0 + 16 * SH - 1024;
            int soff = rg * 32 * SH + (rd ^ ((rg >> 1) & ROT)) * 64;
            if (b >= 64) asm volatile("s_barrier" ::: "memory");
            asm volatile("" : "+v"(xv0), "+v"(xv1), "+s"(soff));
            asm volatile("" :: "s"(p.x), "s"(p.myws), "s"(rank), "s"(pe[0]), "s"(pe[1]), "s"(pe[2]), "s"(pe[3]));
            GAS unsigned* ctr = (GAS unsigned*)(p.myws + WS_CTR) + b * 32 + wave;
            v4i xr;
            xr.x = (int)(size_t)p.x; xr.y = (int)((size_t)p.x >> 32) & 0xffff; xr.z = 4096 * SH; xr.w = 0x00020000;
            const int lst = (int)(size_t)L8 + L_STAGE;
#define SLOADS() _Pragma("unroll") for (int i = 0; i < 4; ++i)     \
                asm volatile("s_mov_b32 m0, %4\n\ts_nop 0\n\t" \
                             "buffer_load_dwordx4 %0, %2, %3 offen lds\n\t" \
                             "buffer_load_dwordx4 %1, %2, %3 offen offset:1024 lds" \
                             :: "v"(xv0), "v"(xv1), "s"(xr), "s"(soff + (((rank + 1 + i0 + i) & 7) << 18)), "s"(lst + i * 2048) : "memory")
            SLOADS();
            unsigned callv;
            asm volatile("global_load_dword %0, %1, %2 sc1" : "=&v"(callv) : "v"(0), "s"(ctr) : "memory");
            if (b < 64 && b >= HOLD) asm volatile("s_barrier" ::: "memory");
            STAMP(1);
            const int rv = L_STAGE + (lane & 31) * 64 + ((((lane >> 5) * 2) ^ ((lane >> 2) & 3)) * 16);
            unsigned callr;
            asm volatile("s_waitcnt vmcnt(0)\n\tv_readfirstlane_b32 %0, %1" : "=s"(callr) : "v"(callv) : "memory");
            const unsigned call = callr + 1u;
            STAMP(2);
            const int bo = (int)WS_A + (int)(call % 3u + 1u) * A_PAR;
            const int fvo = lane < NREP ? lane * FSTRIDE : (int)WS_END;
            ENTRY_TIME(tentry);
            const unsigned tstart = (unsigned)tentry;
            for (;;) {
                v4u d[4][2];
#pragma unroll
                for (int i = 0; i < 4; ++i) {
                    d[i][0] = *(LAS v4u*)(L8 + i * 2048 + rv);
                    d[i][1] = *(LAS v4u*)(L8 + i * 2048 + (rv ^ 16));
                    d[i][0].x = max(~d[i][0].x, 1u);
                    d[i][1].x = max(~d[i][1].x, 1u);
                }
                const int dso = bo + (rg * NREM + rd * 7 + 6 - i0) * FRAG;
                const int fso = bo + A_FLAG + rg * NREP * FSTRIDE + rd * 8 + 6 - i0;
                const unsigned due = SLOT_T0 + ((unsigned)(rd * 16 + rg) * SLOT_Q >> 4);
                while ((unsigned)__builtin_amdgcn_s_memrealtime() - tstart < due) __builtin_amdgcn_s_sleep(1);
                STAMP(3);
#pragma unroll
                for (int i = 0; i < 4; ++i) {
                    const auto dr = __builtin_amdgcn_make_buffer_rsrc((void*)pe[i], 0, (i == 3 && i0) ? 0 : (int)WS_END, 0x00020000);
                    store16<SEND_AUX>(d[i][0], dr, lane * 16, dso - i * FRAG);
                    store16<SEND_AUX>(d[i][1], dr, lane * 16 + 1024, dso - i * FRAG);
                    __builtin_amdgcn_raw_buffer_store_b8((unsigned char)1, dr, fvo, fso - i, SEND_AUX);
                }
                if (rd * 32 < HOLD) asm volatile("s_barrier" ::: "memory");
                if (!slowband || rd >= 2) break;
                rd += 6;
                soff ^= 6 * 64;
                __builtin_amdgcn_s_sleep(64);
                SLOADS();
                WAIT_VM(0);
            }
            STAMP(4);
            if (lane == 0) __hip_atomic_store(ctr, call, __ATOMIC_RELAXED, __HIP_MEMORY_SCOPE_AGENT);
        }
        STAMP_OUT();
    } else if (wave >= 4) {
        __builtin_amdgcn_s_setprio(PRIO_COMP);
        const int rank = p.rank;
        const int cw = wave - 4;
        const int mh = cw & 1, kp = cw >> 1;
        const int rg = mb * 2 + mh;
        const int lb = (int)(size_t)L8;
        const int stg = lb + (kp * RING + RING - 2 + mh) * SLOT;
        {
            int xv0 = (lane >> 3) * SH + (((lane & 7) ^ ((lane >> 4) & 3)) * 16);
            int xv1 = (xv0 ^ 64) + 8 * SH - 1024;
            int xv2 = xv0 + 0x1800, xv3 = xv1 + 0x1800;
            asm volatile("" : "+v"(xv0), "+v"(xv1), "+v"(xv2), "+v"(xv3));
            v4i xr;
            xr.x = (int)(size_t)p.x; xr.y = (int)((size_t)p.x >> 32) & 0xffff; xr.z = 4096 * SH; xr.w = 0x00020000;
            const int xsoff = (rank * SH + rg * 32) * SH + kp * 128;
#define XLD(so_) \
        "buffer_load_dwordx4 %0, %4, " so_ " offen lds\n\t" \
        "buffer_load_dwordx4 %1, %4, " so_ " offen offset:1024 lds\n\t" \
        "buffer_load_dwordx4 %2, %4, " so_ " offen offset:2048 lds\n\t" \
        "buffer_load_dwordx4 %3, %4, " so_ " offen offset:3072 lds\n\t"
            asm volatile("s_mov_b32 m0, %7\n\t"
                         "s_barrier\n\t"
                         XLD("%5")
                         "s_add_u32 m0, m0, 4096\n\t"
                         "s_nop 0\n\t"
                         XLD("%6")
                         :: "v"(xv0), "v"(xv1), "v"(xv2), "v"(xv3), "s"(xr), "s"(xsoff), "s"(xsoff + 256), "s"(stg) : "memory", "scc");
        }
        int lane2 = lane;
        asm volatile("" : "+v"(lane2));
#define lane lane2
        const unsigned char* ab = p.myws + WS_A + rg * NREM * FRAG;
        v4i ar;
        ar.x = (int)(size_t)ab; ar.y = (int)((size_t)ab >> 32) & 0xffff; ar.z = 3 * A_PAR + NREM * FRAG; ar.w = 0x00020000;
        const int avoff = lane * 16;
        const int rv = stg + (lane & 31) * 128 + ((((lane >> 5) * 2) ^ ((lane >> 1) & 7)) * 16);
        const int g = lane >> 4, q = (lane & 15) >> 1;
        const int rdl = lb + kp * RING * SLOT + ((g >> 1) * 32 + q) * 128 + (g & 1) * 16 + (lane & 1) * 8;
        const int sw = (q >> 1) ^ (2 * kp);
        const int rd0 = rdl + ((0 ^ sw) * 32), rd1 = rdl + ((1 ^ sw) * 32);
        const int rd2 = rdl + ((2 ^ sw) * 32), rd3 = rdl + ((3 ^ sw) * 32);
        const int a_syn = lb + L_SYNC + 8 * cw, a_cons = lb + L_SYNC + 32 + 4 * cw;
        int aoff = kp * FRAG;
        static_assert((32 - RING) % RING + 4 <= RING, "the four staging slots are adjacent");
        const int a_scale = lb + L_SYNC + 64;
        const int s0 = ((32 - RING) % RING + 2 * mh) * SLOT;
        int e_w[4], e_r0[2], e_r1[2];
#pragma unroll
        for (int gq = 0; gq < 4; ++gq) e_w[gq] = lb + kp * RING * SLOT + s0 + (lane & 31) * 256 + ((((lane >> 5) ^ (lane & 7)) * 16) ^ (gq * 32));
#pragma unroll
        for (int i = 0; i < 2; ++i) {
            const int ra = (lane >> 4) * 256 + ((((lane & 15) ^ (lane >> 4)) * 16) ^ (i * 64));
            e_r0[i] = lb + kp * RING * SLOT + s0 + ra;
            e_r1[i] = lb + (kp ^ 1) * RING * SLOT + s0 + SLOT + ra;
        }
        LAS unsigned* e_sib = sync + 8 + (cw ^ 1);
        LAS unsigned* e_par = sync + 12 + (cw ^ 2);
        int e_ov = (lane >> 4) * GN * 4 + (lane & 15) * 16;
        int e_os = (rg * 32 * GN + TILE_COL(xg * 4 + cls) + kp * 64) * 4;
        float* e_out = p.out;
        asm volatile("" : "+v"(e_w[0]), "+v"(e_w[1]), "+v"(e_w[2]), "+v"(e_w[3]), "+v"(e_r0[0]), "+v"(e_r0[1]), "+v"(e_r1[0]), "+v"(e_r1[1]),
                          "+v"(e_sib), "+v"(e_par), "+v"(e_ov));
        asm volatile("" : "+s"(e_os));
        asm volatile("" : "+s"(e_out));
#undef lane
        v16f acc0, acc1, acc2, acc3;
        v16i clob0, clob1, clob2, clob3, clob4, clob5;
        int va0, va1, va2, va3, vt2, vt3, s_aend;
        v2i vt01;
        int s_valid, s_need, s_wslot, s_wrdy, s_ardy, s_t0, s_t1, s_so;
        unsigned long long s_ret, s_now;
        ENTRY_TIME(tentry);
        const unsigned tlo = (unsigned)tentry;
#define DBG_ASM(s_)
#define DBG_SIDE(s_)
#define DBG_ASM0
#define DBG_OPS
#define AREG(s_) "v[160+8*" #s_ ":167+8*" #s_ "]"
#define ALO(s_) "v[160+8*" #s_ ":163+8*" #s_ "]"
#define AHI(s_) "v[164+8*" #s_ ":167+8*" #s_ "]"
#define LDLO(p_) "buffer_load_dwordx4 " ALO(p_) ", %[avoff], %[ar], %[so] offen sc1\n\t"
#define LDHI(p_) "buffer_load_dwordx4 " AHI(p_) ", %[avoff], %[ar], %[so] offen offset:1024 sc1\n\t"
#define RDB(base_, va_) \
        "ds_read_b64_tr_b8 v[" #base_ ":" #base_ "+1], " va_ "\n\t" \
        "ds_read_b64_tr_b8 v[" #base_ "+2:" #base_ "+3], " va_ " offset:1024\n\t" \
        "ds_read_b64_tr_b8 v[" #base_ "+4:" #base_ "+5], " va_ " offset:2048\n\t" \
        "ds_read_b64_tr_b8 v[" #base_ "+6:" #base_ "+7], " va_ " offset:3072\n\t"
#define HERE "s_cmp_lt_u32 %[aoff], %[ardy]\n\t"
#define PICK \
        "v_readfirstlane_b32 %[t1], v224\n\t" \
        "v_readfirstlane_b32 %[wrdy], v225\n\t" \
        "s_max_u32 %[ardy], %[ardy], %[t1]\n\t" \
        HERE
#define AHITX
#define TAKE \
        "s_cselect_b32 %[so], %[aoff], %[oob]\n\t" \
        "s_cselect_b32 %[t0], 0x1000, 0\n\t" \
        "s_addc_u32 %[valid], %[valid], %[valid]\n\t" \
        "s_add_u32 %[aoff], %[aoff], %[t0]\n\t" AHITX
#define WHOLE "s_cmp_eq_u64 vcc, 0\n\t"
#define CHKBR(s_) "s_cbranch_vccnz .Lretry" #s_ "_%=\n"
#define SPEC(s_)
#define MF(x_) x_
#define POS(s_, p_) \
        ".Lpos" #s_ "_%=:\n\t" \
        "s_waitcnt vmcnt(12)\n" \
        ".Lchk" #s_ "_%=:\n\t" \
        "v_min_u32 %[vt3], v[160+8*" #s_ "], v[164+8*" #s_ "]\n\t" \
        "v_cmp_eq_u32 vcc, 0, %[vt3]\n\t" \
        "v_not_b32 v[160+8*" #s_ "], v[160+8*" #s_ "]\n\t" \
        "v_not_b32 v[164+8*" #s_ "], v[164+8*" #s_ "]\n\t" \
        WHOLE \
        "s_cselect_b32 %[t0], %[wrdy], 0\n\t" \
        "s_cmp_lt_u32 %[t0], %[need]\n\t" \
        "s_waitcnt lgkmcnt(10)\n\t" \
        "s_cbranch_scc1 .Lslow" #s_ "_%=\n" \
        ".Lcomp" #s_ "_%=:\n\t" \
        MF("v_mfma_f32_32x32x64_f8f6f4 v[64:79], v[128:135], " AREG(s_) ", v[64:79] cbsz:1 blgp:1\n\t") \
        LDLO(p_) \
        RDB(128, "%[va0]") \
        "ds_read_b64 v[224:225], %[asyn]\n\t" \
        "s_add_u32 %[wslot], %[wslot], %[slotb]\n\t" \
        "s_cmp_eq_u32 %[wslot], %[ringb]\n\t" \
        "s_cselect_b32 %[wslot], 0, %[wslot]\n\t" \
        "v_add_u32 %[va0], %[wslot], %[rd0]\n\t" \
        "s_waitcnt lgkmcnt(10)\n\t" \
        MF("v_mfma_f32_32x32x64_f8f6f4 v[80:95], v[136:143], " AREG(s_) ", v[80:95] cbsz:1 blgp:1\n\t") \
        LDHI(p_) \
        RDB(136, "%[va1]") \
        "ds_write_b32 %[acons], %[vt2]\n\t" \
        "v_add_u32 %[va1], %[wslot], %[rd1]\n\t" \
        "v_add_u32 %[vt2], 1, %[vt2]\n\t" \
        "s_add_u32 %[need], %[need], 1\n\t" \
        DBG_ASM(s_) \
        "s_waitcnt lgkmcnt(5)\n\t" \
        MF("v_mfma_f32_32x32x64_f8f6f4 v[96:111], v[144:151], " AREG(s_) ", v[96:111] cbsz:1 blgp:1\n\t") \
        RDB(144, "%[va2]") \
        "v_add_u32 %[va2], %[wslot], %[rd2]\n\t" \
        PICK TAKE \
        MF("v_mfma_f32_32x32x64_f8f6f4 v[112:127], v[152:159], " AREG(s_) ", v[112:127] cbsz:1 blgp:1\n\t") \
        RDB(152, "%[va3]") \
        "v_add_u32 %[va3], %[wslot], %[rd3]\n"
#define SIDE(s_, p_, n_) \
        ".Lslow" #s_ "_%=:\n\t" \
        "s_bitcmp1_b32 %[valid], 7\n\t" \
        "s_cbranch_scc1 .Lfull" #s_ "_%=\n\t" \
        "s_cmp_eq_u32 %[need], 34\n\t" \
        "s_cbranch_scc1 .Ldone_%=\n\t" \
        "ds_write_b32 %[acons], %[vt2]\n" \
        ".Lempty" #s_ "_%=:\n\t" \
        LDLO(p_) LDHI(p_) \
        HERE \
        "s_cbranch_scc1 .Ltake" #s_ "_%=\n\t" \
        "s_cmp_lt_u32 %[aoff], %[aend]\n\t" \
        "s_cbranch_scc0 .Ltake" #s_ "_%=\n" \
        ".Lidle" #s_ "_%=:\n\t" \
        "ds_read_b64 v[224:225], %[asyn]\n\t" \
        "s_and_b32 %[t0], %[valid], 0x7f\n\t" \
        "s_waitcnt lgkmcnt(0)\n\t" \
        PICK \
        "s_cbranch_scc1 .Ltake" #s_ "_%=\n\t" \
        "s_cmp_eq_u32 %[t0], 0\n\t"       \
        "s_cbranch_scc0 .Ltake" #s_ "_%=\n\t" \
        SPEC(s_) \
        ".Ltake" #s_ "_%=:\n\t" \
        TAKE \
        "s_bitcmp1_b32 %[valid], 7\n\t" \
        "s_cbranch_scc0 .Lempty" #n_ "_%=\n\t" \
        "s_branch .Lpos" #n_ "_%=\n" \
        ".Lfull" #s_ "_%=:\n\t" \
        CHKBR(s_) \
        ".Lwspin" #s_ "_%=:\n\t" \
        "ds_read_b64 v[224:225], %[asyn]\n\t" \
        "s_waitcnt lgkmcnt(0)\n\t" \
        "v_readfirstlane_b32 %[wrdy], v225\n\t" \
        "s_cmp_lt_u32 %[wrdy], %[need]\n\t" \
        "s_cbranch_scc1 .Lwspin" #s_ "_%=\n\t" \
        "s_branch .Lcomp" #s_ "_%=\n" \
        ".Lretry" #s_ "_%=:\n\t" \
        "s_getpc_b64 %[ret]\n\t" \
        "s_branch .Lagain_%=\n\t" \
        "s_cbranch_scc1 .Lwspin" #s_ "_%=\n\t" \
        "buffer_load_dwordx4 " ALO(s_) ", %[avoff], %[ar], %[t1] offen sc1\n\t" \
        "buffer_load_dwordx4 " AHI(s_) ", %[avoff], %[ar], %[t1] offen offset:1024 sc1\n\t" \
        "s_waitcnt vmcnt(0)\n\t" \
        "s_branch .Lchk" #s_ "_%=\n" \
        DBG_SIDE(s_)
#define XRD(d_, off_) \
        "ds_read_b128 v[" #d_ ":" #d_ "+3], %[rv] offset:" #off_ "\n\t" \
        "ds_read_b128 v[" #d_ "+4:" #d_ "+7], %[va1] offset:" #off_ "\n\t" \
        "ds_read_b128 v[" #d_ "+8:" #d_ "+11], %[va2] offset:" #off_ "\n\t" \
        "ds_read_b128 v[" #d_ "+12:" #d_ "+15], %[va3] offset:" #off_ "\n\t"
#define XINV(r_) "v_not_b32 v" #r_ ", v" #r_ "\n\tv_max_u32 v" #r_ ", 1, v" #r_ "\n\t"
#define Z8(b_) \
        "v_mov_b64 v[" #b_ ":" #b_ "+1], 0\n\tv_mov_b64 v[" #b_ "+2:" #b_ "+3], 0\n\t" \
        "v_mov_b64 v[" #b_ "+4:" #b_ "+5], 0\n\tv_mov_b64 v[" #b_ "+6:" #b_ "+7], 0\n\t"
        asm volatile(
            "v_xor_b32 %[va1], 16, %[rv]\n\t"
            "v_xor_b32 %[va2], 64, %[rv]\n\t"
            "v_xor_b32 %[va3], 0x50, %[rv]\n\t"
            Z8(64) Z8(72) Z8(80) Z8(88) Z8(96) Z8(104) Z8(112) Z8(120)
            "s_mov_b32 %[valid], 0xf0\n\t"
            "s_mov_b32 %[need], 2\n\t"
            "s_mov_b32 %[wslot], %[slotb]\n\t"
            "s_mov_b32 %[so], %[oob]\n\t"
            "v_mov_b32 %[vt2], 1\n\t"
            "v_mov_b32 v192, 0\n\t"
            "s_waitcnt vmcnt(4)\n\t"
            XRD(160, 0)
            "s_waitcnt vmcnt(0)\n\t"
            XRD(176, 4096)
            ".Lfirst_%=:\n\t"
            "ds_read_b64 v[224:225], %[asyn]\n\t"
            "s_waitcnt lgkmcnt(0)\n\t"
            "v_readfirstlane_b32 %[ardy], v224\n\t"
            "v_readfirstlane_b32 %[wrdy], v225\n\t"
            "s_min_u32 %[t0], %[ardy], %[wrdy]\n\t"
            "s_cmp_eq_u32 %[t0], 0\n\t"
            "s_cbranch_scc1 .Lfirst_%=\n\t"
            "s_and_b32 %[t0], %[ardy], 0x600000\n\t"
            "s_or_b32 %[aoff], %[aoff], %[t0]\n\t"
            "s_or_b32 %[aend], %[t0], %[aendc]\n\t"
            DBG_ASM0
            XINV(160) XINV(164) XINV(168) XINV(172) XINV(176) XINV(180) XINV(184) XINV(188)
            RDB(128, "%[rd0]") RDB(136, "%[rd1]") RDB(144, "%[rd2]") RDB(152, "%[rd3]")
            "ds_write_b32 %[acons], %[vt2]\n\t"
            "v_add_u32 %[va0], %[wslot], %[rd0]\n\t"
            "v_add_u32 %[va1], %[wslot], %[rd1]\n\t"
            "v_add_u32 %[va2], %[wslot], %[rd2]\n\t"
            "v_add_u32 %[va3], %[wslot], %[rd3]\n\t"
            "s_waitcnt lgkmcnt(1)\n"
            POS(0, 7) POS(1, 0) POS(2, 1) POS(3, 2) POS(4, 3) POS(5, 4) POS(6, 5) POS(7, 6)
            "s_branch .Lpos0_%=\n"
            SIDE(0, 7, 1) SIDE(1, 0, 2) SIDE(2, 1, 3) SIDE(3, 2, 4) SIDE(4, 3, 5) SIDE(5, 4, 6) SIDE(6, 5, 7) SIDE(7, 6, 0)
            ".Lagain_%=:\n\t"
            "ds_write_b32 %[acons], %[vt2]\n\t"
            "s_sleep %[rslp]\n\t"
            "s_memrealtime %[now]\n\t"
            "s_add_u32 s40, s40, 4\n\t"
            "s_addc_u32 s41, s41, 0\n\t"
            "s_waitcnt lgkmcnt(0)\n\t"
            "s_lshl_b32 %[t1], %[need], 12\n\t"
            "s_add_u32 %[t1], %[t1], %[abias]\n\t"
            "s_mul_i32 s43, %[t1], 37\n\t"
            "s_lshr_b32 s43, s43, 19\n\t"
            "s_add_u32 s43, s43, 1\n\t"
            "s_mul_i32 s43, s43, 0x3800\n\t"
            "s_and_b32 %[t0], %[ardy], 0x600000\n\t"
            "s_or_b32 %[t1], %[t1], %[t0]\n\t"
            "s_or_b32 s43, s43, %[t0]\n\t"
            "s_max_u32 %[ardy], %[ardy], s43\n\t"
            "s_sub_u32 %[t0], s42, %[tlo]\n\t"
            "s_cmp_gt_u32 %[t0], %[dl]\n\t"
            "s_cbranch_scc0 .Lback_%=\n\t"
            "v_mov_b32 %[vt3], 0x7fc00000\n\t"
            "ds_write_b32 %[ascale], %[vt3]\n"
            ".Lback_%=:\n\t"
            "s_setpc_b64 %[ret]\n"
            ".Ldone_%=:\n\t"
            "s_waitcnt vmcnt(0) lgkmcnt(0)\n\t"
            "s_nop 15"
            : "=&{v[64:79]}"(acc0), "=&{v[80:95]}"(acc1), "=&{v[96:111]}"(acc2), "=&{v[112:127]}"(acc3),
              "=&{v[128:143]}"(clob0), "=&{v[144:159]}"(clob1), "=&{v[160:175]}"(clob2), "=&{v[176:191]}"(clob3),
              "=&{v[192:207]}"(clob4), "=&{v[208:223]}"(clob5),
              [va0] "=&v"(va0), [va1] "=&v"(va1), [va2] "=&v"(va2), [va3] "=&v"(va3),
              "=&{v[224:225]}"(vt01), [vt2] "=&v"(vt2), [vt3] "=&v"(vt3),
              [ret] "=&{s[40:41]}"(s_ret), [now] "=&{s[42:43]}"(s_now),
              [valid] "=&s"(s_valid), [need] "=&s"(s_need), [wslot] "=&s"(s_wslot),
              [wrdy] "=&s"(s_wrdy), [ardy] "=&s"(s_ardy), [t0] "=&s"(s_t0), [t1] "=&s"(s_t1), [so] "=&s"(s_so),
              [aend] "=&s"(s_aend), [aoff] "+s"(aoff) DBG_OPS
            : [ar] "s"(ar), [avoff] "v"(avoff),
              [rd0] "v"(rd0), [rd1] "v"(rd1), [rd2] "v"(rd2), [rd3] "v"(rd3),
              [rv] "v"(rv), [oob] "s"(A_OOB), [asyn] "v"(a_syn), [acons] "v"(a_cons),
              [slotb] "i"(SLOT), [ringb] "i"(RING * SLOT), [aendc] "i"(NREM * FRAG),
              [abias] "s"(kp * FRAG - 6 * 4096), [rslp] "i"(RETRY_SLEEP), [tlo] "s"(tlo), [dl] "i"((unsigned)DEADLINE_TICKS), [ascale] "v"(a_scale)
            : "memory", "scc", "vcc");
        asm volatile("" : "+v"(e_w[0]), "+v"(e_w[1]), "+v"(e_w[2]), "+v"(e_w[3]), "+v"(e_r0[0]), "+v"(e_r0[1]), "+v"(e_r1[0]), "+v"(e_r1[1]));
        STAMP(5);
        {
            while (rfl(*(volatile LAS unsigned*)(e_sib)) < (unsigned)(36 - RING)) {}
            asm volatile("" ::: "memory");
#pragma unroll
            for (int gq = 0; gq < 4; ++gq) {
                *(LAS v4f*)(size_t)(e_w[gq] + SLOT) = v4f{acc2[4 * gq], acc2[4 * gq + 1], acc2[4 * gq + 2], acc2[4 * gq + 3]};
                *(LAS v4f*)(size_t)(e_w[gq] + SLOT + 128) = v4f{acc3[4 * gq], acc3[4 * gq + 1], acc3[4 * gq + 2], acc3[4 * gq + 3]};
            }
            asm volatile("" ::: "memory");
            *(volatile LAS unsigned*)(size_t)(a_cons + 16) = 1u;
            asm volatile("" ::: "memory");
#pragma unroll
            for (int gq = 0; gq < 4; ++gq) {
                *(LAS v4f*)(size_t)(e_w[gq]) = v4f{acc0[4 * gq], acc0[4 * gq + 1], acc0[4 * gq + 2], acc0[4 * gq + 3]};
                *(LAS v4f*)(size_t)(e_w[gq] + 128) = v4f{acc1[4 * gq], acc1[4 * gq + 1], acc1[4 * gq + 2], acc1[4 * gq + 3]};
            }
            v4f u0[8], u1[8];
#pragma unroll
            for (int i = 0; i < 8; ++i) u0[i] = *(LAS v4f*)(size_t)(e_r0[i & 1] + i * 1024);
            while (rfl(*(volatile LAS unsigned*)(e_par)) == 0u) {}
            asm volatile("" ::: "memory");
            const float sc = *(volatile LAS float*)(size_t)(a_scale);
#pragma unroll
            for (int i = 0; i < 8; ++i) u1[i] = *(LAS v4f*)(size_t)(e_r1[i & 1] + i * 1024);
            const auto orr = __builtin_amdgcn_make_buffer_rsrc((void*)e_out, 0, SH * GN * 4, 0x00020000);
#pragma unroll
            for (int i = 0; i < 8; ++i) {
                const v4f y = __builtin_elementwise_maximum((u0[i] + u1[i]) * sc, v4f{0.f, 0.f, 0.f, 0.f});
                store16<OUT_AUX>(__builtin_bit_cast(v4u, y), orr, e_ov, e_os + i * 4 * GN * 4);
            }
        }
        STAMP(6);
        STAMP_OUT();
    } else if (wave < 3) {
        __builtin_amdgcn_s_setprio(PRIO_LOAD);
        const int rank = p.rank;
        const int kp = wave - 1;
        v4i rs;
        rs.x = (int)(size_t)p.w; rs.y = (int)((size_t)p.w >> 32) & 0xffff; rs.z = GK * GN; rs.w = 0x00020000;
        const int v0 = (lane >> 3) * GN + TILE_COL(xg * 4 + cls) + (((lane & 7) ^ (((lane >> 4) & 3) << 1)) * 16);
        const int v1 = v0 + 8 * GN - 1024, v2 = v0 + 16 * GN - 2048, v3 = v0 + 24 * GN - 3072;
        int tab;
        {
            const int n = lane & 31;
            const int ri = max(2 * (n - 4) + kp, 0), jj = (ri * 37) >> 8;
            const int loc = (n - 4) >> 31;
            int src = (1 + ri - jj * 7) & ~loc;
            const int col = (((n >> 1) * 4 + kp * 2 + (n & 1)) & loc) | ((jj ^ (mb & ROT)) & ~loc);
            const int need = max(n - RING + 1, n >= RING - 2 ? 1 : 0);
            const int slot = n - ((n * ((256 + RING - 1) / RING)) >> 8) * RING;
            int rest = (col << 18) + slot * SLOT + (lane < 32 ? need : 63);
            asm volatile("" : "+v"(src), "+v"(rest));
            asm volatile("" :: "s"(p.w), "s"(rank));
            tab = (((rank + src) & 7) << 21) + rest;
        }
        const int lbase = (int)(size_t)L8 + kp * RING * SLOT;
        const int a_wrdy = (int)(size_t)L8 + L_SYNC + (2 * kp + (lane & 1)) * 8 + 4, a_cons = (int)(size_t)L8 + L_SYNC + 32 + 8 * kp;
        int s_n, s_e, s_so, s_so2, s_need, s_t, vpub;
        v2i vc;
#define WLD(v_, so_, off_) "buffer_load_dwordx4 %[" #v_ "], %[rs], %[" #so_ "] offen offset:" #off_ WMOD " lds\n\t"
#define FILL \
        WLD(v0, so, 0) \
        "ds_read_b64 %[vc], %[acons]\n\t" \
        "s_add_u32 %[n], %[n], 1\n\t" \
        WLD(v1, so, 1024) \
        "v_readlane_b32 %[e], %[tab], %[n]\n\t" \
        WLD(v2, so, 2048) \
        "v_add_u32 %[vpub], 1, %[vpub]\n\t" \
        WLD(v3, so, 3072) \
        "s_add_u32 m0, m0, 4096\n\t" \
        "s_and_b32 %[so], %[e], 0xfffc0000\n\t" \
        WLD(v0, so2, 0) \
        "s_and_b32 %[need], %[e], 63\n\t" \
        "s_and_b32 %[e], %[e], 0x1e000\n\t" \
        WLD(v1, so2, 1024) \
        "s_waitcnt lgkmcnt(0)\n\t" \
        "v_min_u32 v20, v20, v21\n\t" \
        WLD(v2, so2, 2048) \
        "v_readfirstlane_b32 %[t], v20\n\t" \
        WLD(v3, so2, 3072) \
        "s_add_u32 m0, %[e], %[lbase]\n\t" \
        "s_add_u32 %[so2], %[so], 0x20000\n\t"
#define PUB(cnt_) \
        "s_waitcnt vmcnt(" #cnt_ ")\n\t" \
        "ds_write_b32 %[awrdy], %[vpub]\n\t"
#define PUBN(cnt_) "v_add_u32 %[vpub], 1, %[vpub]\n\t" PUB(cnt_)
#define LDBG_IN
#define LDBG_OUT
#define LDBG_OPS
        asm volatile(
            "s_mov_b32 %[n], 0\n\t"
            "v_mov_b32 %[vpub], %[pub0]\n\t"
            "v_readlane_b32 %[so], %[tab], 0\n\t"
            "s_mov_b32 m0, %[lbase]\n\t"
            "s_add_u32 %[so2], %[so], 0x20000\n\t"
            "s_nop 2\n\t"
            "s_barrier\n"
            ".Lpro_%=:\n\t"
            FILL
            "s_cmp_lt_u32 %[n], %[nf]\n\t"
            "s_cbranch_scc1 .Lpro_%=\n"
            ".Lmain_%=:\n\t"
            PUB(%[cnt])
            ".Lgo_%=:\n\t"
            LDBG_IN
            FILL
            "s_cmp_lt_u32 %[t], %[need]\n\t"
            "s_cbranch_scc0 .Lmain_%=\n\t"
            PUB(%[cnt])
            "s_cmp_eq_u32 %[n], 32\n\t"
            "s_cbranch_scc1 .Lend_%=\n"
            ".Lspin_%=:\n\t"
            "ds_read_b64 %[vc], %[acons]\n\t"
            "s_waitcnt lgkmcnt(0)\n\t"
            "v_min_u32 v20, v20, v21\n\t"
            "s_nop 0\n\t"
            "v_readfirstlane_b32 %[t], v20\n\t"
            "s_cmp_lt_u32 %[t], %[need]\n\t"
            "s_cbranch_scc0 .Lgo_%=\n\t"
            "s_sleep 1\n\t"
            "s_branch .Lspin_%=\n"
            LDBG_OUT
            ".Lend_%=:\n\t"
            PUBN(16)
            PUBN(8)
            "v_add_u32 %[vpub], 32, %[vpub]\n\t"
            PUB(0)
            "s_waitcnt lgkmcnt(0)"
            : [n] "=&s"(s_n), [e] "=&s"(s_e), [so] "=&s"(s_so), [so2] "=&s"(s_so2), [need] "=&s"(s_need), [t] "=&s"(s_t),
              [vpub] "=&v"(vpub), [vc] "=&{v[20:21]}"(vc) LDBG_OPS
            : [tab] "v"(tab), [rs] "s"(rs), [lbase] "s"(lbase), [v0] "v"(v0), [v1] "v"(v1), [v2] "v"(v2), [v3] "v"(v3),
              [awrdy] "v"(a_wrdy), [acons] "v"(a_cons), [nf] "i"(NFLY), [cnt] "i"(8 * (NFLY - 1)), [pub0] "i"(1 - NFLY)
            : "memory", "scc");
        STAMP(7);
        STAMP_OUT();
    } else {
        __builtin_amdgcn_s_setprio(3);
        sync[lane] = 0u;
        GAS unsigned* ctr = (GAS unsigned*)(p.myws + WS_CTR) + b * 32 + wave;
        const unsigned callv = __hip_atomic_load(ctr, __ATOMIC_RELAXED, __HIP_MEMORY_SCOPE_AGENT);
        const float sx = *(const GAS float*)(p.sx + (lane >> 6)), sw = *(const GAS float*)(p.sw + (lane >> 6));
        asm volatile("s_waitcnt lgkmcnt(0)\n\ts_barrier" ::: "memory");
        *(volatile LAS float*)(sync + 16) = sx * sw;
        const unsigned call = rfl(callv) + 1u;
        __hip_atomic_store(ctr, call, __ATOMIC_RELAXED, __HIP_MEMORY_SCOPE_AGENT);
        GAS unsigned* tmo = (GAS unsigned*)(p.myws + WS_TMO);
        ENTRY_TIME(t0);
        const unsigned tag = (call % 3u + 1u) << 21;
        volatile LAS unsigned* ardy = sync + (lane & 3) * 2;
        *ardy = tag;
        {
            if ((!(xg & 1) || xg == 7) && !(slowband && cls == 3)) {
                v4i wr;
                wr.x = (int)(size_t)p.w; wr.y = (int)((size_t)p.w >> 32) & 0xffff; wr.z = GK * GN; wr.w = 0x00020000;
                const int wo = lane * GN + TILE_COL(xg * 4 + 3);
                asm volatile("s_nop 4\n\tbuffer_load_dword v200, %0, %1, %2 offen\n\tbuffer_load_dword v201, %0, %1, %3 offen"
                             :: "v"(wo), "s"(wr), "s"(idx * 128 * GN), "s"((idx * 128 + 64) * GN) : "memory", "v200", "v201");
                if (slowband && cls == 1)
                    asm volatile("buffer_load_dword v200, %0, %1, %2 offen\n\tbuffer_load_dword v201, %0, %1, %3 offen"
                                 :: "v"(wo), "s"(wr), "s"((idx * 128 + 2048) * GN), "s"((idx * 128 + 2112) * GN) : "memory", "v200", "v201");
            }
        }
        const unsigned char* fl = p.myws + WS_A + tag + A_FLAG + (mb * 2 * NREP + (xg & (NREP - 1))) * FSTRIDE;
        bool zeroed = slowband && cls == 3;
        const auto zr = __builtin_amdgcn_make_buffer_rsrc((void*)(p.myws + WS_A), 0, 4 * A_PAR, 0x00020000);
        const int zb = (int)((call + 2u) % 3u + 1u) * A_PAR, zo = zb + b * 7168;
#define ZERO_FILL() do { if (!zeroed) { zeroed = true; \
            _Pragma("unroll") for (int u = 0; u < 7; ++u) store16<ZERO_AUX>(v4u{0u, 0u, 0u, 0u}, zr, lane * 16, zo + u * 1024); \
            if (slowband && cls == 2) { _Pragma("unroll") for (int u = 0; u < 7; ++u) store16<ZERO_AUX>(v4u{0u, 0u, 0u, 0u}, zr, lane * 16, zo + 64 * 7168 + u * 1024); } \
            store16<ZERO_AUX>(v4u{0u, 0u, 0u, 0u}, zr, lane < 4 && b < 16 * NREP ? lane * 16 : A_OOB, zb + A_FLAG + b * FSTRIDE); } } while (0)
        if (ZERO_AT == 0) ZERO_FILL();
        unsigned spins = 0;
        int r0 = 0, r1 = 0;
        for (;;) {
            v4l f, h;
            asm volatile("s_load_dwordx8 %0, %2, %3 glc\n\ts_load_dwordx8 %1, %2, %4 glc\n\ts_waitcnt lgkmcnt(0)"
                         : "=&s"(f), "=&s"(h) : "s"(fl), "s"(r0 * 8), "s"(NREP * FSTRIDE + r1 * 8) : "memory");
            constexpr unsigned long long ALL7 = 0x0001010101010101ull;
            const int a0 = f[0] != ALL7 ? 0 : f[1] != ALL7 ? 1 : f[2] != ALL7 ? 2 : f[3] != ALL7 ? 3 : 4;
            const int a1 = h[0] != ALL7 ? 0 : h[1] != ALL7 ? 1 : h[2] != ALL7 ? 2 : h[3] != ALL7 ? 3 : 4;
            if (a0 + a1) {
                r0 += a0; r1 += a1;
                *ardy = (unsigned)(lane & 1 ? r1 : r0) * (7 * FRAG) | tag;
                if (r0 + r1 >= ZERO_AT) ZERO_FILL();
                if (r0 + r1 == 16) break;
            }
            if ((++spins & 255u) == 0u) {
                if (rfl(__hip_atomic_load(tmo, __ATOMIC_RELAXED, __HIP_MEMORY_SCOPE_AGENT)) == call
                    || __builtin_amdgcn_s_memrealtime() - t0 > DEADLINE_TICKS) {
                    __hip_atomic_store(tmo, call, __ATOMIC_RELAXED, __HIP_MEMORY_SCOPE_AGENT);
                    *(volatile LAS float*)(sync + 16) = __builtin_nanf("");
                    *ardy = (unsigned)NREM * FRAG | tag;
                    break;
                }
            }
            if (POLL_SLEEP) __builtin_amdgcn_s_sleep(POLL_SLEEP);
        }
        ZERO_FILL();
        STAMP(7);
        STAMP_OUT();
    }
}

extern "C" void kernel_launch(int rank, int nranks, ncclComm_t comm,
                              hipStream_t stream,
                              void* const* d_in, const int* in_sizes, int n_in,
                              void* d_out, int out_size,
                              const int* in_dtypes, int out_dtype,
                              void* const* d_out_all,
                              void* const* d_ws_all, size_t ws_bytes) {
    if (nranks != NR || (unsigned)rank >= (unsigned)NR || ws_bytes < WS_END || n_in < 4 ||
        in_sizes[0] != 4096 * SH || in_sizes[1] != GK * GN || out_size != SH * GN) {
        fprintf(stderr, "unexpected problem geometry\n");
        return;
    }
    HIP_CHECK(hipFuncSetAttribute((const void*)a2a_gemm_kernel, hipFuncAttributeMaxDynamicSharedMemorySize, LDS_BYTES));
    Params p;
    p.x = (const unsigned char*)d_in[0];
    p.w = (const unsigned char*)d_in[1];
    p.sx = (const float*)d_in[2];
    p.sw = (const float*)d_in[3];
    p.out = (float*)d_out;
    p.myws = (unsigned char*)d_ws_all[rank];
    p.rank = rank;
    p.pad = 0;
    for (int i = 0; i < NR; ++i) p.peer[i] = (unsigned char*)d_ws_all[(rank + 1 + i) & 7];
    a2a_gemm_kernel<<<NWG, NTHREADS, LDS_BYTES, stream>>>(p);
    HIP_CHECK(hipGetLastError());
}
```

```cpp
#include <hip/hip_runtime.h>
#include <rccl/rccl.h>
#include <cstdio>

#define HIP_CHECK(expr) do { hipError_t _e = (expr); if (_e != hipSuccess) { \
    fprintf(stderr, "HIP error %s at %s:%d\n", hipGetErrorString(_e), __FILE__, __LINE__); \
    return; } } while (0)

#define GAS __attribute__((address_space(1)))
#define LAS __attribute__((address_space(3)))
typedef int v2i __attribute__((ext_vector_type(2)));
typedef int v4i __attribute__((ext_vector_type(4)));
typedef int v8i __attribute__((ext_vector_type(8)));
typedef unsigned v2u __attribute__((ext_vector_type(2)));
typedef unsigned v4u __attribute__((ext_vector_type(4)));
typedef float v4f __attribute__((ext_vector_type(4)));
typedef float v16f __attribute__((ext_vector_type(16)));
typedef int v16i __attribute__((ext_vector_type(16)));
typedef unsigned long long v4l __attribute__((ext_vector_type(4)));

constexpr int NR = 8;
constexpr int GN = 4096;
constexpr int GK = 4096;
constexpr int SH = 512;
constexpr int NWG = 256;
constexpr int NTHREADS = 8 * 64;
constexpr int NREM = 56;
constexpr int FRAG = 2048;

constexpr size_t WS_CTR = 0;
constexpr size_t WS_DBG = 64 * 1024;
constexpr size_t WS_A = 0;
constexpr int A_PAR = 1 << 21;
static_assert(16 * NREM * FRAG <= A_PAR, "");
constexpr size_t WS_END = WS_A + 4 * (size_t)A_PAR;

#define RING 9
#define SLOT_Q 76
#define SLOT_T0 30
#define NFLY 4
#define PRIO_COMP 2
#define PRIO_LOAD 1
#define SEND_AUX 17
#define ROT 3
#define RETRY_SLEEP 4
#define HOLD 32
#define WMOD ""
#define ZERO_AUX 17
#define SRCX 1
#define PDIST 3
#define EXEMPT 1
#define OUT_AUX 17
constexpr int SLOT = 8192;
constexpr int L_SYNC = 2 * RING * SLOT;
constexpr int L_STAGE = L_SYNC + 256;
constexpr int LDS_BYTES = L_STAGE + 14336;
static_assert(RING >= 8 && RING <= 9 && LDS_BYTES <= 160 * 1024, "");

#define DLT 40000000ull
constexpr unsigned long long DEADLINE_TICKS = DLT;

struct Params {
    const unsigned char* x;
    const unsigned char* w;
    const float* sx;
    const float* sw;
    float* out;
    unsigned char* myws;
    int rank;
    int pad;
    unsigned char* peer[NR];
};

#define STAMP(k) do {} while (0)
#define STAMP_OUT() do {} while (0)

#define WAIT_VM(n) asm volatile("s_waitcnt vmcnt(%0)" :: "i"(n) : "memory")

template <int AUX>
__device__ __forceinline__ void store16(v4u v, __amdgpu_buffer_rsrc_t r, int voff, int soff) {
    __builtin_amdgcn_raw_buffer_store_b128(v, r, voff, soff, AUX);
    asm volatile("s_nop 1" :: "v"(v) : "memory");
}

__device__ __forceinline__ unsigned rfl(unsigned v) { return __builtin_amdgcn_readfirstlane(v); }

__global__ void __launch_bounds__(NTHREADS, 2) a2a_gemm_kernel(Params p) {
    extern __shared__ __attribute__((aligned(16))) unsigned char smem[];
    LAS unsigned char* L8 = (LAS unsigned char*)smem;
    LAS unsigned* sync = (LAS unsigned*)(L8 + L_SYNC);
    STAMP(0);
    asm volatile("s_memrealtime s[96:97]\n\ts_load_dword s98, %0, 0x0\n\ts_load_dword s99, %0, 0x40"
                 :: "s"(__builtin_amdgcn_kernarg_segment_ptr()) : "s96", "s97", "s98", "s99");
#define ENTRY_TIME(t_) unsigned long long t_; asm volatile("s_waitcnt lgkmcnt(0)\n\ts_mov_b64 %0, s[96:97]" : "=s"(t_))
    const int tid = threadIdx.x;
    const int lane = tid & 63;
    const int wave = rfl(tid >> 6);
    const int b = blockIdx.x;

    const int xg = b & 7, idx = b >> 3;
    const int mb = idx & 7;
#define TILE_COL(v_) ((((v_) - (int)((size_t)p.w >> 7)) & 31) * 128)
    const int cls = idx >> 3;

    if (__builtin_expect(wave < 2, 1)) {
        __builtin_amdgcn_s_setprio(3);
        const int h = wave;
        sync[h * 32 + (lane & 31)] = 0u;
        const int rg = (b >> 1) & 15, rd = b >> 5, i0 = (~b & 1) * 4 + h * 2;
        const bool slow = EXEMPT && cls == 3 && !(b & 1), both = EXEMPT && cls == 3 && (b & 1);
        const int rank = p.rank;
        unsigned char* const pe0 = p.peer[i0];
        unsigned char* const pe1 = p.peer[i0 + 1];
        int xv0 = (lane >> 2) * SH + (((lane & 3) ^ ((lane >> 4) & 3)) * 16);
        int xv1 = xv0 + 16 * SH - 1024;
        int soff = rg * 32 * SH + (rd ^ ((rg >> 1) & ROT)) * 64;
        int lst = (int)(size_t)L8 + L_STAGE + h * 4096;
        int rv = L_STAGE + h * 4096 + (lane & 31) * 64 + ((((lane >> 5) * 2) ^ ((lane >> 2) & 3)) * 16);
        asm volatile("" : "+v"(xv0), "+v"(xv1), "+v"(rv));
        asm volatile("" : "+s"(soff), "+s"(lst));
        asm volatile("s_waitcnt lgkmcnt(0)" :: "s"(p.x), "s"(p.myws), "s"(rank), "s"(pe0), "s"(pe1), "s"(p.sx), "s"(p.sw) : "memory");
        if (b >= 64) asm volatile("s_barrier" ::: "memory");
        v4i xr;
        xr.x = (int)(size_t)p.x; xr.y = (int)((size_t)p.x >> 32) & 0xffff; xr.z = 4096 * SH; xr.w = 0x00020000;
#define SLOAD(i_, l_) \
        asm volatile("s_mov_b32 m0, %4\n\ts_nop 0\n\t" \
                     "buffer_load_dwordx4 %0, %2, %3 offen lds\n\t" \
                     "buffer_load_dwordx4 %1, %2, %3 offen offset:1024 lds" \
                     :: "v"(xv0), "v"(xv1), "s"(xr), "s"((soff ^ (((i_) & SRCX) * 64)) + (((rank + 1 + i0 + (i_)) & 7) << 18)), "s"(lst + (l_)) : "memory")
        if (!slow) {
            SLOAD(0, 0);
            if (i0 < 6) SLOAD(1, 2048);
        }
        GAS unsigned* ctr = (GAS unsigned*)(p.myws + WS_CTR) + b * 32 + wave;
        asm volatile("global_load_dword v199, %0, %1 sc1\n\ts_load_dword s94, %2, 0x0\n\ts_load_dword s95, %3, 0x0"
                     :: "v"(0), "s"(ctr), "s"(p.sx), "s"(p.sw) : "memory", "v199", "s94", "s95");
        if (b < 64) asm volatile("s_barrier" ::: "memory");
        STAMP(1);
        if (both) {
            SLOAD(4, 8192);
            if (!h) SLOAD(5, 10240);
        }
        if ((!(b & 1) || xg == 7) && (cls == 1 || cls == 2)) {
            v4i wr;
            wr.x = (int)(size_t)p.w; wr.y = (int)((size_t)p.w >> 32) & 0xffff; wr.z = GK * GN; wr.w = 0x00020000;
            const int wo = lane * GN + TILE_COL(xg * 4 + 3), ws = (((idx - 8) * 2 + h) * 128) * GN;
            asm volatile("s_nop 4\n\tbuffer_load_dword v200, %0, %1, %2 offen\n\tbuffer_load_dword v200, %0, %1, %3 offen"
                         :: "v"(wo), "s"(wr), "s"(ws), "s"(ws + 64 * GN) : "memory", "v200");
            WAIT_VM(2);
        } else
        WAIT_VM(0);
        unsigned callr;
        asm volatile("v_readfirstlane_b32 %0, v199" : "=s"(callr) :: "memory");
        STAMP(2);
        v4u d[4][2];
#pragma unroll
        for (int i = 0; i < 2; ++i) {
            d[i][0] = *(LAS v4u*)(L8 + i * 2048 + rv);
            d[i][1] = *(LAS v4u*)(L8 + i * 2048 + (rv ^ 16));
        }
        const auto d0 = __builtin_amdgcn_make_buffer_rsrc((void*)pe0, 0, (int)WS_END, 0x00020000);
        const auto d1 = __builtin_amdgcn_make_buffer_rsrc((void*)pe1, 0, (int)WS_END, 0x00020000);
        const unsigned call = callr + 1u;
        const int bo = (int)WS_A + (int)(call % 3u + 1u) * A_PAR;
        const int dso = bo + (rg * NREM + rd * 7 + 6 - i0) * FRAG;
#pragma unroll
        for (int i = 0; i < 2; ++i) {
            d[i][0].x = max(~d[i][0].x, 1u);
            d[i][1].x = max(~d[i][1].x, 1u);
        }
#define BURST() do { \
            store16<SEND_AUX>(d[0][0], d0, lane * 16, dso); \
            store16<SEND_AUX>(d[0][1], d0, lane * 16 + 1024, dso); \
            if (i0 < 6) { \
                store16<SEND_AUX>(d[1][0], d1, lane * 16, dso - FRAG); \
                store16<SEND_AUX>(d[1][1], d1, lane * 16 + 1024, dso - FRAG); \
            } } while (0)
        if (b < HOLD) BURST();
        if (b < HOLD) asm volatile("s_barrier" ::: "memory");
        float scale;
        asm volatile("s_waitcnt lgkmcnt(0)\n\tv_mov_b32 %0, s94\n\tv_mul_f32 %0, s95, %0" : "=v"(scale));
        *(volatile LAS float*)(sync + 16) = scale;
        *(volatile LAS unsigned*)(sync + (lane & 1) * 2) = (unsigned)bo;
        if (lane == 0) __hip_atomic_store(ctr, call, __ATOMIC_RELAXED, __HIP_MEMORY_SCOPE_AGENT);
        if (!slow) {
            ENTRY_TIME(tentry);
            const unsigned tstart = (unsigned)tentry;
            if (both) {
#pragma unroll
                for (int i = 2; i < 4; ++i) {
                    d[i][0] = *(LAS v4u*)(L8 + (i + 2) * 2048 + rv);
                    d[i][1] = *(LAS v4u*)(L8 + (i + 2) * 2048 + (rv ^ 16));
                    d[i][0].x = max(~d[i][0].x, 1u);
                    d[i][1].x = max(~d[i][1].x, 1u);
                }
            }
            const auto d2 = __builtin_amdgcn_make_buffer_rsrc((void*)p.peer[4 + 2 * h], 0, (int)WS_END, 0x00020000);
            const auto d3 = __builtin_amdgcn_make_buffer_rsrc((void*)p.peer[5], 0, (int)WS_END, 0x00020000);
            const unsigned due = SLOT_T0 + ((unsigned)(b >> 1) * SLOT_Q >> 4);
            while ((unsigned)__builtin_amdgcn_s_memrealtime() - tstart < due) __builtin_amdgcn_s_sleep(1);
            STAMP(3);
            if (b >= HOLD) BURST();
            if (both) {
                store16<SEND_AUX>(d[2][0], d2, lane * 16, dso - 4 * FRAG);
                store16<SEND_AUX>(d[2][1], d2, lane * 16 + 1024, dso - 4 * FRAG);
                if (!h) {
                    store16<SEND_AUX>(d[3][0], d3, lane * 16, dso - 5 * FRAG);
                    store16<SEND_AUX>(d[3][1], d3, lane * 16 + 1024, dso - 5 * FRAG);
                }
            }
            STAMP(4);
            if (!EXEMPT || cls < 3) {
                const auto zr = __builtin_amdgcn_make_buffer_rsrc((void*)(p.myws + WS_A), 0, (int)WS_END, 0x00020000);
                const v4u z = {0u, 0u, 0u, 0u};
                if (EXEMPT) {
                    const int zo = (int)((call + 2u) % 3u + 1u) * A_PAR + (b * 28 / 3 + h * 5) * 1024;
#pragma unroll
                    for (int u = 0; u < 5; ++u) store16<ZERO_AUX>(z, zr, lane * 16, zo + u * 1024);
                } else {
                    const int zo = (int)((call + 2u) % 3u + 1u) * A_PAR + b * 7168 + h * 3072;
#pragma unroll
                    for (int u = 0; u < 4; ++u) if (u < 3 || h) store16<ZERO_AUX>(z, zr, lane * 16, zo + u * 1024);
                }
            }
        }
        STAMP_OUT();
    } else if (wave >= 4) {
        __builtin_amdgcn_s_setprio(PRIO_COMP);
        const int rank = p.rank;
        const int cw = wave - 4;
        const int mh = cw & 1, kp = cw >> 1;
        const int rg = mb * 2 + mh;
        const int lb = (int)(size_t)L8;
        const int stg = lb + (kp * RING + RING - 2 + mh) * SLOT;
        {
            int xv0 = (lane >> 3) * SH + (((lane & 7) ^ ((lane >> 4) & 3)) * 16);
            int xv1 = (xv0 ^ 64) + 8 * SH - 1024;
            int xv2 = xv0 + 0x1800, xv3 = xv1 + 0x1800;
            asm volatile("" : "+v"(xv0), "+v"(xv1), "+v"(xv2), "+v"(xv3));
            v4i xr;
            xr.x = (int)(size_t)p.x; xr.y = (int)((size_t)p.x >> 32) & 0xffff; xr.z = 4096 * SH; xr.w = 0x00020000;
            const int xsoff = (rank * SH + rg * 32) * SH + kp * 128;
#define XLD(so_) \
        "buffer_load_dwordx4 %0, %4, " so_ " offen lds\n\t" \
        "buffer_load_dwordx4 %1, %4, " so_ " offen offset:1024 lds\n\t" \
        "buffer_load_dwordx4 %2, %4, " so_ " offen offset:2048 lds\n\t" \
        "buffer_load_dwordx4 %3, %4, " so_ " offen offset:3072 lds\n\t"
            asm volatile("s_mov_b32 m0, %7\n\t"
                         "s_barrier\n\t"
                         XLD("%5")
                         "s_add_u32 m0, m0, 4096\n\t"
                         "s_nop 0\n\t"
                         XLD("%6")
                         :: "v"(xv0), "v"(xv1), "v"(xv2), "v"(xv3), "s"(xr), "s"(xsoff), "s"(xsoff + 256), "s"(stg) : "memory", "scc");
            if (b < HOLD) asm volatile("s_barrier" ::: "memory");
        }
        int lane2 = lane;
        asm volatile("" : "+v"(lane2));
#define lane lane2
        const unsigned char* ab = p.myws + WS_A + rg * NREM * FRAG;
        v4i ar;
        ar.x = (int)(size_t)ab; ar.y = (int)((size_t)ab >> 32) & 0xffff; ar.z = NREM * FRAG; ar.w = 0x00020000;
        const int avoff = lane * 16;
        const int rv = stg + (lane & 31) * 128 + ((((lane >> 5) * 2) ^ ((lane >> 1) & 7)) * 16);
        const int g = lane >> 4, q = (lane & 15) >> 1;
        const int rdl = lb + kp * RING * SLOT + ((g >> 1) * 32 + q) * 128 + (g & 1) * 16 + (lane & 1) * 8;
        const int sw = (q >> 1) ^ (2 * kp);
        const int rd0 = rdl + ((0 ^ sw) * 32), rd1 = rdl + ((1 ^ sw) * 32);
        const int rd2 = rdl + ((2 ^ sw) * 32), rd3 = rdl + ((3 ^ sw) * 32);
        const int a_syn = lb + L_SYNC + 8 * kp, a_cons = lb + L_SYNC + 32 + 4 * cw;
        int aoff = kp * FRAG;
        static_assert((32 - RING) % RING + 4 <= RING, "the four staging slots are adjacent");
        const int a_scale = lb + L_SYNC + 64;
        const int s0 = ((32 - RING) % RING + 2 * mh) * SLOT;
        int e_w[4], e_r0[2], e_r1[2];
#pragma unroll
        for (int gq = 0; gq < 4; ++gq) e_w[gq] = lb + kp * RING * SLOT + s0 + (lane & 31) * 256 + ((((lane >> 5) ^ (lane & 7)) * 16) ^ (gq * 32));
#pragma unroll
        for (int i = 0; i < 2; ++i) {
            const int ra = (lane >> 4) * 256 + ((((lane & 15) ^ (lane >> 4)) * 16) ^ (i * 64));
            e_r0[i] = lb + kp * RING * SLOT + s0 + ra;
            e_r1[i] = lb + (kp ^ 1) * RING * SLOT + s0 + SLOT + ra;
        }
        LAS unsigned* e_sib = sync + 8 + (cw ^ 1);
        LAS unsigned* e_par = sync + 12 + (cw ^ 2);
        int e_ov = (lane >> 4) * GN * 4 + (lane & 15) * 16;
        int e_os = (rg * 32 * GN + TILE_COL(xg * 4 + cls) + kp * 64) * 4;
        float* e_out = p.out;
        asm volatile("" : "+v"(e_w[0]), "+v"(e_w[1]), "+v"(e_w[2]), "+v"(e_w[3]), "+v"(e_r0[0]), "+v"(e_r0[1]), "+v"(e_r1[0]), "+v"(e_r1[1]),
                          "+v"(e_sib), "+v"(e_par), "+v"(e_ov));
        asm volatile("" : "+s"(e_os));
        asm volatile("" : "+s"(e_out));
#undef lane
        v16f acc0, acc1, acc2, acc3;
        v16i clob0, clob1, clob2, clob3, clob4;
        int va0, va1, va2, va3, vt2, vt3;
        int s_need, s_wslot, s_wrdy, s_t0, s_t1;
        unsigned long long s_ret, s_now;
        ENTRY_TIME(tentry);
        const unsigned tlo = (unsigned)tentry;
#define DBG_ASM(s_)
#define DBG_SIDE(s_)
#define DBG_ASM0
#define DBG_OPS
#define AREG(s_) "v[160+8*" #s_ ":167+8*" #s_ "]"
#define ALO(s_) "v[160+8*" #s_ ":163+8*" #s_ "]"
#define AHI(s_) "v[164+8*" #s_ ":167+8*" #s_ "]"
#define LDLO(p_, so_) "buffer_load_dwordx4 " ALO(p_) ", %[avoff], s[48:51], " so_ " offen sc1\n\t"
#define LDHI(p_, so_) "buffer_load_dwordx4 " AHI(p_) ", %[avoff], s[48:51], " so_ " offen offset:1024 sc1\n\t"
#define RDB(base_, va_) \
        "ds_read_b64_tr_b8 v[" #base_ ":" #base_ "+1], " va_ "\n\t" \
        "ds_read_b64_tr_b8 v[" #base_ "+2:" #base_ "+3], " va_ " offset:1024\n\t" \
        "ds_read_b64_tr_b8 v[" #base_ "+4:" #base_ "+5], " va_ " offset:2048\n\t" \
        "ds_read_b64_tr_b8 v[" #base_ "+6:" #base_ "+7], " va_ " offset:3072\n\t"
#define WHOLE "s_cmp_eq_u64 vcc, 0\n\t"
#define CHKBR(s_) "s_cbranch_vccnz .Lretry" #s_ "_%=\n"
#define CONSW(s_) "ds_write_b32 %[acons], %[vt2]\n\t"
#define LG5(s_) "s_waitcnt lgkmcnt(5)\n\t"
#define POS(s_, lo_, hi_, last_) \
        ".Lpos" #s_ "_%=:\n\t" \
        "s_waitcnt vmcnt(%[vmd])\n" \
        ".Lchk" #s_ "_%=:\n\t" \
        "v_min_u32 %[vt3], v[160+8*" #s_ "], v[164+8*" #s_ "]\n\t" \
        "v_cmp_eq_u32 vcc, 0, %[vt3]\n\t" \
        "v_not_b32 v[160+8*" #s_ "], v[160+8*" #s_ "]\n\t" \
        "v_not_b32 v[164+8*" #s_ "], v[164+8*" #s_ "]\n\t" \
        WHOLE \
        "s_cselect_b32 %[t0], %[wrdy], 0\n\t" \
        "s_cmp_lt_u32 %[t0], %[need]\n\t" \
        "s_waitcnt lgkmcnt(10)\n\t" \
        "s_cbranch_scc1 .Lslow" #s_ "_%=\n" \
        ".Lcomp" #s_ "_%=:\n\t" \
        "v_mfma_f32_32x32x64_f8f6f4 v[64:79], v[128:135], " AREG(s_) ", v[64:79] cbsz:1 blgp:1\n\t" \
        lo_ \
        RDB(128, "%[va0]") \
        "s_add_u32 %[wslot], %[wslot], %[slotb]\n\t" \
        "s_cmp_eq_u32 %[wslot], %[ringb]\n\t" \
        "s_cselect_b32 %[wslot], 0, %[wslot]\n\t" \
        "v_add_u32 %[va0], %[wslot], %[rd0]\n\t" \
        "s_waitcnt lgkmcnt(10)\n\t" \
        "v_mfma_f32_32x32x64_f8f6f4 v[80:95], v[136:143], " AREG(s_) ", v[80:95] cbsz:1 blgp:1\n\t" \
        hi_ \
        RDB(136, "%[va1]") \
        "ds_read_b32 v201, %[asyn] offset:4\n\t" \
        CONSW(s_) \
        "v_add_u32 %[va1], %[wslot], %[rd1]\n\t" \
        "s_waitcnt lgkmcnt(10)\n\t" \
        "v_mfma_f32_32x32x64_f8f6f4 v[96:111], v[144:151], " AREG(s_) ", v[96:111] cbsz:1 blgp:1\n\t" \
        RDB(144, "%[va2]") \
        "v_add_u32 %[va2], %[wslot], %[rd2]\n\t" \
        "v_add_u32 %[vt2], 1, %[vt2]\n\t" \
        "s_add_u32 %[need], %[need], 1\n\t" \
        DBG_ASM(s_) \
        LG5(s_) \
        "v_readfirstlane_b32 %[wrdy], v201\n\t" \
        "v_mfma_f32_32x32x64_f8f6f4 v[112:127], v[152:159], " AREG(s_) ", v[112:127] cbsz:1 blgp:1\n\t" \
        last_ \
        RDB(152, "%[va3]") \
        "v_add_u32 %[va3], %[wslot], %[rd3]\n"
#define POSL(s_, p_, last_) POS(s_, LDLO(p_, "%[aoff]"), LDHI(p_, "%[aoff]") "s_add_u32 %[aoff], %[aoff], 0x1000\n\t", last_)
#define POS0(s_) POS(s_, "", "", "")
#define LAST "s_cmp_eq_u32 %[need], 34\n\ts_cbranch_scc1 .Ldone_%=\n\t"
#define AGAIN(n_) "s_add_u32 %[t1], %[t1], 0x1000\n\t" LDLO(n_, "%[t1]") LDHI(n_, "%[t1]")
#define SIDE(s_, more_) \
        ".Lslow" #s_ "_%=:\n\t" \
        CHKBR(s_) \
        ".Lwspin" #s_ "_%=:\n\t" \
        "ds_read_b32 v201, %[asyn] offset:4\n\t" \
        "s_waitcnt lgkmcnt(0)\n\t" \
        "v_readfirstlane_b32 %[wrdy], v201\n\t" \
        "s_cmp_lt_u32 %[wrdy], %[need]\n\t" \
        "s_cbranch_scc1 .Lwspin" #s_ "_%=\n\t" \
        "s_branch .Lcomp" #s_ "_%=\n" \
        ".Lretry" #s_ "_%=:\n\t" \
        "s_getpc_b64 %[ret]\n\t" \
        "s_branch .Lagain_%=\n\t" \
        "s_cbranch_scc1 .Lwspin" #s_ "_%=\n\t" \
        LDLO(s_, "%[t1]") LDHI(s_, "%[t1]") \
        "s_waitcnt vmcnt(0)\n\t" \
        "v_min_u32 %[vt3], v[160+8*" #s_ "], v[164+8*" #s_ "]\n\t" \
        "v_cmp_eq_u32 vcc, 0, %[vt3]\n\t" \
        "s_nop 1\n\t" \
        "s_cbranch_vccnz .Lretry" #s_ "_%=\n\t" \
        more_ \
        "s_branch .Lchk" #s_ "_%=\n" \
        DBG_SIDE(s_)
#define XRD(d_, off_) \
        "ds_read_b128 v[" #d_ ":" #d_ "+3], %[rv] offset:" #off_ "\n\t" \
        "ds_read_b128 v[" #d_ "+4:" #d_ "+7], %[va1] offset:" #off_ "\n\t" \
        "ds_read_b128 v[" #d_ "+8:" #d_ "+11], %[va2] offset:" #off_ "\n\t" \
        "ds_read_b128 v[" #d_ "+12:" #d_ "+15], %[va3] offset:" #off_ "\n\t"
#define XINV(r_) "v_not_b32 v" #r_ ", v" #r_ "\n\tv_max_u32 v" #r_ ", 1, v" #r_ "\n\t"
#define Z8(b_) \
        "v_mov_b64 v[" #b_ ":" #b_ "+1], 0\n\tv_mov_b64 v[" #b_ "+2:" #b_ "+3], 0\n\t" \
        "v_mov_b64 v[" #b_ "+4:" #b_ "+5], 0\n\tv_mov_b64 v[" #b_ "+6:" #b_ "+7], 0\n\t"
        asm volatile(
            "v_xor_b32 %[va1], 16, %[rv]\n\t"
            "v_xor_b32 %[va2], 64, %[rv]\n\t"
            "v_xor_b32 %[va3], 0x50, %[rv]\n\t"
            Z8(64) Z8(72) Z8(80) Z8(88) Z8(96) Z8(104) Z8(112) Z8(120)
            "s_mov_b32 %[need], 2\n\t"
            "s_mov_b32 %[wslot], %[slotb]\n\t"
            "v_mov_b32 %[vt2], 1\n\t"
            "s_waitcnt vmcnt(4)\n\t"
            XRD(160, 0)
            "s_waitcnt vmcnt(0)\n\t"
            XRD(176, 4096)
            ".Lfirst_%=:\n\t"
            "ds_read_b64 v[200:201], %[asyn]\n\t"
            "s_waitcnt lgkmcnt(0)\n\t"
            "v_readfirstlane_b32 %[t1], v200\n\t"
            "v_readfirstlane_b32 %[wrdy], v201\n\t"
            "s_min_u32 %[t0], %[t1], %[wrdy]\n\t"
            "s_cmp_eq_u32 %[t0], 0\n\t"
            "s_cbranch_scc1 .Lfirst_%=\n\t"
            "s_add_u32 s48, s48, %[t1]\n\t"
            "s_addc_u32 s49, s49, 0\n\t"
            DBG_ASM0
            XINV(160) XINV(164) XINV(168) XINV(172) XINV(176) XINV(180) XINV(184) XINV(188)
            RDB(128, "%[rd0]") RDB(136, "%[rd1]") RDB(144, "%[rd2]") RDB(152, "%[rd3]")
            "ds_write_b32 %[acons], %[vt2]\n\t"
            "v_add_u32 %[va0], %[wslot], %[rd0]\n\t"
            "v_add_u32 %[va1], %[wslot], %[rd1]\n\t"
            "v_add_u32 %[va2], %[wslot], %[rd2]\n\t"
            "v_add_u32 %[va3], %[wslot], %[rd3]\n\t"
            "s_waitcnt lgkmcnt(1)\n"
            POS0(0) POSL(1, 4, "") POSL(2, 1, "") POSL(3, 2, LAST) POSL(4, 3, "")
            "s_branch .Lpos1_%=\n"
            SIDE(0, "") SIDE(1, AGAIN(2) AGAIN(3)) SIDE(2, AGAIN(3) AGAIN(4)) SIDE(3, AGAIN(4) AGAIN(1)) SIDE(4, AGAIN(1) AGAIN(2))
            ".Lagain_%=:\n\t"
            "ds_write_b32 %[acons], %[vt2]\n\t"
            "s_sleep %[rslp]\n\t"
            "s_memrealtime %[now]\n\t"
            "s_add_u32 s40, s40, 4\n\t"
            "s_addc_u32 s41, s41, 0\n\t"
            "s_sub_u32 %[t1], %[aoff], %[back]\n\t"
            "s_waitcnt lgkmcnt(0)\n\t"
            "s_sub_u32 %[t0], s42, %[tlo]\n\t"
            "s_cmp_gt_u32 %[t0], %[dl]\n\t"
            "s_cbranch_scc0 .Lback_%=\n\t"
            "v_mov_b32 %[vt3], 0x7fc00000\n\t"
            "ds_write_b32 %[ascale], %[vt3]\n"
            ".Lback_%=:\n\t"
            "s_setpc_b64 %[ret]\n"
            ".Ldone_%=:\n\t"
            "s_waitcnt vmcnt(0) lgkmcnt(0)\n\t"
            "s_nop 15\n\t"
            "s_nop 3"
            : "=&{v[64:79]}"(acc0), "=&{v[80:95]}"(acc1), "=&{v[96:111]}"(acc2), "=&{v[112:127]}"(acc3),
              "=&{v[128:143]}"(clob0), "=&{v[144:159]}"(clob1), "=&{v[160:175]}"(clob2), "=&{v[176:191]}"(clob3),
              "=&{v[192:207]}"(clob4),
              [va0] "=&v"(va0), [va1] "=&v"(va1), [va2] "=&v"(va2), [va3] "=&v"(va3), [vt2] "=&v"(vt2), [vt3] "=&v"(vt3),
              [ret] "=&{s[40:41]}"(s_ret), [now] "=&{s[42:43]}"(s_now), "+{s[48:51]}"(ar),
              [need] "=&s"(s_need), [wslot] "=&s"(s_wslot), [wrdy] "=&s"(s_wrdy), [t0] "=&s"(s_t0), [t1] "=&s"(s_t1),
              [aoff] "+s"(aoff) DBG_OPS
            : [avoff] "v"(avoff), [rd0] "v"(rd0), [rd1] "v"(rd1), [rd2] "v"(rd2), [rd3] "v"(rd3),
              [rv] "v"(rv), [asyn] "v"(a_syn), [acons] "v"(a_cons), [slotb] "i"(SLOT), [ringb] "i"(RING * SLOT),
              [rslp] "i"(RETRY_SLEEP), [vmd] "i"(2 * (PDIST - 1)), [back] "i"(PDIST * 4096),
              [tlo] "s"(tlo), [dl] "i"((unsigned)DEADLINE_TICKS), [ascale] "v"(a_scale)
            : "memory", "scc", "vcc");
        asm volatile("" : "+v"(e_w[0]), "+v"(e_w[1]), "+v"(e_w[2]), "+v"(e_w[3]), "+v"(e_r0[0]), "+v"(e_r0[1]), "+v"(e_r1[0]), "+v"(e_r1[1]));
        STAMP(5);
        {
            while (rfl(*(volatile LAS unsigned*)(e_sib)) < (unsigned)(36 - RING)) {}
            asm volatile("" ::: "memory");
#pragma unroll
            for (int gq = 0; gq < 4; ++gq) {
                *(LAS v4f*)(size_t)(e_w[gq] + SLOT) = v4f{acc2[4 * gq], acc2[4 * gq + 1], acc2[4 * gq + 2], acc2[4 * gq + 3]};
                *(LAS v4f*)(size_t)(e_w[gq] + SLOT + 128) = v4f{acc3[4 * gq], acc3[4 * gq + 1], acc3[4 * gq + 2], acc3[4 * gq + 3]};
            }
            asm volatile("" ::: "memory");
            *(volatile LAS unsigned*)(size_t)(a_cons + 16) = 1u;
            asm volatile("" ::: "memory");
#pragma unroll
            for (int gq = 0; gq < 4; ++gq) {
                *(LAS v4f*)(size_t)(e_w[gq]) = v4f{acc0[4 * gq], acc0[4 * gq + 1], acc0[4 * gq + 2], acc0[4 * gq + 3]};
                *(LAS v4f*)(size_t)(e_w[gq] + 128) = v4f{acc1[4 * gq], acc1[4 * gq + 1], acc1[4 * gq + 2], acc1[4 * gq + 3]};
            }
            v4f u0[8], u1[8];
#pragma unroll
            for (int i = 0; i < 8; ++i) u0[i] = *(LAS v4f*)(size_t)(e_r0[i & 1] + i * 1024);
            while (rfl(*(volatile LAS unsigned*)(e_par)) == 0u) {}
            asm volatile("" ::: "memory");
            const float sc = *(volatile LAS float*)(size_t)(a_scale);
#pragma unroll
            for (int i = 0; i < 8; ++i) u1[i] = *(LAS v4f*)(size_t)(e_r1[i & 1] + i * 1024);
            const auto orr = __builtin_amdgcn_make_buffer_rsrc((void*)e_out, 0, SH * GN * 4, 0x00020000);
#pragma unroll
            for (int i = 0; i < 8; ++i) {
                const v4f y = __builtin_elementwise_maximum((u0[i] + u1[i]) * sc, v4f{0.f, 0.f, 0.f, 0.f});
                store16<OUT_AUX>(__builtin_bit_cast(v4u, y), orr, e_ov, e_os + i * 4 * GN * 4);
            }
        }
        STAMP(6);
        STAMP_OUT();
    } else {
        __builtin_amdgcn_s_setprio(PRIO_LOAD);
        const int rank = p.rank;
        const int kp = wave - 2;
        v4i rs;
        rs.x = (int)(size_t)p.w; rs.y = (int)((size_t)p.w >> 32) & 0xffff; rs.z = GK * GN; rs.w = 0x00020000;
        const int v0 = (lane >> 3) * GN + TILE_COL(xg * 4 + cls) + (((lane & 7) ^ (((lane >> 4) & 3) << 1)) * 16);
        const int v1 = v0 + 8 * GN - 1024, v2 = v0 + 16 * GN - 2048, v3 = v0 + 24 * GN - 3072;
        int tab;
        {
            const int n = lane & 31;
            const int ri = max(2 * (n - 4) + kp, 0), jj = (ri * 37) >> 8;
            const int loc = (n - 4) >> 31;
            int src = (1 + ri - jj * 7) & ~loc;
            const int col = (((n >> 1) * 4 + kp * 2 + (n & 1)) & loc) | ((jj ^ (mb & ROT) ^ ((src - 1) & SRCX)) & ~loc);
            const int need = max(n - RING + 1, n >= RING - 2 ? 1 : 0);
            const int slot = n - ((n * ((256 + RING - 1) / RING)) >> 8) * RING;
            int rest = (col << 18) + slot * SLOT + (lane < 32 ? need : 63);
            asm volatile("" : "+v"(src), "+v"(rest));
            asm volatile("" :: "s"(p.w), "s"(rank));
            tab = (((rank + src) & 7) << 21) + rest;
        }
        const int lbase = (int)(size_t)L8 + kp * RING * SLOT;
        const int a_wrdy = (int)(size_t)L8 + L_SYNC + 8 * kp + 4, a_cons = (int)(size_t)L8 + L_SYNC + 32 + 8 * kp;
        int s_n, s_e, s_so, s_so2, s_need, s_t, vpub;
        v2i vc;
#define WLD(v_, so_, off_) "buffer_load_dwordx4 %[" #v_ "], %[rs], %[" #so_ "] offen offset:" #off_ WMOD " lds\n\t"
#define FILL \
        WLD(v0, so, 0) \
        "ds_read_b64 %[vc], %[acons]\n\t" \
        "s_add_u32 %[n], %[n], 1\n\t" \
        WLD(v1, so, 1024) \
        "v_readlane_b32 %[e], %[tab], %[n]\n\t" \
        WLD(v2, so, 2048) \
        "v_add_u32 %[vpub], 1, %[vpub]\n\t" \
        WLD(v3, so, 3072) \
        "s_add_u32 m0, m0, 4096\n\t" \
        "s_and_b32 %[so], %[e], 0xfffc0000\n\t" \
        WLD(v0, so2, 0) \
        "s_and_b32 %[need], %[e], 63\n\t" \
        "s_and_b32 %[e], %[e], 0x1e000\n\t" \
        WLD(v1, so2, 1024) \
        "s_waitcnt lgkmcnt(0)\n\t" \
        "v_min_u32 v20, v20, v21\n\t" \
        WLD(v2, so2, 2048) \
        "v_readfirstlane_b32 %[t], v20\n\t" \
        WLD(v3, so2, 3072) \
        "s_add_u32 m0, %[e], %[lbase]\n\t" \
        "s_add_u32 %[so2], %[so], 0x20000\n\t"
#define PUB(cnt_) \
        "s_waitcnt vmcnt(" #cnt_ ")\n\t" \
        "ds_write_b32 %[awrdy], %[vpub]\n\t"
#define PUBN(cnt_) "v_add_u32 %[vpub], 1, %[vpub]\n\t" PUB(cnt_)
#define LDBG_IN
#define LDBG_OUT
#define LDBG_OPS
        if (b < HOLD) asm volatile("s_barrier" ::: "memory");
        asm volatile(
            "s_mov_b32 %[n], 0\n\t"
            "v_mov_b32 %[vpub], %[pub0]\n\t"
            "v_readlane_b32 %[so], %[tab], 0\n\t"
            "s_mov_b32 m0, %[lbase]\n\t"
            "s_add_u32 %[so2], %[so], 0x20000\n\t"
            "s_nop 2\n\t"
            "s_barrier\n"
            ".Lpro_%=:\n\t"
            FILL
            "s_cmp_lt_u32 %[n], %[nf]\n\t"
            "s_cbranch_scc1 .Lpro_%=\n"
            ".Lmain_%=:\n\t"
            PUB(%[cnt])
            ".Lgo_%=:\n\t"
            LDBG_IN
            FILL
            "s_cmp_lt_u32 %[t], %[need]\n\t"
            "s_cbranch_scc0 .Lmain_%=\n\t"
            PUB(%[cnt])
            "s_cmp_eq_u32 %[n], 32\n\t"
            "s_cbranch_scc1 .Lend_%=\n"
            ".Lspin_%=:\n\t"
            "ds_read_b64 %[vc], %[acons]\n\t"
            "s_waitcnt lgkmcnt(0)\n\t"
            "v_min_u32 v20, v20, v21\n\t"
            "s_nop 0\n\t"
            "v_readfirstlane_b32 %[t], v20\n\t"
            "s_cmp_lt_u32 %[t], %[need]\n\t"
            "s_cbranch_scc0 .Lgo_%=\n\t"
            "s_sleep 1\n\t"
            "s_branch .Lspin_%=\n"
            LDBG_OUT
            ".Lend_%=:\n\t"
            PUBN(16)
            PUBN(8)
            "v_add_u32 %[vpub], 32, %[vpub]\n\t"
            PUB(0)
            "s_waitcnt lgkmcnt(0)"
            : [n] "=&s"(s_n), [e] "=&s"(s_e), [so] "=&s"(s_so), [so2] "=&s"(s_so2), [need] "=&s"(s_need), [t] "=&s"(s_t),
              [vpub] "=&v"(vpub), [vc] "=&{v[20:21]}"(vc) LDBG_OPS
            : [tab] "v"(tab), [rs] "s"(rs), [lbase] "s"(lbase), [v0] "v"(v0), [v1] "v"(v1), [v2] "v"(v2), [v3] "v"(v3),
              [awrdy] "v"(a_wrdy), [acons] "v"(a_cons), [nf] "i"(NFLY), [cnt] "i"(8 * (NFLY - 1)), [pub0] "i"(1 - NFLY)
            : "memory", "scc");
        STAMP(7);
        STAMP_OUT();
    }
}

extern "C" void kernel_launch(int rank, int nranks, ncclComm_t comm,
                              hipStream_t stream,
                              void* const* d_in, const int* in_sizes, int n_in,
                              void* d_out, int out_size,
                              const int* in_dtypes, int out_dtype,
                              void* const* d_out_all,
                              void* const* d_ws_all, size_t ws_bytes) {
    if (nranks != NR || (unsigned)rank >= (unsigned)NR || ws_bytes < WS_END || n_in < 4 ||
        in_sizes[0] != 4096 * SH || in_sizes[1] != GK * GN || out_size != SH * GN) {
        fprintf(stderr, "unexpected problem geometry\n");
        return;
    }
    HIP_CHECK(hipFuncSetAttribute((const void*)a2a_gemm_kernel, hipFuncAttributeMaxDynamicSharedMemorySize, LDS_BYTES));
    Params p;
    p.x = (const unsigned char*)d_in[0];
    p.w = (const unsigned char*)d_in[1];
    p.sx = (const float*)d_in[2];
    p.sw = (const float*)d_in[3];
    p.out = (float*)d_out;
    p.myws = (unsigned char*)d_ws_all[rank];
    p.rank = rank;
    p.pad = 0;
    for (int i = 0; i < NR; ++i) p.peer[i] = (unsigned char*)d_ws_all[(rank + 1 + i) & 7];
    a2a_gemm_kernel<<<NWG, NTHREADS, LDS_BYTES, stream>>>(p);
    HIP_CHECK(hipGetLastError());
}
```

```cpp
#include <hip/hip_runtime.h>
#include <rccl/rccl.h>
#include <cstdio>

#define HIP_CHECK(expr) do { hipError_t _e = (expr); if (_e != hipSuccess) { \
    fprintf(stderr, "HIP error %s at %s:%d\n", hipGetErrorString(_e), __FILE__, __LINE__); \
    return; } } while (0)

#define GAS __attribute__((address_space(1)))
#define LAS __attribute__((address_space(3)))
typedef int v2i __attribute__((ext_vector_type(2)));
typedef int v4i __attribute__((ext_vector_type(4)));
typedef unsigned v4u __attribute__((ext_vector_type(4)));
typedef float v4f __attribute__((ext_vector_type(4)));
typedef float v16f __attribute__((ext_vector_type(16)));
typedef int v16i __attribute__((ext_vector_type(16)));

constexpr int NR = 8;
constexpr int GN = 4096;
constexpr int GK = 4096;
constexpr int SH = 512;
constexpr int NWG = 256;
constexpr int NTHREADS = 8 * 64;
constexpr int NREM = 56;
constexpr int FRAG = 2048;

constexpr size_t WS_CTR = 0;
constexpr size_t WS_A = 0;
constexpr int A_PAR = 1 << 21;
static_assert(16 * NREM * FRAG <= A_PAR, "");
constexpr size_t WS_END = WS_A + 4 * (size_t)A_PAR;

constexpr int RING = 9;
constexpr int NFLY = 4;
constexpr int PDIST = 3;
constexpr unsigned SLOT_Q = 70;
constexpr unsigned SLOT_T0 = 30;
constexpr int HOLD = 32;
constexpr int ROT = 3;
constexpr int WT = 17;
constexpr int SLOT = 8192;
constexpr int L_SYNC = 2 * RING * SLOT;
constexpr int L_STAGE = L_SYNC + 256;
constexpr int LDS_BYTES = L_STAGE + 14336;
static_assert(LDS_BYTES <= 160 * 1024 && NFLY == 4 && PDIST == 3, "the asm below is written for these values");

constexpr unsigned DEADLINE_TICKS = 40000000u;

struct Params {
    const unsigned char* x;
    const unsigned char* w;
    const float* sx;
    const float* sw;
    float* out;
    unsigned char* myws;
    int rank;
    int pad;
    unsigned char* peer[NR];
};

#define WAIT_VM(n) asm volatile("s_waitcnt vmcnt(%0)" :: "i"(n) : "memory")

template <int AUX>
__device__ __forceinline__ void store16(v4u v, __amdgpu_buffer_rsrc_t r, int voff, int soff) {
    __builtin_amdgcn_raw_buffer_store_b128(v, r, voff, soff, AUX);
    asm volatile("s_nop 1" :: "v"(v) : "memory");
}

__device__ __forceinline__ unsigned rfl(unsigned v) { return __builtin_amdgcn_readfirstlane(v); }

__global__ void __launch_bounds__(NTHREADS, 2) a2a_gemm_kernel(Params p) {
    extern __shared__ __attribute__((aligned(16))) unsigned char smem[];
    LAS unsigned char* L8 = (LAS unsigned char*)smem;
    LAS unsigned* sync = (LAS unsigned*)(L8 + L_SYNC);
    asm volatile("s_memrealtime s[96:97]\n\ts_load_dword s98, %0, 0x0\n\ts_load_dword s99, %0, 0x40"
                 :: "s"(__builtin_amdgcn_kernarg_segment_ptr()) : "s96", "s97", "s98", "s99");
#define ENTRY_TIME(t_) unsigned long long t_; asm volatile("s_waitcnt lgkmcnt(0)\n\ts_mov_b64 %0, s[96:97]" : "=s"(t_))
    const int tid = threadIdx.x;
    const int lane = tid & 63;
    const int wave = rfl(tid >> 6);
    const int b = blockIdx.x;

    const int xg = b & 7, idx = b >> 3;
    const int mb = idx & 7;
#define TILE_COL(v_) ((((v_) - (int)((size_t)p.w >> 7)) & 31) * 128)
    const int cls = idx >> 3;

    if (__builtin_expect(wave < 2, 1)) {
        __builtin_amdgcn_s_setprio(1);
        const int rank = p.rank;
        const int kp = wave;
        v4i rs;
        rs.x = (int)(size_t)p.w; rs.y = (int)((size_t)p.w >> 32) & 0xffff; rs.z = GK * GN; rs.w = 0x00020000;
        const int v0 = (lane >> 3) * GN + TILE_COL(xg * 4 + cls) + (((lane & 7) ^ (((lane >> 4) & 3) << 1)) * 16);
        const int v1 = v0 + 8 * GN - 1024, v2 = v0 + 16 * GN - 2048, v3 = v0 + 24 * GN - 3072;
        int tab;
        {
            const int n = lane & 31;
            const int ri = max(2 * (n - 4) + kp, 0), jj = (ri * 37) >> 8;
            const int loc = (n - 4) >> 31;
            int src = (1 + ri - jj * 7) & ~loc;
            const int col = (((n >> 1) * 4 + kp * 2 + (n & 1)) & loc) | ((jj ^ (mb & ROT) ^ ((src - 1) & 1)) & ~loc);
            const int need = max(n - RING + 1, n >= RING - 2 ? 1 : 0);
            const int slot = n - ((n * ((256 + RING - 1) / RING)) >> 8) * RING;
            int rest = (col << 18) + slot * SLOT + (lane < 32 ? need : 63);
            asm volatile("" : "+v"(src), "+v"(rest));
            asm volatile("" :: "s"(p.w), "s"(rank));
            tab = (((rank + src) & 7) << 21) + rest;
        }
        const int lbase = (int)(size_t)L8 + kp * RING * SLOT;
        const int a_wrdy = (int)(size_t)L8 + L_SYNC + 8 * kp + 4, a_cons = (int)(size_t)L8 + L_SYNC + 32 + 8 * kp;
        int s_n, s_e, s_so, s_so2, s_need, s_t, vpub;
        v2i vc;
#define WLD(v_, so_, off_) "buffer_load_dwordx4 %[" #v_ "], %[rs], %[" #so_ "] offen offset:" #off_ " lds\n\t"
#define FILL \
        WLD(v0, so, 0) \
        "ds_read_b64 %[vc], %[acons]\n\t" \
        "s_add_u32 %[n], %[n], 1\n\t" \
        WLD(v1, so, 1024) \
        "v_readlane_b32 %[e], %[tab], %[n]\n\t" \
        WLD(v2, so, 2048) \
        "v_add_u32 %[vpub], 1, %[vpub]\n\t" \
        WLD(v3, so, 3072) \
        "s_add_u32 m0, m0, 4096\n\t" \
        "s_and_b32 %[so], %[e], 0xfffc0000\n\t" \
        WLD(v0, so2, 0) \
        "s_and_b32 %[need], %[e], 63\n\t" \
        "s_and_b32 %[e], %[e], 0x1e000\n\t" \
        WLD(v1, so2, 1024) \
        "s_waitcnt lgkmcnt(0)\n\t" \
        "v_min_u32 v20, v20, v21\n\t" \
        WLD(v2, so2, 2048) \
        "v_readfirstlane_b32 %[t], v20\n\t" \
        WLD(v3, so2, 3072) \
        "s_add_u32 m0, %[e], %[lbase]\n\t" \
        "s_add_u32 %[so2], %[so], 0x20000\n\t"
#define PUB(cnt_) \
        "s_waitcnt vmcnt(" #cnt_ ")\n\t" \
        "ds_write_b32 %[awrdy], %[vpub]\n\t"
#define PUBN(cnt_) "v_add_u32 %[vpub], 1, %[vpub]\n\t" PUB(cnt_)
        if (b < 64) asm volatile("s_barrier" ::: "memory");
        if (b < HOLD) asm volatile("s_barrier" ::: "memory");
        asm volatile(
            "s_mov_b32 %[n], 0\n\t"
            "v_mov_b32 %[vpub], %[pub0]\n\t"
            "v_readlane_b32 %[so], %[tab], 0\n\t"
            "s_mov_b32 m0, %[lbase]\n\t"
            "s_add_u32 %[so2], %[so], 0x20000\n\t"
            "s_nop 3\n"
            ".Lpro_%=:\n\t"
            FILL
            "s_cmp_lg_u32 %[n], %[barn]\n\t"
            "s_cbranch_scc1 .Lnob_%=\n\t"
            "s_barrier\n"
            ".Lnob_%=:\n\t"
            "s_cmp_lt_u32 %[n], %[nf]\n\t"
            "s_cbranch_scc1 .Lpro_%=\n"
            ".Lmain_%=:\n\t"
            PUB(%[cnt])
            ".Lgo_%=:\n\t"
            FILL
            "s_cmp_lt_u32 %[t], %[need]\n\t"
            "s_cbranch_scc0 .Lmain_%=\n\t"
            PUB(%[cnt])
            "s_cmp_eq_u32 %[n], 32\n\t"
            "s_cbranch_scc1 .Lend_%=\n"
            ".Lspin_%=:\n\t"
            "ds_read_b64 %[vc], %[acons]\n\t"
            "s_waitcnt lgkmcnt(0)\n\t"
            "v_min_u32 v20, v20, v21\n\t"
            "s_nop 0\n\t"
            "v_readfirstlane_b32 %[t], v20\n\t"
            "s_cmp_lt_u32 %[t], %[need]\n\t"
            "s_cbranch_scc0 .Lgo_%=\n\t"
            "s_sleep 1\n\t"
            "s_branch .Lspin_%=\n"
            ".Lend_%=:\n\t"
            PUBN(16)
            PUBN(8)
            "v_add_u32 %[vpub], 32, %[vpub]\n\t"
            PUB(0)
            "s_waitcnt lgkmcnt(0)"
            : [n] "=&s"(s_n), [e] "=&s"(s_e), [so] "=&s"(s_so), [so2] "=&s"(s_so2), [need] "=&s"(s_need), [t] "=&s"(s_t),
              [vpub] "=&v"(vpub), [vc] "=&{v[20:21]}"(vc)
            : [tab] "v"(tab), [rs] "s"(rs), [lbase] "s"(lbase), [v0] "v"(v0), [v1] "v"(v1), [v2] "v"(v2), [v3] "v"(v3),
              [awrdy] "v"(a_wrdy), [acons] "v"(a_cons), [nf] "i"(NFLY), [cnt] "i"(8 * (NFLY - 1)), [pub0] "i"(1 - NFLY), [barn] "s"(min(b >> 6, 1))
            : "memory", "scc");
    } else if (__builtin_expect(wave < 4, 1)) {
        __builtin_amdgcn_s_setprio(3);
        const int h = wave - 2;
        sync[h * 32 + (lane & 31)] = 0u;
        const int rg = (b >> 1) & 15, rd = b >> 5, i0 = (~b & 1) * 4 + h * 2;
        const bool slow = cls == 3 && !(b & 1), both = cls == 3 && (b & 1);
        const int rank = p.rank;
        unsigned char* const pe0 = p.peer[i0];
        unsigned char* const pe1 = p.peer[i0 + 1];
        int xv0 = (lane >> 2) * SH + (((lane & 3) ^ ((lane >> 4) & 3)) * 16);
        int xv1 = xv0 + 16 * SH - 1024;
        int soff = rg * 32 * SH + (rd ^ ((rg >> 1) & ROT)) * 64;
        int lst = (int)(size_t)L8 + L_STAGE + h * 4096;
        int rv = L_STAGE + h * 4096 + (lane & 31) * 64 + ((((lane >> 5) * 2) ^ ((lane >> 2) & 3)) * 16);
        asm volatile("" : "+v"(xv0), "+v"(xv1), "+v"(rv));
        asm volatile("" : "+s"(soff), "+s"(lst));
        asm volatile("s_waitcnt lgkmcnt(0)" :: "s"(p.x), "s"(p.myws), "s"(rank), "s"(pe0), "s"(pe1), "s"(p.sx), "s"(p.sw) : "memory");
        v4i xr;
        xr.x = (int)(size_t)p.x; xr.y = (int)((size_t)p.x >> 32) & 0xffff; xr.z = 4096 * SH; xr.w = 0x00020000;
#define SLOAD(i_, l_) \
        asm volatile("s_mov_b32 m0, %4\n\ts_nop 0\n\t" \
                     "buffer_load_dwordx4 %0, %2, %3 offen lds\n\t" \
                     "buffer_load_dwordx4 %1, %2, %3 offen offset:1024 lds" \
                     :: "v"(xv0), "v"(xv1), "s"(xr), "s"((soff ^ (((i_) & 1) * 64)) + (((rank + 1 + i0 + (i_)) & 7) << 18)), "s"(lst + (l_)) : "memory")
        if (!slow) {
            SLOAD(0, 0);
            if (i0 < 6) SLOAD(1, 2048);
        }
        GAS unsigned* ctr = (GAS unsigned*)(p.myws + WS_CTR) + b * 32 + wave;
        asm volatile("global_load_dword v199, %0, %1 sc1\n\ts_load_dword s94, %2, 0x0\n\ts_load_dword s95, %3, 0x0"
                     :: "v"(0), "s"(ctr), "s"(p.sx), "s"(p.sw) : "memory", "v199", "s94", "s95");
        asm volatile("s_barrier" ::: "memory");
        if (both) {
            SLOAD(4, 8192);
            if (!h) SLOAD(5, 10240);
        }
        if ((!(b & 1) || xg == 7) && (cls == 1 || cls == 2)) {
            v4i wr;
            wr.x = (int)(size_t)p.w; wr.y = (int)((size_t)p.w >> 32) & 0xffff; wr.z = GK * GN; wr.w = 0x00020000;
            const int wo = lane * GN + TILE_COL(xg * 4 + 3), ws = (((idx - 8) * 2 + h) * 128) * GN;
            asm volatile("s_nop 4\n\tbuffer_load_dword v200, %0, %1, %2 offen\n\tbuffer_load_dword v200, %0, %1, %3 offen"
                         :: "v"(wo), "s"(wr), "s"(ws), "s"(ws + 64 * GN) : "memory", "v200");
            WAIT_VM(2);
        } else
        WAIT_VM(0);
        unsigned callr;
        asm volatile("v_readfirstlane_b32 %0, v199" : "=s"(callr) :: "memory");
        v4u d[4][2];
#pragma unroll
        for (int i = 0; i < 2; ++i) {
            d[i][0] = *(LAS v4u*)(L8 + i * 2048 + rv);
            d[i][1] = *(LAS v4u*)(L8 + i * 2048 + (rv ^ 16));
        }
        const auto d0 = __builtin_amdgcn_make_buffer_rsrc((void*)pe0, 0, (int)WS_END, 0x00020000);
        const auto d1 = __builtin_amdgcn_make_buffer_rsrc((void*)pe1, 0, (int)WS_END, 0x00020000);
        const unsigned call = callr + 1u;
        const int bo = (int)WS_A + (int)(call % 3u + 1u) * A_PAR;
        const int dso = bo + (rg * NREM + rd * 7 + 6 - i0) * FRAG;
#pragma unroll
        for (int i = 0; i < 2; ++i) {
            d[i][0].x = max(~d[i][0].x, 1u);
            d[i][1].x = max(~d[i][1].x, 1u);
        }
#define BURST() do { \
            store16<WT>(d[0][0], d0, lane * 16, dso); \
            store16<WT>(d[0][1], d0, lane * 16 + 1024, dso); \
            if (i0 < 6) { \
                store16<WT>(d[1][0], d1, lane * 16, dso - FRAG); \
                store16<WT>(d[1][1], d1, lane * 16 + 1024, dso - FRAG); \
            } } while (0)
        if (b < HOLD) BURST();
        if (b < HOLD) asm volatile("s_barrier" ::: "memory");
        float scale;
        asm volatile("s_waitcnt lgkmcnt(0)\n\tv_mov_b32 %0, s94\n\tv_mul_f32 %0, s95, %0" : "=v"(scale));
        *(volatile LAS float*)(sync + 16) = scale;
        *(volatile LAS unsigned*)(sync + (lane & 1) * 2) = (unsigned)bo;
        if (lane == 0) __hip_atomic_store(ctr, call, __ATOMIC_RELAXED, __HIP_MEMORY_SCOPE_AGENT);
        if (!slow) {
            ENTRY_TIME(tentry);
            const unsigned tstart = (unsigned)tentry;
            if (both) {
#pragma unroll
                for (int i = 2; i < 4; ++i) {
                    d[i][0] = *(LAS v4u*)(L8 + (i + 2) * 2048 + rv);
                    d[i][1] = *(LAS v4u*)(L8 + (i + 2) * 2048 + (rv ^ 16));
                    d[i][0].x = max(~d[i][0].x, 1u);
                    d[i][1].x = max(~d[i][1].x, 1u);
                }
            }
            const auto d2 = __builtin_amdgcn_make_buffer_rsrc((void*)p.peer[4 + 2 * h], 0, (int)WS_END, 0x00020000);
            const auto d3 = __builtin_amdgcn_make_buffer_rsrc((void*)p.peer[5], 0, (int)WS_END, 0x00020000);
            const unsigned due = SLOT_T0 + ((unsigned)(b >> 1) * SLOT_Q >> 4);
            while ((unsigned)__builtin_amdgcn_s_memrealtime() - tstart < due) __builtin_amdgcn_s_sleep(1);
            if (b >= HOLD) BURST();
            if (both) {
                store16<WT>(d[2][0], d2, lane * 16, dso - 4 * FRAG);
                store16<WT>(d[2][1], d2, lane * 16 + 1024, dso - 4 * FRAG);
                if (!h) {
                    store16<WT>(d[3][0], d3, lane * 16, dso - 5 * FRAG);
                    store16<WT>(d[3][1], d3, lane * 16 + 1024, dso - 5 * FRAG);
                }
            }
            if (cls < 3) {
                const auto zr = __builtin_amdgcn_make_buffer_rsrc((void*)(p.myws + WS_A), 0, (int)WS_END, 0x00020000);
                const v4u z = {0u, 0u, 0u, 0u};
                const int zo = (int)((call + 2u) % 3u + 1u) * A_PAR + (b * 28 / 3 + h * 5) * 1024;
#pragma unroll
                for (int u = 0; u < 5; ++u) store16<WT>(z, zr, lane * 16, zo + u * 1024);
            }
        }
    } else {
        __builtin_amdgcn_s_setprio(2);
        const int rank = p.rank;
        const int cw = wave - 4;
        const int mh = cw & 1, kp = cw >> 1;
        const int rg = mb * 2 + mh;
        const int lb = (int)(size_t)L8;
        const int stg = lb + (kp * RING + RING - 2 + mh) * SLOT;
        {
            int xv0 = (lane >> 3) * SH + (((lane & 7) ^ ((lane >> 4) & 3)) * 16);
            int xv1 = (xv0 ^ 64) + 8 * SH - 1024;
            int xv2 = xv0 + 0x1800, xv3 = xv1 + 0x1800;
            asm volatile("" : "+v"(xv0), "+v"(xv1), "+v"(xv2), "+v"(xv3));
            v4i xr;
            xr.x = (int)(size_t)p.x; xr.y = (int)((size_t)p.x >> 32) & 0xffff; xr.z = 4096 * SH; xr.w = 0x00020000;
            const int xsoff = (rank * SH + rg * 32) * SH + kp * 128;
#define XLD(so_) \
        "buffer_load_dwordx4 %0, %4, " so_ " offen lds\n\t" \
        "buffer_load_dwordx4 %1, %4, " so_ " offen offset:1024 lds\n\t" \
        "buffer_load_dwordx4 %2, %4, " so_ " offen offset:2048 lds\n\t" \
        "buffer_load_dwordx4 %3, %4, " so_ " offen offset:3072 lds\n\t"
            if (b < 64) asm volatile("s_barrier" ::: "memory");
            asm volatile("s_mov_b32 m0, %7\n\t"
                         "s_nop 0\n\t"
                         XLD("%5")
                         "s_add_u32 m0, m0, 4096\n\t"
                         "s_nop 0\n\t"
                         XLD("%6")
                         :: "v"(xv0), "v"(xv1), "v"(xv2), "v"(xv3), "s"(xr), "s"(xsoff), "s"(xsoff + 256), "s"(stg) : "memory", "scc");
            if (b < HOLD || b >= 64) asm volatile("s_barrier" ::: "memory");
        }
        int lane2 = lane;
        asm volatile("" : "+v"(lane2));
#define lane lane2
        const unsigned char* ab = p.myws + WS_A + rg * NREM * FRAG;
        v4i ar;
        ar.x = (int)(size_t)ab; ar.y = (int)((size_t)ab >> 32) & 0xffff; ar.z = NREM * FRAG; ar.w = 0x00020000;
        const int avoff = lane * 16;
        const int rv = stg + (lane & 31) * 128 + ((((lane >> 5) * 2) ^ ((lane >> 1) & 7)) * 16);
        const int g = lane >> 4, q = (lane & 15) >> 1;
        const int rdl = lb + kp * RING * SLOT + ((g >> 1) * 32 + q) * 128 + (g & 1) * 16 + (lane & 1) * 8;
        const int sw = (q >> 1) ^ (2 * kp);
        const int rd0 = rdl + ((0 ^ sw) * 32), rd1 = rdl + ((1 ^ sw) * 32);
        const int rd2 = rdl + ((2 ^ sw) * 32), rd3 = rdl + ((3 ^ sw) * 32);
        const int a_syn = lb + L_SYNC + 8 * kp, a_cons = lb + L_SYNC + 32 + 4 * cw;
        int aoff = kp * FRAG;
        static_assert((32 - RING) % RING + 4 <= RING, "the four staging slots are adjacent");
        const int a_scale = lb + L_SYNC + 64;
        const int s0 = ((32 - RING) % RING + 2 * mh) * SLOT;
        int e_w[4], e_r0[2], e_r1[2];
#pragma unroll
        for (int gq = 0; gq < 4; ++gq) e_w[gq] = lb + kp * RING * SLOT + s0 + (lane & 31) * 256 + ((((lane >> 5) ^ (lane & 7)) * 16) ^ (gq * 32));
#pragma unroll
        for (int i = 0; i < 2; ++i) {
            const int ra = (lane >> 4) * 256 + ((((lane & 15) ^ (lane >> 4)) * 16) ^ (i * 64));
            e_r0[i] = lb + kp * RING * SLOT + s0 + ra;
            e_r1[i] = lb + (kp ^ 1) * RING * SLOT + s0 + SLOT + ra;
        }
        LAS unsigned* e_sib = sync + 8 + (cw ^ 1);
        LAS unsigned* e_par = sync + 12 + (cw ^ 2);
        int e_ov = (lane >> 4) * GN * 4 + (lane & 15) * 16;
        int e_os = (rg * 32 * GN + TILE_COL(xg * 4 + cls) + kp * 64) * 4;
        float* e_out = p.out;
        asm volatile("" : "+v"(e_w[0]), "+v"(e_w[1]), "+v"(e_w[2]), "+v"(e_w[3]), "+v"(e_r0[0]), "+v"(e_r0[1]), "+v"(e_r1[0]), "+v"(e_r1[1]),
                          "+v"(e_sib), "+v"(e_par), "+v"(e_ov));
        asm volatile("" : "+s"(e_os));
        asm volatile("" : "+s"(e_out));
#undef lane
        v16f acc0, acc1, acc2, acc3;
        v16i clob0, clob1, clob2, clob3, clob4;
        int va0, va1, va2, va3, vt2, vt3;
        int s_need, s_wslot, s_wrdy, s_t0, s_t1;
        unsigned long long s_ret, s_now;
        ENTRY_TIME(tentry);
        const unsigned tlo = (unsigned)tentry;
#define AREG(s_) "v[160+8*" #s_ ":167+8*" #s_ "]"
#define ALO(s_) "v[160+8*" #s_ ":163+8*" #s_ "]"
#define AHI(s_) "v[164+8*" #s_ ":167+8*" #s_ "]"
#define LDLO(p_, so_) "buffer_load_dwordx4 " ALO(p_) ", %[avoff], s[48:51], " so_ " offen sc0 sc1\n\t"
#define LDHI(p_, so_) "buffer_load_dwordx4 " AHI(p_) ", %[avoff], s[48:51], " so_ " offen offset:1024 sc0 sc1\n\t"
#define RDB(base_, va_) \
        "ds_read_b64_tr_b8 v[" #base_ ":" #base_ "+1], " va_ "\n\t" \
        "ds_read_b64_tr_b8 v[" #base_ "+2:" #base_ "+3], " va_ " offset:1024\n\t" \
        "ds_read_b64_tr_b8 v[" #base_ "+4:" #base_ "+5], " va_ " offset:2048\n\t" \
        "ds_read_b64_tr_b8 v[" #base_ "+6:" #base_ "+7], " va_ " offset:3072\n\t"
#define POS(s_, lo_, hi_, last_) \
        ".Lpos" #s_ "_%=:\n\t" \
        "s_waitcnt vmcnt(%[vmd])\n" \
        ".Lchk" #s_ "_%=:\n\t" \
        "v_min_u32 %[vt3], v[160+8*" #s_ "], v[164+8*" #s_ "]\n\t" \
        "v_cmp_eq_u32 vcc, 0, %[vt3]\n\t" \
        "v_not_b32 v[160+8*" #s_ "], v[160+8*" #s_ "]\n\t" \
        "v_not_b32 v[164+8*" #s_ "], v[164+8*" #s_ "]\n\t" \
        "s_cmp_eq_u64 vcc, 0\n\t" \
        "s_cselect_b32 %[t0], %[wrdy], 0\n\t" \
        "s_cmp_lt_u32 %[t0], %[need]\n\t" \
        "s_waitcnt lgkmcnt(10)\n\t" \
        "s_cbranch_scc1 .Lslow" #s_ "_%=\n" \
        ".Lcomp" #s_ "_%=:\n\t" \
        "v_mfma_f32_32x32x64_f8f6f4 v[64:79], v[128:135], " AREG(s_) ", v[64:79] cbsz:1 blgp:1\n\t" \
        lo_ \
        RDB(128, "%[va0]") \
        "s_add_u32 %[wslot], %[wslot], %[slotb]\n\t" \
        "s_cmp_eq_u32 %[wslot], %[ringb]\n\t" \
        "s_cselect_b32 %[wslot], 0, %[wslot]\n\t" \
        "v_add_u32 %[va0], %[wslot], %[rd0]\n\t" \
        "s_waitcnt lgkmcnt(10)\n\t" \
        "v_mfma_f32_32x32x64_f8f6f4 v[80:95], v[136:143], " AREG(s_) ", v[80:95] cbsz:1 blgp:1\n\t" \
        hi_ \
        RDB(136, "%[va1]") \
        "ds_read_b32 v201, %[asyn] offset:4\n\t" \
        "ds_write_b32 %[acons], %[vt2]\n\t" \
        "v_add_u32 %[va1], %[wslot], %[rd1]\n\t" \
        "s_waitcnt lgkmcnt(10)\n\t" \
        "v_mfma_f32_32x32x64_f8f6f4 v[96:111], v[144:151], " AREG(s_) ", v[96:111] cbsz:1 blgp:1\n\t" \
        RDB(144, "%[va2]") \
        "v_add_u32 %[va2], %[wslot], %[rd2]\n\t" \
        "v_add_u32 %[vt2], 1, %[vt2]\n\t" \
        "s_add_u32 %[need], %[need], 1\n\t" \
        "s_waitcnt lgkmcnt(5)\n\t" \
        "v_readfirstlane_b32 %[wrdy], v201\n\t" \
        "v_mfma_f32_32x32x64_f8f6f4 v[112:127], v[152:159], " AREG(s_) ", v[112:127] cbsz:1 blgp:1\n\t" \
        last_ \
        RDB(152, "%[va3]") \
        "v_add_u32 %[va3], %[wslot], %[rd3]\n"
#define POSL(s_, p_, last_) POS(s_, LDLO(p_, "%[aoff]"), LDHI(p_, "%[aoff]") "s_add_u32 %[aoff], %[aoff], 0x1000\n\t", last_)
#define POS0(s_) POS(s_, "", "", "")
#define LAST "s_cmp_eq_u32 %[need], 34\n\ts_cbranch_scc1 .Ldone_%=\n\t"
#define AGAIN(n_) "s_add_u32 %[t1], %[t1], 0x1000\n\t" LDLO(n_, "%[t1]") LDHI(n_, "%[t1]")
#define SIDE(s_, more_) \
        ".Lslow" #s_ "_%=:\n\t" \
        "s_cbranch_vccnz .Lretry" #s_ "_%=\n" \
        ".Lwspin" #s_ "_%=:\n\t" \
        "ds_read_b32 v201, %[asyn] offset:4\n\t" \
        "s_waitcnt lgkmcnt(0)\n\t" \
        "v_readfirstlane_b32 %[wrdy], v201\n\t" \
        "s_cmp_lt_u32 %[wrdy], %[need]\n\t" \
        "s_cbranch_scc1 .Lwspin" #s_ "_%=\n\t" \
        "s_branch .Lcomp" #s_ "_%=\n" \
        ".Lretry" #s_ "_%=:\n\t" \
        "s_getpc_b64 %[ret]\n\t" \
        "s_branch .Lagain_%=\n\t" \
        "s_cbranch_scc1 .Lwspin" #s_ "_%=\n\t" \
        LDLO(s_, "%[t1]") LDHI(s_, "%[t1]") \
        "s_waitcnt vmcnt(0)\n\t" \
        "v_min_u32 %[vt3], v[160+8*" #s_ "], v[164+8*" #s_ "]\n\t" \
        "v_cmp_eq_u32 vcc, 0, %[vt3]\n\t" \
        "s_nop 1\n\t" \
        "s_cbranch_vccnz .Lretry" #s_ "_%=\n\t" \
        more_ \
        "s_branch .Lchk" #s_ "_%=\n"
#define XRD(d_, off_) \
        "ds_read_b128 v[" #d_ ":" #d_ "+3], %[rv] offset:" #off_ "\n\t" \
        "ds_read_b128 v[" #d_ "+4:" #d_ "+7], %[va1] offset:" #off_ "\n\t" \
        "ds_read_b128 v[" #d_ "+8:" #d_ "+11], %[va2] offset:" #off_ "\n\t" \
        "ds_read_b128 v[" #d_ "+12:" #d_ "+15], %[va3] offset:" #off_ "\n\t"
#define XINV(r_) "v_not_b32 v" #r_ ", v" #r_ "\n\tv_max_u32 v" #r_ ", 1, v" #r_ "\n\t"
#define Z8(b_) \
        "v_mov_b64 v[" #b_ ":" #b_ "+1], 0\n\tv_mov_b64 v[" #b_ "+2:" #b_ "+3], 0\n\t" \
        "v_mov_b64 v[" #b_ "+4:" #b_ "+5], 0\n\tv_mov_b64 v[" #b_ "+6:" #b_ "+7], 0\n\t"
        asm volatile(
            "v_xor_b32 %[va1], 16, %[rv]\n\t"
            "v_xor_b32 %[va2], 64, %[rv]\n\t"
            "v_xor_b32 %[va3], 0x50, %[rv]\n\t"
            Z8(64) Z8(72) Z8(80) Z8(88) Z8(96) Z8(104) Z8(112) Z8(120)
            "s_mov_b32 %[need], 2\n\t"
            "s_mov_b32 %[wslot], %[slotb]\n\t"
            "v_mov_b32 %[vt2], 1\n\t"
            "s_waitcnt vmcnt(4)\n\t"
            XRD(160, 0)
            "s_waitcnt vmcnt(0)\n\t"
            XRD(176, 4096)
            ".Lfirst_%=:\n\t"
            "ds_read_b64 v[200:201], %[asyn]\n\t"
            "s_waitcnt lgkmcnt(0)\n\t"
            "v_readfirstlane_b32 %[t1], v200\n\t"
            "v_readfirstlane_b32 %[wrdy], v201\n\t"
            "s_min_u32 %[t0], %[t1], %[wrdy]\n\t"
            "s_cmp_eq_u32 %[t0], 0\n\t"
            "s_cbranch_scc1 .Lfirst_%=\n\t"
            "s_add_u32 s48, s48, %[t1]\n\t"
            "s_addc_u32 s49, s49, 0\n\t"
            XINV(160) XINV(164) XINV(168) XINV(172) XINV(176) XINV(180) XINV(184) XINV(188)
            RDB(128, "%[rd0]") RDB(136, "%[rd1]") RDB(144, "%[rd2]") RDB(152, "%[rd3]")
            "ds_write_b32 %[acons], %[vt2]\n\t"
            "v_add_u32 %[va0], %[wslot], %[rd0]\n\t"
            "v_add_u32 %[va1], %[wslot], %[rd1]\n\t"
            "v_add_u32 %[va2], %[wslot], %[rd2]\n\t"
            "v_add_u32 %[va3], %[wslot], %[rd3]\n\t"
            "s_waitcnt lgkmcnt(1)\n"
            POS0(0) POSL(1, 4, "") POSL(2, 1, "") POSL(3, 2, LAST) POSL(4, 3, "")
            "s_branch .Lpos1_%=\n"
            SIDE(0, "") SIDE(1, AGAIN(2) AGAIN(3)) SIDE(2, AGAIN(3) AGAIN(4)) SIDE(3, AGAIN(4) AGAIN(1)) SIDE(4, AGAIN(1) AGAIN(2))
            ".Lagain_%=:\n\t"
            "ds_write_b32 %[acons], %[vt2]\n\t"
            "s_sleep %[rslp]\n\t"
            "s_memrealtime %[now]\n\t"
            "s_add_u32 s40, s40, 4\n\t"
            "s_addc_u32 s41, s41, 0\n\t"
            "s_sub_u32 %[t1], %[aoff], %[back]\n\t"
            "s_waitcnt lgkmcnt(0)\n\t"
            "s_sub_u32 %[t0], s42, %[tlo]\n\t"
            "s_cmp_gt_u32 %[t0], %[dl]\n\t"
            "s_cbranch_scc0 .Lback_%=\n\t"
            "v_mov_b32 %[vt3], 0x7fc00000\n\t"
            "ds_write_b32 %[ascale], %[vt3]\n"
            ".Lback_%=:\n\t"
            "s_setpc_b64 %[ret]\n"
            ".Ldone_%=:\n\t"
            "s_waitcnt lgkmcnt(0)\n\t"
            "s_nop 15\n\t"
            "s_nop 3"
            : "=&{v[64:79]}"(acc0), "=&{v[80:95]}"(acc1), "=&{v[96:111]}"(acc2), "=&{v[112:127]}"(acc3),
              "=&{v[128:143]}"(clob0), "=&{v[144:159]}"(clob1), "=&{v[160:175]}"(clob2), "=&{v[176:191]}"(clob3),
              "=&{v[192:207]}"(clob4),
              [va0] "=&v"(va0), [va1] "=&v"(va1), [va2] "=&v"(va2), [va3] "=&v"(va3), [vt2] "=&v"(vt2), [vt3] "=&v"(vt3),
              [ret] "=&{s[40:41]}"(s_ret), [now] "=&{s[42:43]}"(s_now), "+{s[48:51]}"(ar),
              [need] "=&s"(s_need), [wslot] "=&s"(s_wslot), [wrdy] "=&s"(s_wrdy), [t0] "=&s"(s_t0), [t1] "=&s"(s_t1),
              [aoff] "+&s"(aoff)
            : [avoff] "v"(avoff), [rd0] "v"(rd0), [rd1] "v"(rd1), [rd2] "v"(rd2), [rd3] "v"(rd3),
              [rv] "v"(rv), [asyn] "v"(a_syn), [acons] "v"(a_cons), [slotb] "i"(SLOT), [ringb] "i"(RING * SLOT),
              [rslp] "i"(4), [vmd] "i"(2 * (PDIST - 1)), [back] "i"(PDIST * 4096),
              [tlo] "s"(tlo), [dl] "i"((unsigned)DEADLINE_TICKS), [ascale] "v"(a_scale)
            : "memory", "scc", "vcc");
        asm volatile("" : "+v"(e_w[0]), "+v"(e_w[1]), "+v"(e_w[2]), "+v"(e_w[3]), "+v"(e_r0[0]), "+v"(e_r0[1]), "+v"(e_r1[0]), "+v"(e_r1[1]));
        {
            while (rfl(*(volatile LAS unsigned*)(e_sib)) < (unsigned)(36 - RING)) {}
            asm volatile("" ::: "memory");
#pragma unroll
            for (int gq = 0; gq < 4; ++gq) {
                *(LAS v4f*)(size_t)(e_w[gq] + SLOT) = v4f{acc2[4 * gq], acc2[4 * gq + 1], acc2[4 * gq + 2], acc2[4 * gq + 3]};
                *(LAS v4f*)(size_t)(e_w[gq] + SLOT + 128) = v4f{acc3[4 * gq], acc3[4 * gq + 1], acc3[4 * gq + 2], acc3[4 * gq + 3]};
            }
            asm volatile("" ::: "memory");
            *(volatile LAS unsigned*)(size_t)(a_cons + 16) = 1u;
            asm volatile("" ::: "memory");
#pragma unroll
            for (int gq = 0; gq < 4; ++gq) {
                *(LAS v4f*)(size_t)(e_w[gq]) = v4f{acc0[4 * gq], acc0[4 * gq + 1], acc0[4 * gq + 2], acc0[4 * gq + 3]};
                *(LAS v4f*)(size_t)(e_w[gq] + 128) = v4f{acc1[4 * gq], acc1[4 * gq + 1], acc1[4 * gq + 2], acc1[4 * gq + 3]};
            }
            v4f u0[8], u1[8];
#pragma unroll
            for (int i = 0; i < 8; ++i) u0[i] = *(LAS v4f*)(size_t)(e_r0[i & 1] + i * 1024);
            while (rfl(*(volatile LAS unsigned*)(e_par)) == 0u) {}
            asm volatile("" ::: "memory");
            const float sc = *(volatile LAS float*)(size_t)(a_scale);
#pragma unroll
            for (int i = 0; i < 8; ++i) u1[i] = *(LAS v4f*)(size_t)(e_r1[i & 1] + i * 1024);
            const auto orr = __builtin_amdgcn_make_buffer_rsrc((void*)e_out, 0, SH * GN * 4, 0x00020000);
#pragma unroll
            for (int i = 0; i < 8; ++i) {
                const v4f y = __builtin_elementwise_maximum((u0[i] + u1[i]) * sc, v4f{0.f, 0.f, 0.f, 0.f});
                store16<WT>(__builtin_bit_cast(v4u, y), orr, e_ov, e_os + i * 4 * GN * 4);
            }
        }
        asm volatile("" :: "{v[160:175]}"(clob2), "{v[176:191]}"(clob3), "{v[192:207]}"(clob4));
    }
}

extern "C" void kernel_launch(int rank, int nranks, ncclComm_t comm,
                              hipStream_t stream,
                              void* const* d_in, const int* in_sizes, int n_in,
                              void* d_out, int out_size,
                              const int* in_dtypes, int out_dtype,
                              void* const* d_out_all,
                              void* const* d_ws_all, size_t ws_bytes) {
    if (nranks != NR || (unsigned)rank >= (unsigned)NR || ws_bytes < WS_END || n_in < 4 ||
        in_sizes[0] != 4096 * SH || in_sizes[1] != GK * GN || out_size != SH * GN) {
        fprintf(stderr, "unexpected problem geometry\n");
        return;
    }
    HIP_CHECK(hipFuncSetAttribute((const void*)a2a_gemm_kernel, hipFuncAttributeMaxDynamicSharedMemorySize, LDS_BYTES));
    Params p;
    p.x = (const unsigned char*)d_in[0];
    p.w = (const unsigned char*)d_in[1];
    p.sx = (const float*)d_in[2];
    p.sw = (const float*)d_in[3];
    p.out = (float*)d_out;
    p.myws = (unsigned char*)d_ws_all[rank];
    p.rank = rank;
    p.pad = 0;
    for (int i = 0; i < NR; ++i) p.peer[i] = (unsigned char*)d_ws_all[(rank + 1 + i) & 7];
    a2a_gemm_kernel<<<NWG, NTHREADS, LDS_BYTES, stream>>>(p);
    HIP_CHECK(hipGetLastError());
}
```
